```python
import numpy as np

kernel_name = "gpu_ag_gemm_m2048_k2048_n2048_bf16_gelu"


ABSMAX_RTOL = 0.008

COMPUTE_DTYPE = "bfloat16"
EPILOGUE = "gelu"
IN_QUANT = None
OUT_QUANT = None
IN_DTYPES = {"x": "bfloat16", "w_mat": "bfloat16"}
WORLD_SIZE = 8

def _to_bf16(x):
    u = np.ascontiguousarray(x, dtype=np.float32).view(np.uint32)
    return ((u + 0x7FFF + ((u >> 16) & 1)) & 0xFFFF0000).view(np.float32)

def setup_inputs():

    x = _to_bf16(np.random.randn(2048, 2048).astype(np.float32))
    w_mat = _to_bf16((np.random.randn(2048, 2048) / np.sqrt(2048)).astype(np.float32))
    return {"x": x, "w_mat": w_mat}

def _epilogue(y):
    c = 0.7978845608028654
    return (0.5 * y * (1.0 + np.tanh(c * (y + 0.044715 * y ** 3)))).astype(np.float32)

def reference(x, w_mat):
    y = (x @ w_mat).astype(np.float32)
    return _epilogue(y)
```

```cpp
#include <hip/hip_runtime.h>
#include <rccl/rccl.h>
#include <cstdio>
#include <cstdint>
#include <cstring>

namespace agk {

typedef unsigned u32x4 __attribute__((ext_vector_type(4)));
typedef unsigned u32x2 __attribute__((ext_vector_type(2)));
typedef float f32x4 __attribute__((ext_vector_type(4)));
typedef __bf16 bf16x8 __attribute__((ext_vector_type(8)));
typedef unsigned long long u64;
#define GAS __attribute__((address_space(1)))

constexpr int NRANK = 8;
constexpr int MDIM = 2048, KDIM = 2048, NLOC = 256, MLOC = 256;
#define PBLK 11
constexpr int P = PBLK;
constexpr int PO = 16 - P;
constexpr int NSLOT = 4 * P + 4 * PO;
static_assert(NSLOT == 64, "");
#define POLL_SLEEP 1
#define FSTRIDE 32
constexpr int FS = FSTRIDE;

constexpr int NSEND_WG = 28;
constexpr int NCOMP_WG = 192;
constexpr int NWG = NSEND_WG + NCOMP_WG;
constexpr int NTHR = 512;

constexpr size_t OFF_CNT = 0;
constexpr size_t CNT_BYTES = (size_t)NWG * 8 * 64;
constexpr size_t OFF_TMO = 128 << 10;
constexpr size_t OFF_XFLAG = 132 << 10;
constexpr size_t OFF_WFLAG = OFF_XFLAG + (size_t)1024 * FS * 4;
constexpr size_t OFF_OFLAG = OFF_WFLAG + (size_t)512 * FS * 4;
constexpr size_t OFF_FLAG_END = OFF_OFLAG + (size_t)512 * FS * 4;
constexpr size_t OFF_DBG = 512 << 10;
constexpr size_t OFF_STAGE = 1 << 20;
constexpr size_t X_BYTES = (size_t)NSLOT * 65536;
constexpr size_t W_BYTES = (size_t)8 * 262144;
constexpr size_t STAGE_BYTES = X_BYTES + W_BYTES;
constexpr size_t WS_NEED = OFF_STAGE + 2 * STAGE_BYTES;
static_assert(CNT_BYTES <= OFF_TMO && OFF_FLAG_END <= OFF_DBG, "");
constexpr u64 DEADLINE = 50000000ull;

struct Args {
  char* ws[NRANK];
  char* out[NRANK];
  const char* x;
  const char* w;
  int rank;
  int pad;
};

__device__ __forceinline__ void st16_wt(char* p, u32x4 v, int) {
  asm volatile("global_store_dwordx4 %0, %1, off sc0 sc1\n\ts_nop 1" ::"v"(p), "v"(v) : "memory");
}
template <int OFF>
__device__ __forceinline__ void st16_wt_off(char* p, u32x4 v) {
  asm volatile("global_store_dwordx4 %0, %1, off offset:%c2 sc0 sc1\n\ts_nop 1" ::"v"(p), "v"(v), "i"(OFF) : "memory");
}
__device__ __forceinline__ void st_flag(char* p, unsigned v) {
  asm volatile("global_store_dword %0, %1, off sc0 sc1\n\ts_nop 1" ::"v"(p), "v"(v) : "memory");
}
#define WAIT_VM0() asm volatile("s_waitcnt vmcnt(0)" ::: "memory")
#define PIN(x) asm volatile("" : "+v"(x))

__device__ __forceinline__ u64 rt() { return __builtin_amdgcn_s_memrealtime(); }

__device__ __forceinline__ bool wait_flags(const char* fp, unsigned call, unsigned* tmo) {
  const u64 t0 = rt();
  for (;;) {
    const unsigned v = __hip_atomic_load((const GAS unsigned*)fp, __ATOMIC_RELAXED, __HIP_MEMORY_SCOPE_SYSTEM);
    const unsigned t = __hip_atomic_load((GAS unsigned*)tmo, __ATOMIC_RELAXED, __HIP_MEMORY_SCOPE_AGENT);
    if (__all(v >= call)) { __builtin_amdgcn_fence(__ATOMIC_ACQUIRE, "wavefront"); return t != call; }
    if (t == call || rt() - t0 > DEADLINE) {
      __hip_atomic_store((GAS unsigned*)tmo, call, __ATOMIC_RELAXED, __HIP_MEMORY_SCOPE_AGENT);
      return false;
    }
    __builtin_amdgcn_s_sleep(POLL_SLEEP);
  }
}

__device__ __forceinline__ void load_x_unit(const char* x, int blk, int u, int lane, u32x4 (&v)[4]) {
  const char* p = x + (size_t)(blk * 16 + (lane & 15)) * (KDIM * 2) + (u * 128 + (lane >> 4) * 8) * 2;
#pragma unroll
  for (int i = 0; i < 4; ++i) v[i] = *(const GAS u32x4*)(p + i * 64);
}
__device__ __forceinline__ void load_w_raw(const char* w, int cgj, int ks, int lane, u32x2 (&r)[8]) {
  const int c = lane & 15, q = lane >> 4;
  const char* p = w + (size_t)(ks * 32 + q * 8) * (NLOC * 2) + (cgj * 64 + 8 * (c & 7) + 4 * (c >> 3)) * 2;
#pragma unroll
  for (int e = 0; e < 8; ++e) r[e] = *(const GAS u32x2*)(p + e * (NLOC * 2));
}
__device__ __forceinline__ void w_transpose(const u32x2 (&r)[8], u32x4 (&v)[4]) {
#pragma unroll
  for (int jj = 0; jj < 4; ++jj)
#pragma unroll
    for (int i = 0; i < 4; ++i)
      v[jj][i] = __builtin_amdgcn_perm(r[2 * i + 1][jj >> 1], r[2 * i][jj >> 1], (jj & 1) ? 0x07060302u : 0x05040100u);
}
__device__ __forceinline__ void store_unit(char* dst, const u32x4 (&v)[4]) {
  st16_wt_off<0>(dst, v[0]);
  st16_wt_off<1024>(dst, v[1]);
  st16_wt_off<2048>(dst, v[2]);
  st16_wt_off<3072>(dst, v[3]);
}

__device__ __forceinline__ float gelu_f(float y) {
  const float c = 0.7978845608028654f;
  const float k2 = -2.0f * 1.4426950408889634f * c;
  const float t = y * (1.0f + 0.044715f * y * y);
  const float e = __builtin_amdgcn_exp2f(k2 * t);
  return y * __builtin_amdgcn_rcpf(1.0f + e);
}
__device__ __forceinline__ unsigned pack_bf16(float lo, float hi) {
  typedef __bf16 bf2 __attribute__((ext_vector_type(2)));
  typedef float f2 __attribute__((ext_vector_type(2)));
  f2 f = {lo, hi};
  return __builtin_bit_cast(unsigned, __builtin_convertvector(f, bf2));
}

#define STAMP(idx) do {} while (0)

__device__ __forceinline__ int seq_to_slot(int q, int idx_self) {
  if (q < P) return idx_self * P + q;
  q -= P;
  constexpr int WST = 4;
  for (int s = 0; s < 32; ++s) {
    if (s < P) {
      if (q < 3) { int ia = q + (q >= idx_self ? 1 : 0); return ia * P + s; }
      q -= 3;
    }
    const int j = s - WST;
    if (j >= 0 && j < PO) {
      if (q < 4) return 4 * P + q * PO + j;
      q -= 4;
    }
  }
  return 0;
}

__global__ void __launch_bounds__(NTHR, 2) ag_gemm_kernel(Args a) {
  __shared__ __attribute__((aligned(16))) char lds[2 * 8 * 4096];
  const int tid = threadIdx.x, lane = tid & 63;
  const int wv = __builtin_amdgcn_readfirstlane(tid >> 6);
  const int bid = blockIdx.x;
  const int rank = a.rank, type = rank & 1, idx = rank >> 1;
  char* const myws = a.ws[rank];
  unsigned* const tmo = (unsigned*)(myws + OFF_TMO);
  STAMP(0);
  auto get_call = [&]() -> unsigned {
    asm volatile("" ::: "memory");
    unsigned ticket = 0;
    if (lane == 0)
      ticket = __hip_atomic_fetch_add((GAS unsigned*)(myws + OFF_CNT + (size_t)(bid * 8 + wv) * 64), 1u, __ATOMIC_RELAXED, __HIP_MEMORY_SCOPE_AGENT);
    return (unsigned)__builtin_amdgcn_readfirstlane(ticket) + 1u;
  };

  if (bid < NSEND_WG) {
    const int link = bid >> 2;
    if (wv >= 4) {
      const unsigned call = get_call();
      if (bid == 0 && wv == 4) {
        bool ok = true;
        {
          const char* fp = myws + OFF_OFLAG + (size_t)lane * FS * 4;
          const u64 t0 = rt();
          for (;;) {
            unsigned v[8];
#pragma unroll
            for (int i = 0; i < 8; ++i)
              v[i] = __hip_atomic_load((const GAS unsigned*)(fp + (size_t)i * 64 * FS * 4), __ATOMIC_RELAXED, __HIP_MEMORY_SCOPE_SYSTEM);
            const unsigned t = __hip_atomic_load((GAS unsigned*)tmo, __ATOMIC_RELAXED, __HIP_MEMORY_SCOPE_AGENT);
            unsigned mn = v[0];
#pragma unroll
            for (int i = 1; i < 8; ++i) mn = mn < v[i] ? mn : v[i];
            if (__all(mn >= call)) { ok = t != call; break; }
            if (t == call || rt() - t0 > DEADLINE) {
              __hip_atomic_store((GAS unsigned*)tmo, call, __ATOMIC_RELAXED, __HIP_MEMORY_SCOPE_AGENT);
              ok = false;
              break;
            }
            __builtin_amdgcn_s_sleep(1);
          }
        }
        if (!ok && lane == 0) {
          *(GAS unsigned short*)(a.out[rank] + (size_t)(rank * MLOC) * (NLOC * 2)) = 0x7fc0;
        }
        STAMP(15);
      }
      return;
    }
    const int j = (bid & 3) * 4 + wv;
    const bool same = link < 3;
    const int didx = same ? link + (link >= idx ? 1 : 0) : link - 3;
    const unsigned dest = (unsigned)(2 * didx + (same ? type : 1 - type));
    if (dest >= (unsigned)NRANK) { get_call(); return; }
    char* const dws = a.ws[dest];
    u32x4 v[4], vn[4];
    const int b0 = same ? 0 : P, nb = same ? P : PO;
    const int slot0 = same ? idx * P : 4 * P + idx * PO;
    load_x_unit(a.x, b0, j, lane, v);
    u32x2 r[8];
    if (!same) load_w_raw(a.w, didx, j, lane, r);
    const unsigned call = get_call();
    char* const stage = dws + OFF_STAGE + (size_t)(call & 1) * STAGE_BYTES;
    STAMP(1);
    if (!same) {
      char* wdst = stage + X_BYTES + (size_t)(4 + idx) * 262144 + lane * 16;
      char* wfl = dws + OFF_WFLAG + (size_t)((4 + idx) * 64) * FS * 4;
      for (int it = 0; it < 4; ++it) {
        const int ks = it * 16 + j;
        u32x4 t[4];
        w_transpose(r, t);
        const int ksn = it < 3 ? ks + 16 : ks;
        load_w_raw(a.w, didx, ksn, lane, r);
        store_unit(wdst + (size_t)ks * 4096, t);
        WAIT_VM0();
#pragma unroll
        for (int e = 0; e < 8; ++e) PIN(r[e]);
        if (lane == 0) st_flag(wfl + (size_t)ks * FS * 4, call);
      }
    }
#pragma unroll
    for (int i = 0; i < 4; ++i) PIN(v[i]);
    char* xdst = stage + (size_t)slot0 * 65536 + j * 4096 + lane * 16;
    char* xfl = dws + OFF_XFLAG + (size_t)(slot0 * 16 + j) * FS * 4;
    for (int b = 0; b < nb; ++b) {
      const int bn = b + 1 < nb ? b + 1 : b;
      load_x_unit(a.x, b0 + bn, j, lane, vn);
      store_unit(xdst + (size_t)b * 65536, v);
      WAIT_VM0();
#pragma unroll
      for (int i = 0; i < 4; ++i) PIN(vn[i]);
      if (lane == 0) st_flag(xfl + (size_t)b * 16 * FS * 4, call);
#pragma unroll
      for (int i = 0; i < 4; ++i) v[i] = vn[i];
    }
    STAMP(2);
    return;
  }

  const int cb = bid - NSEND_WG;
  const int xl = cb & 7, loc = cb >> 3;
  const int cg = loc & 7, set = loc >> 3;
  unsigned call;
  {
    const int u = wv * NCOMP_WG + cb;
    if (u < 256) {
      u32x2 r[8]; u32x4 t[4];
      load_w_raw(a.w, u >> 6, u & 63, lane, r);
      call = get_call();
      w_transpose(r, t);
      char* stage = myws + OFF_STAGE + (size_t)(call & 1) * STAGE_BYTES;
      store_unit(stage + X_BYTES + (size_t)u * 4096 + lane * 16, t);
      WAIT_VM0();
      if (lane == 0) st_flag(myws + OFF_WFLAG + (size_t)u * FS * 4, call);
    } else if (u < 256 + 16 * P) {
      const int xu = u - 256;
      u32x4 t[4];
      load_x_unit(a.x, xu >> 4, xu & 15, lane, t);
      call = get_call();
      char* stage = myws + OFF_STAGE + (size_t)(call & 1) * STAGE_BYTES;
      store_unit(stage + (size_t)(idx * P * 16 + xu) * 4096 + lane * 16, t);
      WAIT_VM0();
      if (lane == 0) st_flag(myws + OFF_XFLAG + (size_t)(idx * P * 16 + xu) * FS * 4, call);
    } else {
      call = get_call();
    }
  }
  STAMP(1);
  const char* stage_base = myws + OFF_STAGE + (size_t)(call & 1) * STAGE_BYTES;
  const __amdgpu_buffer_rsrc_t rs = __builtin_amdgcn_make_buffer_rsrc((void*)stage_base, 0, (int)STAGE_BYTES, 0x00020000);
  bool ok = wait_flags(myws + OFF_WFLAG + (size_t)(cg * 64 + wv * 8 + (lane & 7)) * FS * 4, call, tmo);
  STAMP(2);
  u32x4 wf[8][4];
  {
    const int soff = (int)X_BYTES + cg * 262144 + wv * 8 * 4096;
#pragma unroll
    for (int s = 0; s < 8; ++s)
#pragma unroll
      for (int jj = 0; jj < 4; ++jj)
        wf[s][jj] = __builtin_bit_cast(u32x4, __builtin_amdgcn_raw_buffer_load_b128(rs, lane * 16 + jj * 1024, soff + s * 4096, 17));
  }
  const unsigned odest = cg < 4 ? (unsigned)rank : (unsigned)(2 * (cg - 4) + (1 - type));
  const int ocol = cg < 4 ? cg * 64 : idx * 64;
  char* const obase = a.out[odest < (unsigned)NRANK ? odest : rank] + ocol * 2;
  char* const oflag = a.ws[odest < (unsigned)NRANK ? odest : rank] + OFF_OFLAG + (size_t)(idx * 128) * FS * 4;

  int it = 0;
  for (int q = xl + 8 * set; q < NSLOT; q += 24, ++it) {
    const int slot = seq_to_slot(q, idx);
    bool okb = wait_flags(myws + OFF_XFLAG + (size_t)(slot * 16 + wv * 2 + (lane & 1)) * FS * 4, call, tmo) && ok;
    STAMP(3 + it * 4);
    u32x4 af[8];
#pragma unroll
    for (int s = 0; s < 8; ++s)
      af[s] = __builtin_bit_cast(u32x4, __builtin_amdgcn_raw_buffer_load_b128(rs, lane * 16 + (s & 3) * 1024, slot * 65536 + wv * 8192 + (s >> 2) * 4096, 17));
    __builtin_amdgcn_sched_barrier(0);
    f32x4 acc[4];
#pragma unroll
    for (int jj = 0; jj < 4; ++jj) acc[jj] = f32x4{0.f, 0.f, 0.f, 0.f};
#pragma unroll
    for (int s = 0; s < 8; ++s)
#pragma unroll
      for (int jj = 0; jj < 4; ++jj)
        acc[jj] = __builtin_amdgcn_mfma_f32_16x16x32_bf16(__builtin_bit_cast(bf16x8, af[s]), __builtin_bit_cast(bf16x8, wf[s][jj]), acc[jj], 0, 0, 0);
    if (!okb) {
      const float qn = __builtin_nanf("");
#pragma unroll
      for (int jj = 0; jj < 4; ++jj) acc[jj] = f32x4{qn, qn, qn, qn};
    }
    char* buf = lds + (it & 1) * 32768;
    {
      const int qd = lane >> 4, c = lane & 15;
#pragma unroll
      for (int r = 0; r < 4; ++r) {
        f32x4 t = {acc[0][r], acc[1][r], acc[2][r], acc[3][r]};
        *(f32x4*)(buf + wv * 4096 + (4 * qd + r) * 256 + ((c ^ (8 * (r >> 1))) * 16)) = t;
      }
    }
    STAMP(4 + it * 4);
    __syncthreads();
    if (wv < 2) {
      const int row = 8 * wv + (lane >> 3), m = lane & 7;
      const int sw = 8 * ((row >> 1) & 1);
      f32x4 lo = {0.f, 0.f, 0.f, 0.f}, hi = {0.f, 0.f, 0.f, 0.f};
#pragma unroll
      for (int p = 0; p < 8; ++p) {
        lo += *(const f32x4*)(buf + p * 4096 + row * 256 + ((m ^ sw) * 16));
        hi += *(const f32x4*)(buf + p * 4096 + row * 256 + (((m + 8) ^ sw) * 16));
      }
      u32x4 o;
      o[0] = pack_bf16(gelu_f(lo[0]), gelu_f(lo[1]));
      o[1] = pack_bf16(gelu_f(lo[2]), gelu_f(lo[3]));
      o[2] = pack_bf16(gelu_f(hi[0]), gelu_f(hi[1]));
      o[3] = pack_bf16(gelu_f(hi[2]), gelu_f(hi[3]));
      int src, blk;
      if (slot < 4 * P) { const int ia = slot / P; src = 2 * ia + type; blk = slot - ia * P; }
      else { const int s2 = slot - 4 * P; const int ia = s2 / PO; src = 2 * ia + (1 - type); blk = P + s2 - ia * PO; }
      char* op = obase + (size_t)(src * MLOC + blk * 16 + row) * (NLOC * 2) + m * 16;
      if (cg < 4) {
        *(GAS u32x4*)op = o;
      } else {
        st16_wt(op, o, 0);
        WAIT_VM0();
        if (lane == 0) st_flag(oflag + (size_t)(slot * 2 + wv) * FS * 4, call);
      }
      STAMP(5 + it * 4);
    }
  }
  STAMP(14);
}

}

extern "C" void kernel_launch(int rank, int nranks, ncclComm_t comm, hipStream_t stream, void* const* d_in, const int* in_sizes,
                              int n_in, void* d_out, int out_size, const int* in_dtypes, int out_dtype, void* const* d_out_all,
                              void* const* d_ws_all, size_t ws_bytes) {
  using namespace agk;
  (void)comm; (void)in_dtypes; (void)out_dtype; (void)d_out;
  if (nranks != NRANK || n_in < 2 || rank < 0 || rank >= NRANK || ws_bytes < WS_NEED || in_sizes[0] != MLOC * KDIM || in_sizes[1] != KDIM * NLOC ||
      out_size != MDIM * NLOC) {
    fprintf(stderr, "kernel_launch: unexpected problem shape\n");
    return;
  }
  Args a;
  memset(&a, 0, sizeof a);
  for (int p = 0; p < NRANK; ++p) { a.ws[p] = (char*)d_ws_all[p]; a.out[p] = (char*)d_out_all[p]; }
  a.x = (const char*)d_in[0];
  a.w = (const char*)d_in[1];
  a.rank = rank;
  hipLaunchKernelGGL(ag_gemm_kernel, dim3(NWG), dim3(NTHR), 0, stream, a);
}
```

```cpp
#include <hip/hip_runtime.h>
#include <rccl/rccl.h>
#include <cstdio>
#include <cstdint>
#include <cstring>

namespace agk {

typedef unsigned u32x4 __attribute__((ext_vector_type(4)));
typedef unsigned u32x2 __attribute__((ext_vector_type(2)));
typedef unsigned u32x16 __attribute__((ext_vector_type(16)));
typedef float f32x4 __attribute__((ext_vector_type(4)));
typedef __bf16 bf16x8 __attribute__((ext_vector_type(8)));
typedef unsigned long long u64;
#define GAS __attribute__((address_space(1)))
#define LAS __attribute__((address_space(3)))

constexpr int NRANK = 8;
constexpr int MDIM = 2048, KDIM = 2048, NLOC = 256, MLOC = 256;
#define PBLK 11
constexpr int P = PBLK;
constexpr int PO = 16 - P;
constexpr int NSLOT = 4 * P + 4 * PO;
static_assert(NSLOT == 64, "");
#define POLL_SLEEP 1
#define SWV 8
#define LINKMAP 1
#define WSTEPS 4
constexpr int FS = 32;

constexpr int NSEND_WG = 28;
constexpr int SW = SWV;
constexpr int WPL = 4 * SW;
constexpr int NCOMP_WG = 192;
constexpr int NWG = NSEND_WG + NCOMP_WG + 1;
constexpr int NWV = 12;
constexpr int NTHR = NWV * 64;
constexpr int NSET = 12;
constexpr int MAXB = (NSLOT + NSET - 1) / NSET;
static_assert(SW <= NWV, "");
constexpr int LDS_SCR = MAXB * 16384;
constexpr int LDS_RDY = LDS_SCR + 4 * 4096;
constexpr int LDS_BYTES = LDS_RDY + 64;
static_assert(NWV * 4096 <= LDS_RDY, "");

constexpr size_t OFF_CNT = 0;
constexpr size_t CNT_BYTES = (size_t)NWG * NWV * 32;
constexpr size_t OFF_TMO = 128 << 10;
constexpr size_t OFF_TICKET = OFF_TMO + 1024;
constexpr size_t OFF_START = OFF_TMO + 2048;
constexpr size_t XFLAG_BYTES = (size_t)1024 * FS * 4, WFLAG_BYTES = (size_t)512 * FS * 4;
constexpr size_t OFF_XFLAG = 132 << 10;
constexpr size_t PAR_FLAG = XFLAG_BYTES + WFLAG_BYTES;
constexpr size_t OFF_WFLAG = OFF_XFLAG + XFLAG_BYTES;
constexpr size_t OFF_OFLAG = OFF_XFLAG + 2 * PAR_FLAG;
constexpr size_t OFF_FLAG_END = OFF_OFLAG + (size_t)512 * FS * 4;
constexpr size_t OFF_DBG = 14 << 20;
constexpr size_t OFF_STAGE = 1 << 20;
constexpr size_t X_BYTES = (size_t)NSLOT * 65536;
constexpr size_t W_BYTES = (size_t)8 * 262144;
constexpr size_t STAGE_BYTES = X_BYTES + W_BYTES;
constexpr size_t WS_NEED = OFF_STAGE + 2 * STAGE_BYTES;
static_assert(CNT_BYTES <= OFF_TMO && OFF_FLAG_END <= OFF_STAGE, "");
constexpr u64 DEADLINE = 50000000ull;

constexpr unsigned META_RAW = 0x80000000u;

struct Args {
  char* link_ws[8];
  char* obase[16];
  char* oflag[16];
  char* all_ws[8];
  char* myws;
  char* myout;
  const char* x;
  const char* w;
  int rank;
  int pad;
};

template <int OFF>
__device__ __forceinline__ void st16_wt(char* p, u32x4 v) {
  asm volatile("global_store_dwordx4 %0, %1, off offset:%c2 sc0 sc1\n\ts_nop 1" ::"v"(p), "v"(v), "i"(OFF) : "memory");
}
__device__ __forceinline__ void st4_wt(char* p, unsigned v) {
  asm volatile("global_store_dword %0, %1, off sc0 sc1\n\ts_nop 1" ::"v"(p), "v"(v) : "memory");
}
__device__ __forceinline__ void st8_wt(char* p, u32x2 v) {
  asm volatile("global_store_dwordx2 %0, %1, off sc0 sc1\n\ts_nop 1" ::"v"(p), "v"(v) : "memory");
}
__device__ __forceinline__ unsigned ticket_issue(char* p) {
  unsigned r = 0, one = 1;
  u64 save;
  asm volatile("s_mov_b64 %1, exec\n\ts_mov_b64 exec, 1\n\tglobal_atomic_add %0, %2, %3, off sc0\n\ts_mov_b64 exec, %1"
               : "+v"(r), "=&s"(save) : "v"(p), "v"(one) : "memory");
  return r;
}
#define WAIT_VM0() asm volatile("s_waitcnt vmcnt(0)" ::: "memory")
#define PIN(x) asm volatile("" : "+v"(x))
#define LIKELY(x) __builtin_expect(!!(x), 1)
#define UNLIKELY(x) __builtin_expect(!!(x), 0)

__device__ __forceinline__ u64 rt() { return __builtin_amdgcn_s_memrealtime(); }

constexpr unsigned OOB = 0x7ffffff0u;
__device__ __forceinline__ u32x4 poll_load(__amdgpu_buffer_rsrc_t rc, unsigned voff) {
  return __builtin_bit_cast(u32x4, __builtin_amdgcn_raw_buffer_load_b128(rc, voff, 0, 17));
}
__device__ __forceinline__ bool lines_ready(const u32x4& v, bool call_lane, unsigned call) { return __all(!call_lane || v[0] >= call); }
__device__ __forceinline__ bool wait_lines(__amdgpu_buffer_rsrc_t rc, unsigned voff, bool call_lane, int tmo_lane, unsigned call, unsigned* tmo, u32x4& fl) {
  const u64 t0 = rt();
  for (;;) {
    fl = poll_load(rc, voff);
    const unsigned t = (unsigned)__builtin_amdgcn_readlane((int)fl[0], tmo_lane);
    if (lines_ready(fl, call_lane, call)) { __builtin_amdgcn_fence(__ATOMIC_ACQUIRE, "wavefront"); return t != call; }
    if (t == call || rt() - t0 > DEADLINE) {
      __hip_atomic_store((GAS unsigned*)tmo, call, __ATOMIC_RELAXED, __HIP_MEMORY_SCOPE_AGENT);
      fl = u32x4{META_RAW, META_RAW, META_RAW, META_RAW};
      return false;
    }
    __builtin_amdgcn_s_sleep(POLL_SLEEP);
  }
}

__device__ __forceinline__ void load_x_unit(const char* x, int blk, int u, int lane, u32x4 (&v)[4]) {
  const char* p = x + (size_t)(blk * 16 + (lane & 15)) * (KDIM * 2) + (u * 128 + (lane >> 4) * 8) * 2;
#pragma unroll
  for (int i = 0; i < 4; ++i) v[i] = *(const GAS u32x4*)(p + i * 64);
}
__device__ __forceinline__ void w_fetch_at(const char* p, char* scratch) {
  __builtin_amdgcn_global_load_lds((const GAS void*)p, (LAS void*)scratch, 16, 0, 0);
  __builtin_amdgcn_global_load_lds((const GAS void*)(p + 16 * (NLOC * 2)), (LAS void*)(scratch + 1024), 16, 0, 0);
  __builtin_amdgcn_global_load_lds((const GAS void*)(p + 32 * (NLOC * 2)), (LAS void*)(scratch + 2048), 16, 0, 0);
  __builtin_amdgcn_global_load_lds((const GAS void*)(p + 48 * (NLOC * 2)), (LAS void*)(scratch + 3072), 16, 0, 0);
}
__device__ __forceinline__ void w_fetch(const char* w, int cg32, int ks2, int lane, char* scratch) {
  w_fetch_at(w + (size_t)(ks2 * 64 + (lane >> 2)) * (NLOC * 2) + cg32 * 64 + (lane & 3) * 16, scratch);
}
__device__ __forceinline__ void w_read(const char* scratch, int lane, u32x4 (&v)[4]) {
  typedef short s16x4 __attribute__((ext_vector_type(4)));
  const int g = lane >> 4, i = lane & 15;
  const char* p = scratch + (8 * g + (i >> 2)) * 64 + (i & 3) * 8;
#pragma unroll
  for (int f = 0; f < 4; ++f)
#pragma unroll
    for (int h = 0; h < 2; ++h) {
      const u32x2 t = __builtin_bit_cast(u32x2, __builtin_amdgcn_ds_read_tr16_b64_v4i16((LAS s16x4*)(p + ((f >> 1) * 32 + 4 * h) * 64 + (f & 1) * 32)));
      v[f][2 * h] = t[0];
      v[f][2 * h + 1] = t[1];
    }
}

__device__ __forceinline__ unsigned umax_(unsigned a, unsigned b) { return a > b ? a : b; }
__device__ __forceinline__ unsigned wave_max(unsigned v) {
  v = umax_(v, (unsigned)__builtin_amdgcn_update_dpp(0, (int)v, 0x111, 0xf, 0xf, false));
  v = umax_(v, (unsigned)__builtin_amdgcn_update_dpp(0, (int)v, 0x112, 0xf, 0xf, false));
  v = umax_(v, (unsigned)__builtin_amdgcn_update_dpp(0, (int)v, 0x114, 0xf, 0xf, false));
  v = umax_(v, (unsigned)__builtin_amdgcn_update_dpp(0, (int)v, 0x118, 0xf, 0xf, false));
  v = umax_(v, (unsigned)__builtin_amdgcn_update_dpp(0, (int)v, 0x142, 0xa, 0xf, false));
  v = umax_(v, (unsigned)__builtin_amdgcn_update_dpp(0, (int)v, 0x143, 0xc, 0xf, false));
  return (unsigned)__builtin_amdgcn_readlane((int)v, 63);
}
typedef unsigned short u16x2 __attribute__((ext_vector_type(2)));
__device__ __forceinline__ unsigned pk_max_u16(unsigned a, unsigned b) {
  return __builtin_bit_cast(unsigned, __builtin_elementwise_max(__builtin_bit_cast(u16x2, a), __builtin_bit_cast(u16x2, b)));
}
__device__ __forceinline__ unsigned pack_store(char* dst, char* fline, const u32x4 (&f)[4], int lane) {
  unsigned m = 0;
#pragma unroll
  for (int i = 0; i < 4; ++i)
#pragma unroll
    for (int d = 0; d < 4; ++d) m = pk_max_u16(m, f[i][d] & 0x7fff7fffu);
  m = umax_(m & 0xffffu, m >> 16);
  const unsigned mx = wave_max(m) >> 8;
  const unsigned base = mx > 7u ? mx - 7u : 0u;
  const unsigned base4 = base * 0x01010101u;
  u32x4 L0, L1, C;
  unsigned E[4];
  unsigned hq[8];
#pragma unroll
  for (int i = 0; i < 4; ++i) {
    const unsigned l0 = __builtin_amdgcn_perm(f[i][1], f[i][0], 0x06040200u), l1 = __builtin_amdgcn_perm(f[i][3], f[i][2], 0x06040200u);
    const unsigned h0 = __builtin_amdgcn_perm(f[i][1], f[i][0], 0x07050301u), h1 = __builtin_amdgcn_perm(f[i][3], f[i][2], 0x07050301u);
    const unsigned t0 = (h0 | 0x80808080u) - base4, t1 = (h1 | 0x80808080u) - base4;
    E[i] = t0 & t1;
    hq[2 * i] = h0; hq[2 * i + 1] = h1;
    const unsigned c0 = ((h0 >> 4) & 0x08080808u) | (t0 & 0x07070707u);
    const unsigned c1 = (h1 & 0x80808080u) | ((t1 & 0x07070707u) << 4);
    C[i] = c0 | c1;
    if (i < 2) { L0[2 * i] = l0; L0[2 * i + 1] = l1; } else { L1[2 * i - 4] = l0; L1[2 * i - 3] = l1; }
  }
  unsigned fmask = 0;
  bool raw = false;
  if (UNLIKELY(__any((E[0] & E[1] & E[2] & E[3] & 0x80808080u) != 0x80808080u))) {
    u32x2 ent = {0u, 0u};
#pragma unroll
    for (int i = 0; i < 4; ++i) {
      u64 bm = __ballot((E[i] & 0x80808080u) != 0x80808080u);
      unsigned cnt = 0, e0 = 0, e1 = 0;
      while (bm && cnt <= 2u) {
        const int l = __builtin_ctzll(bm);
        bm &= bm - 1;
#pragma unroll
        for (int q = 0; q < 2; ++q) {
          const unsigned h = (unsigned)__builtin_amdgcn_readlane((int)hq[2 * i + q], l);
          unsigned m2 = ~((h | 0x80808080u) - base4) & 0x80808080u;
          while (m2) {
            const unsigned b = (unsigned)__builtin_ctz(m2) >> 3;
            m2 &= m2 - 1;
            const unsigned e = 0x80000000u | ((unsigned)l << 16) | ((4u * q + b) << 8) | ((h >> (8u * b)) & 0xffu);
            e1 = cnt == 1u ? e : e1;
            e0 = cnt == 0u ? e : e0;
            ++cnt;
          }
        }
      }
      raw = raw || cnt > 2u;
      fmask |= cnt ? 1u << i : 0u;
      ent[0] = lane == i ? e0 : ent[0];
      ent[1] = lane == i ? e1 : ent[1];
    }
    if (!raw && lane < 4 && ((fmask >> lane) & 1u)) st8_wt(fline + lane * 16 + 8, ent);
  }
  if (UNLIKELY(raw)) {
    st16_wt<0>(dst, f[0]);
    st16_wt<1024>(dst, f[1]);
    st16_wt<2048>(dst, f[2]);
    st16_wt<3072>(dst, f[3]);
    return META_RAW;
  }
  st16_wt<0>(dst, L0);
  st16_wt<1024>(dst, L1);
  st16_wt<2048>(dst, C);
  return (fmask << 8) | base;
}
__device__ __forceinline__ void patch_frag(u32x4& o, unsigned e, int lane) {
  const unsigned sh = 8u + 16u * ((e >> 8) & 1u), d = (e >> 9) & 3u;
  const unsigned vm = (unsigned)lane == ((e >> 16) & 63u) ? 0xffu << sh : 0u, put = (e & 0xffu) << sh;
#pragma unroll
  for (int t = 0; t < 4; ++t) {
    const unsigned m = d == (unsigned)t ? vm : 0u;
    o[t] = (o[t] & ~m) | (put & m);
  }
}
template <int I, bool EXC>
__device__ __forceinline__ u32x4 unpack_frag(const u32x4 (&v)[3], unsigned meta, unsigned e0, unsigned e1, int fb, int lane) {
  const unsigned base4 = (meta & 0x7fu) * 0x01010101u;
  const unsigned l0 = v[I >> 1][2 * (I & 1)], l1 = v[I >> 1][2 * (I & 1) + 1];
  const unsigned c = v[2][I];
  const unsigned h0 = ((c & 0x08080808u) << 4) | ((c & 0x07070707u) + base4);
  const unsigned h1 = (c & 0x80808080u) | (((c >> 4) & 0x07070707u) + base4);
  u32x4 o;
  o[0] = __builtin_amdgcn_perm(h0, l0, 0x05010400u);
  o[1] = __builtin_amdgcn_perm(h0, l0, 0x07030602u);
  o[2] = __builtin_amdgcn_perm(h1, l1, 0x05010400u);
  o[3] = __builtin_amdgcn_perm(h1, l1, 0x07030602u);
  if (EXC && (meta & (0x100u << I))) {
    patch_frag(o, (unsigned)__builtin_amdgcn_readlane((int)e0, fb + I), lane);
    const unsigned e = (unsigned)__builtin_amdgcn_readlane((int)e1, fb + I);
    if (e) patch_frag(o, e, lane);
  }
  return o;
}

__device__ __forceinline__ float gelu_f(float y) {
  const float c = 0.7978845608028654f;
  const float k2 = -2.0f * 1.4426950408889634f * c;
  const float t = y * (1.0f + 0.044715f * y * y);
  const float e = __builtin_amdgcn_exp2f(k2 * t);
  return y * __builtin_amdgcn_rcpf(1.0f + e);
}
__device__ __forceinline__ unsigned pack_bf16(float lo, float hi) {
  typedef __bf16 bf2 __attribute__((ext_vector_type(2)));
  typedef float f2 __attribute__((ext_vector_type(2)));
  f2 f = {lo, hi};
  return __builtin_bit_cast(unsigned, __builtin_convertvector(f, bf2));
}

#define STAMP(idx) do {} while (0)
#define STAMP_FLUSH() do {} while (0)

__device__ __forceinline__ int seq_to_slot(int q, int idx_self) {
  if (q < P) return idx_self * P + q;
  q -= P;
  constexpr int WST = WSTEPS;
  for (int s = 0; s < 32; ++s) {
    if (s < P) {
      if (q < 3) { int ia = q + (q >= idx_self ? 1 : 0); return ia * P + s; }
      q -= 3;
    }
    const int j = s - WST;
    if (j >= 0 && j < PO) {
      if (q < 4) return 4 * P + q * PO + j;
      q -= 4;
    }
  }
  return 0;
}

__global__ void __launch_bounds__(NTHR, 3) ag_gemm_kernel(Args a) {
  __shared__ __attribute__((aligned(16))) char lds[LDS_BYTES];
  const int tid = threadIdx.x, lane = tid & 63;
  const int wv = __builtin_amdgcn_readfirstlane(tid >> 6);
  const int bid = blockIdx.x;
  const int rank = a.rank, type = rank & 1, idx = rank >> 1;
  char* const myws = a.myws;
  unsigned* const tmo = (unsigned*)(myws + OFF_TMO);
  STAMP(0);
  auto get_call = [&]() -> unsigned {
    asm volatile("" ::: "memory");
    unsigned ticket = 0;
    if (lane == 0)
      ticket = __hip_atomic_fetch_add((GAS unsigned*)(myws + OFF_CNT + (size_t)(bid * NWV + wv) * 32), 1u, __ATOMIC_RELAXED, __HIP_MEMORY_SCOPE_AGENT);
    return (unsigned)__builtin_amdgcn_readfirstlane(ticket) + 1u;
  };

  if (bid == NWG - 1) {
    const unsigned call = get_call();
    if (wv >= 8) { STAMP_FLUSH(); return; }
    bool ok = true;
    {
      const u64 t0 = rt();
      const char* fp = myws + OFF_OFLAG + (size_t)tid * FS * 4;
      for (;;) {
        const unsigned v = __hip_atomic_load((const GAS unsigned*)fp, __ATOMIC_RELAXED, __HIP_MEMORY_SCOPE_SYSTEM);
        const unsigned t = __hip_atomic_load((GAS unsigned*)tmo, __ATOMIC_RELAXED, __HIP_MEMORY_SCOPE_AGENT);
        if (__all(v >= call)) { ok = t != call; break; }
        if (t == call || rt() - t0 > DEADLINE) {
          __hip_atomic_store((GAS unsigned*)tmo, call, __ATOMIC_RELAXED, __HIP_MEMORY_SCOPE_AGENT);
          ok = false;
          break;
        }
        __builtin_amdgcn_s_sleep(1);
      }
    }
    if (!ok && lane == 0) {
      *(GAS unsigned short*)(a.myout + (size_t)(rank * MLOC) * (NLOC * 2)) = 0x7fc0;
    }
    STAMP(23);
    STAMP_FLUSH();
    return;
  }
  if (bid < NSEND_WG) {
    const int link = bid % 7, lq = bid / 7;
    if (wv >= SW) { get_call(); STAMP_FLUSH(); return; }
    const int j = lq * SW + wv;
    const bool same = link < 3;
    const int didx = same ? link + (link >= idx ? 1 : 0) : link - 3;
    char* const dws = a.link_ws[link];
    u32x4 v[4];
    const int b0 = same ? 0 : P, nb = same ? P : PO;
    const int slot0 = same ? idx * P : 4 * P + idx * PO;
    char* const scr = lds + wv * 4096;
    const int n = nb * 16;
    load_x_unit(a.x, b0 + (j >> 4), j & 15, lane, v);
    if (!same) w_fetch(a.w, didx * 2 + (j & 1), j >> 1, lane, scr);
    const unsigned call = get_call();
    char* const stage = dws + OFF_STAGE + (size_t)(call & 1) * STAGE_BYTES;
    auto next_ticket = [&](unsigned tk, int cnt) -> int {
      const unsigned d = (unsigned)__builtin_amdgcn_readfirstlane(tk) - (call - 1u) * (unsigned)cnt;
      return d < (unsigned)cnt ? 2 * WPL + (int)d : cnt + 2 * WPL;
    };
#pragma unroll
    for (int i = 0; i < 4; ++i) PIN(v[i]);
    STAMP(1);
    if (!same) {
      char* wdst = stage + X_BYTES + (size_t)(8 + 2 * idx) * 131072 + lane * 16;
      char* wfl = dws + OFF_WFLAG + (call & 1) * PAR_FLAG + (size_t)((8 + 2 * idx) * 32) * FS * 4;
      char* const tkw = myws + OFF_TICKET + link * 128 + 64;
      WAIT_VM0();
      u32x4 t[4];
      w_read(scr, lane, t);
      int cur = j, nxt = j + WPL, wit = 0;
      while (cur < 64) {
        const int slot = (cur & 1) * 32 + (cur >> 1);
        const int un = nxt < 64 ? nxt : cur;
        const char* wsrc = a.w + (size_t)((un >> 1) * 64 + (lane >> 2)) * (NLOC * 2) + (didx * 2 + (un & 1)) * 64 + (lane & 3) * 16;
        asm volatile("" : "+v"(wsrc));
        const unsigned meta = pack_store(wdst + (size_t)slot * 4096, wfl + (size_t)slot * FS * 4, t, lane);
        unsigned tk = ticket_issue(tkw);
        w_fetch_at(wsrc, scr);
        WAIT_VM0();
        PIN(tk);
        w_read(scr, lane, t);
#pragma unroll
        for (int i = 0; i < 4; ++i) PIN(t[i]);
        if (lane == 0) st8_wt(wfl + (size_t)slot * FS * 4, u32x2{call, meta});
        cur = nxt;
        nxt = next_ticket(tk, 64);
        if (wit < 4) STAMP(12 + wit);
        ++wit;
      }
    }
    char* const xdst = stage + (size_t)slot0 * 65536 + lane * 16;
    char* const xfl = dws + OFF_XFLAG + (call & 1) * PAR_FLAG + (size_t)(slot0 * 16) * FS * 4;
    char* const tkx = myws + OFF_TICKET + link * 128;
    int cur = j, nxt = j + WPL, sit = 0;
    while (cur < n) {
      const unsigned meta = pack_store(xdst + (size_t)cur * 4096, xfl + (size_t)cur * FS * 4, v, lane);
      unsigned tk = ticket_issue(tkx);
      const int nc = nxt < n ? nxt : cur;
      load_x_unit(a.x, b0 + (nc >> 4), nc & 15, lane, v);
      WAIT_VM0();
      PIN(tk);
#pragma unroll
      for (int i = 0; i < 4; ++i) PIN(v[i]);
      if (lane == 0) st8_wt(xfl + (size_t)cur * FS * 4, u32x2{call, meta});
      cur = nxt;
      nxt = next_ticket(tk, n);
      if (sit < 9) STAMP(3 + sit);
      ++sit;
    }
    STAMP(2);
    STAMP_FLUSH();
    return;
  }

  const int cb = bid - NSEND_WG;
  const int xl = cb & 7, loc = cb >> 3;
  const int cgl = loc & 7, set = (xl >> 1) + 4 * (loc >> 3);
  const int cg = cgl < 4 ? (xl & 1) * 4 + cgl : 8 + (xl & 1) * 4 + (cgl - 4);
  unsigned* const ready = (unsigned*)(lds + LDS_RDY);
  if (tid < 16) ready[tid] = 0;
  asm volatile("s_waitcnt lgkmcnt(0)" ::: "memory");
  __builtin_amdgcn_s_barrier();
  if (wv >= 8) {
    const int h = wv - 8;
    unsigned call;
    const int u = h * NCOMP_WG + cb;
    if (u < 256) {
      u32x4 t[4];
      char* const scr = lds + LDS_SCR + h * 4096;
      w_fetch(a.w, u >> 5, u & 31, lane, scr);
      call = get_call();
      WAIT_VM0();
      w_read(scr, lane, t);
      char* stage = myws + OFF_STAGE + (size_t)(call & 1) * STAGE_BYTES;
      char* fline = myws + OFF_WFLAG + (call & 1) * PAR_FLAG + (size_t)u * FS * 4;
      const unsigned meta = pack_store(stage + X_BYTES + (size_t)u * 4096 + lane * 16, fline, t, lane);
      WAIT_VM0();
      if (lane == 0) st8_wt(fline, u32x2{call, meta});
    } else if (u < 256 + 16 * P) {
      const int xu = u - 256;
      u32x4 t[4];
      load_x_unit(a.x, xu >> 4, xu & 15, lane, t);
      call = get_call();
      char* stage = myws + OFF_STAGE + (size_t)(call & 1) * STAGE_BYTES;
      char* fline = myws + OFF_XFLAG + (call & 1) * PAR_FLAG + (size_t)(idx * P * 16 + xu) * FS * 4;
      const unsigned meta = pack_store(stage + (size_t)(idx * P * 16 + xu) * 4096 + lane * 16, fline, t, lane);
      WAIT_VM0();
      if (lane == 0) st8_wt(fline, u32x2{call, meta});
    } else {
      call = get_call();
    }
    STAMP(1);
    char* const obase = a.obase[cg];
    char* const oflag = a.oflag[cg];
    for (int it = h; set + NSET * it < NSLOT; it += 4) {
      const int slot = seq_to_slot(set + NSET * it, idx);
      int src, blk;
      if (slot < 4 * P) { const int ia = slot / P; src = 2 * ia + type; blk = slot - ia * P; }
      else { const int s2 = slot - 4 * P; const int ia = s2 / PO; src = 2 * ia + (1 - type); blk = P + s2 - ia * PO; }
      const int row = lane >> 2, m = lane & 3;
      char* const op = obase + (size_t)(src * MLOC + blk * 16 + row) * (NLOC * 2) + m * 16;
      while (__hip_atomic_load(&ready[it], __ATOMIC_RELAXED, __HIP_MEMORY_SCOPE_WORKGROUP) < 8u) __builtin_amdgcn_s_sleep(1);
      __builtin_amdgcn_fence(__ATOMIC_ACQUIRE, "workgroup");
      const char* buf = lds + it * 16384 + row * 128;
      const int sw = ((row & 1) << 2) | (row & 2) | ((row >> 2) & 1);
      f32x4 pl[8], ph[8];
#pragma unroll
      for (int p = 0; p < 8; ++p) {
        pl[p] = *(const f32x4*)(buf + p * 2048 + (((2 * m) ^ sw) * 16));
        ph[p] = *(const f32x4*)(buf + p * 2048 + (((2 * m + 1) ^ sw) * 16));
      }
      __builtin_amdgcn_sched_barrier(0);
      const f32x4 lo = ((pl[0] + pl[1]) + (pl[2] + pl[3])) + ((pl[4] + pl[5]) + (pl[6] + pl[7]));
      const f32x4 hi = ((ph[0] + ph[1]) + (ph[2] + ph[3])) + ((ph[4] + ph[5]) + (ph[6] + ph[7]));
      u32x4 o;
      o[0] = pack_bf16(gelu_f(lo[0]), gelu_f(lo[1]));
      o[1] = pack_bf16(gelu_f(lo[2]), gelu_f(lo[3]));
      o[2] = pack_bf16(gelu_f(hi[0]), gelu_f(hi[1]));
      o[3] = pack_bf16(gelu_f(hi[2]), gelu_f(hi[3]));
      if (cg < 8) {
        *(GAS u32x4*)op = o;
      } else {
        st16_wt<0>(op, o);
        WAIT_VM0();
        if (lane == 0) st8_wt(oflag + (size_t)(slot * 2) * FS * 4, u32x2{call, 0u});
      }
      if (it < 6) STAMP(5 + it * 3);
    }
    STAMP(22);
    STAMP_FLUSH();
    return;
  }
  u64 slotpk = 0;
#pragma unroll
  for (int i = 0; i < MAXB; ++i) slotpk |= (u64)(set + NSET * i < NSLOT ? seq_to_slot(set + NSET * i, idx) : 0) << (8 * i);
  const int nblk = (NSLOT - set + NSET - 1) / NSET;
  const unsigned call = get_call();
  const char* stage_base = myws + OFF_STAGE + (size_t)(call & 1) * STAGE_BYTES;
  const __amdgpu_buffer_rsrc_t rs = __builtin_amdgcn_make_buffer_rsrc((void*)stage_base, 0, (int)STAGE_BYTES, 0x00020000);
  const __amdgpu_buffer_rsrc_t rc = __builtin_amdgcn_make_buffer_rsrc((void*)myws, 0, (int)OFF_STAGE, 0x00020000);
  const unsigned par_off = (call & 1) * (unsigned)PAR_FLAG;
  const bool xcall = lane == 0 || lane == 4;
  const unsigned xlane = lane < 8 ? (unsigned)OFF_XFLAG + par_off + (unsigned)((wv * 2 + (lane >> 2)) * FS * 4 + (lane & 3) * 16) : OOB;
  auto xoff = [&](int slot) -> unsigned { return lane == 8 ? (unsigned)OFF_TMO : xlane + (lane < 8 ? (unsigned)slot * 16 * FS * 4 : 0u); };
  auto ldp = [&](int soff, int piece) -> u32x4 {
    return __builtin_bit_cast(u32x4, __builtin_amdgcn_raw_buffer_load_b128(rs, lane * 16 + piece * 1024, soff, 17));
  };
  auto load_x = [&](u32x4 (&af)[2][3], int slot) {
    const int so = slot * 65536 + wv * 8192;
#pragma unroll
    for (int h = 0; h < 2; ++h)
#pragma unroll
      for (int p = 0; p < 3; ++p) af[h][p] = ldp(so + h * 4096, p);
  };
  auto slot_of = [&](int it) -> int { return (int)(slotpk >> (8 * it)) & 63; };
  u32x4 pv;
  auto issue = [&](u32x4 (&af)[2][3], int it) {
    load_x(af, slot_of(it));
    pv = poll_load(rc, xoff(slot_of(it + 1)));
  };
  struct Fl { unsigned meta0, meta1, e0, e1; bool ok; };
  auto take = [&](const u32x4& v, bool ok) -> Fl {
    return Fl{(unsigned)__builtin_amdgcn_readlane((int)v[1], 0), (unsigned)__builtin_amdgcn_readlane((int)v[1], 4), v[2], v[3], ok};
  };
  u32x4 bufA[2][3], bufB[2][3];
  Fl flA, flB = {META_RAW, META_RAW, 0u, 0u, true};
  { const bool ok = wait_lines(rc, xoff(slot_of(0)), xcall, 8, call, tmo, pv); flA = take(pv, ok); }
  STAMP(3);
  load_x(bufA, slot_of(0));
  u32x4 wfl;
  const bool okw = wait_lines(rc, lane < 16 ? (unsigned)OFF_WFLAG + par_off + (unsigned)((cg * 32 + wv * 4 + (lane >> 2)) * FS * 4 + (lane & 3) * 16)
                                            : lane == 16 ? (unsigned)OFF_TMO : OOB,
                              lane < 16 && (lane & 3) == 0, 16, call, tmo, wfl);
  STAMP(2);
  u32x4 wf[4][4];
  {
    const int soff = (int)X_BYTES + cg * 131072 + wv * 4 * 4096;
    u32x4 wp[4][3];
#pragma unroll
    for (int s = 0; s < 4; ++s)
#pragma unroll
      for (int p = 0; p < 3; ++p) wp[s][p] = ldp(soff + s * 4096, p);
#pragma unroll
    for (int s = 0; s < 4; ++s)
#pragma unroll
      for (int p = 0; p < 3; ++p) PIN(wp[s][p]);
    STAMP(1);
#pragma unroll
    for (int s = 0; s < 4; ++s) {
      const unsigned meta = (unsigned)__builtin_amdgcn_readlane((int)wfl[1], 4 * s);
      if (LIKELY((meta >> 8) == 0u)) {
        wf[s][0] = unpack_frag<0, false>(wp[s], meta, 0u, 0u, 0, lane);
        wf[s][1] = unpack_frag<1, false>(wp[s], meta, 0u, 0u, 0, lane);
        wf[s][2] = unpack_frag<2, false>(wp[s], meta, 0u, 0u, 0, lane);
        wf[s][3] = unpack_frag<3, false>(wp[s], meta, 0u, 0u, 0, lane);
      } else if (meta & META_RAW) {
        wf[s][0] = wp[s][0]; wf[s][1] = wp[s][1]; wf[s][2] = wp[s][2]; wf[s][3] = ldp(soff + s * 4096, 3);
      } else {
        wf[s][0] = unpack_frag<0, true>(wp[s], meta, wfl[2], wfl[3], 4 * s, lane);
        wf[s][1] = unpack_frag<1, true>(wp[s], meta, wfl[2], wfl[3], 4 * s, lane);
        wf[s][2] = unpack_frag<2, true>(wp[s], meta, wfl[2], wfl[3], 4 * s, lane);
        wf[s][3] = unpack_frag<3, true>(wp[s], meta, wfl[2], wfl[3], 4 * s, lane);
      }
    }
  }
  STAMP(21);
  pv = poll_load(rc, xoff(slot_of(1)));
  auto step = [&](int it, u32x4 (&cur)[2][3], const Fl& cfl, u32x4 (&nxt)[2][3], Fl& nfl) {
    const bool more = it + 1 < nblk;
    const int cslot = slot_of(it);
#pragma unroll
    for (int h = 0; h < 2; ++h)
#pragma unroll
      for (int p = 0; p < 3; ++p) PIN(cur[h][p]);
    PIN(pv);
    const bool rdy = more && lines_ready(pv, xcall, call);
    const bool cok = cfl.ok && (unsigned)__builtin_amdgcn_readlane((int)pv[0], 8) != call;
    __builtin_amdgcn_fence(__ATOMIC_ACQUIRE, "wavefront");
    if (rdy) { nfl = take(pv, true); issue(nxt, it + 1); }
    __builtin_amdgcn_sched_barrier(0);
    f32x4 acc[2];
#pragma unroll
    for (int jj = 0; jj < 2; ++jj) acc[jj] = f32x4{0.f, 0.f, 0.f, 0.f};
    auto mm = [&](int h, int s, u32x4 xf) {
#pragma unroll
      for (int jj = 0; jj < 2; ++jj)
        acc[jj] = __builtin_amdgcn_mfma_f32_16x16x32_bf16(__builtin_bit_cast(bf16x8, wf[(4 * h + s) >> 1][(s & 1) * 2 + jj]),
                                                          __builtin_bit_cast(bf16x8, xf), acc[jj], 0, 0, 0);
    };
    const unsigned meta0 = cfl.meta0, meta1 = cfl.meta1;
    if (LIKELY(((meta0 | meta1) >> 8) == 0u)) {
#pragma unroll
      for (int h = 0; h < 2; ++h) {
        const unsigned meta = h ? meta1 : meta0;
        mm(h, 0, unpack_frag<0, false>(cur[h], meta, 0u, 0u, 0, lane));
        mm(h, 1, unpack_frag<1, false>(cur[h], meta, 0u, 0u, 0, lane));
        mm(h, 2, unpack_frag<2, false>(cur[h], meta, 0u, 0u, 0, lane));
        mm(h, 3, unpack_frag<3, false>(cur[h], meta, 0u, 0u, 0, lane));
      }
    } else {
#pragma unroll
      for (int h = 0; h < 2; ++h) {
        const unsigned meta = h ? meta1 : meta0;
        if (UNLIKELY(meta & META_RAW)) {
          mm(h, 0, cur[h][0]); mm(h, 1, cur[h][1]); mm(h, 2, cur[h][2]);
          mm(h, 3, ldp(cslot * 65536 + wv * 8192 + h * 4096, 3));
        } else {
          mm(h, 0, unpack_frag<0, true>(cur[h], meta, cfl.e0, cfl.e1, 4 * h, lane));
          mm(h, 1, unpack_frag<1, true>(cur[h], meta, cfl.e0, cfl.e1, 4 * h, lane));
          mm(h, 2, unpack_frag<2, true>(cur[h], meta, cfl.e0, cfl.e1, 4 * h, lane));
          mm(h, 3, unpack_frag<3, true>(cur[h], meta, cfl.e0, cfl.e1, 4 * h, lane));
        }
      }
    }
    if (UNLIKELY(!(cok && okw))) {
      const float qn = __builtin_nanf("");
#pragma unroll
      for (int jj = 0; jj < 2; ++jj) acc[jj] = f32x4{qn, qn, qn, qn};
    }
    {
      const int m = lane & 15, qd = lane >> 4;
      const int sw = ((m & 1) << 2) | (m & 2) | ((m >> 2) & 1);
      char* buf = lds + it * 16384 + wv * 2048 + m * 128;
#pragma unroll
      for (int jj = 0; jj < 2; ++jj) *(f32x4*)(buf + (((4 * jj + qd) ^ sw) * 16)) = acc[jj];
    }
    if (lane == 0) __hip_atomic_fetch_add(&ready[it], 1u, __ATOMIC_RELEASE, __HIP_MEMORY_SCOPE_WORKGROUP);
    if (it < 6) STAMP(4 + it * 3);
    if (more && !rdy) {
      const bool ok = wait_lines(rc, xoff(slot_of(it + 1)), xcall, 8, call, tmo, pv);
      nfl = take(pv, ok);
      issue(nxt, it + 1);
    }
    if (more && it < 5) STAMP(6 + it * 3);
  };
  for (int it = 0; it < nblk; it += 2) {
    step(it, bufA, flA, bufB, flB);
    if (it + 1 < nblk) step(it + 1, bufB, flB, bufA, flA);
  }
  STAMP(22);
  STAMP_FLUSH();
}

}

extern "C" void kernel_launch(int rank, int nranks, ncclComm_t comm, hipStream_t stream, void* const* d_in, const int* in_sizes,
                              int n_in, void* d_out, int out_size, const int* in_dtypes, int out_dtype, void* const* d_out_all,
                              void* const* d_ws_all, size_t ws_bytes) {
  using namespace agk;
  (void)comm; (void)in_dtypes; (void)out_dtype;
  if (nranks != NRANK || n_in < 2 || rank < 0 || rank >= NRANK || ws_bytes < WS_NEED || in_sizes[0] != MLOC * KDIM ||
      in_sizes[1] != KDIM * NLOC || out_size != MDIM * NLOC) {
    fprintf(stderr, "kernel_launch: unexpected problem shape\n");
    return;
  }
  const int type = rank & 1, idx = rank >> 1;
  Args a;
  memset(&a, 0, sizeof a);
  for (int link = 0; link < 7; ++link) {
    const bool same = link < 3;
    const int didx = same ? link + (link >= idx ? 1 : 0) : link - 3;
    a.link_ws[link] = (char*)d_ws_all[2 * didx + (same ? type : 1 - type)];
  }
  a.link_ws[7] = (char*)d_ws_all[rank];
  for (int cg = 0; cg < 16; ++cg) {
    const int dest = cg < 8 ? rank : 2 * ((cg - 8) >> 1) + (1 - type);
    a.obase[cg] = (char*)d_out_all[dest] + (cg < 8 ? cg * 32 : idx * 64 + (cg & 1) * 32) * 2;
    a.oflag[cg] = (char*)d_ws_all[dest] + OFF_OFLAG + (size_t)(idx * 128 + (cg & 1)) * FS * 4;
  }
  for (int p = 0; p < NRANK; ++p) a.all_ws[p] = (char*)d_ws_all[p];
  a.myws = (char*)d_ws_all[rank];
  a.myout = (char*)d_out;
  a.x = (const char*)d_in[0];
  a.w = (const char*)d_in[1];
  a.rank = rank;
  hipLaunchKernelGGL(ag_gemm_kernel, dim3(NWG), dim3(NTHR), 0, stream, a);
}
```

```cpp
#include <hip/hip_runtime.h>
#include <rccl/rccl.h>
#include <cstdio>
#include <cstdint>
#include <cstring>

namespace agk {

typedef unsigned u32x4 __attribute__((ext_vector_type(4)));
typedef unsigned u32x2 __attribute__((ext_vector_type(2)));
typedef unsigned u32x16 __attribute__((ext_vector_type(16)));
typedef float f32x4 __attribute__((ext_vector_type(4)));
typedef __bf16 bf16x8 __attribute__((ext_vector_type(8)));
typedef unsigned long long u64;
#define GAS __attribute__((address_space(1)))
#define LAS __attribute__((address_space(3)))

constexpr int NRANK = 8;
constexpr int MDIM = 2048, KDIM = 2048, NLOC = 256, MLOC = 256;
#define PBLK 11
constexpr int P = PBLK;
constexpr int PO = 16 - P;
constexpr int NSLOT = 4 * P + 4 * PO;
static_assert(NSLOT == 64, "");
#define POLL_SLEEP 1
#define SWV 2
#define LINKMAP 1
#define DATA_AUX 17
#define ZERO_DELAY 100
#define WSTEPS 4
constexpr int FS = 32;

#define WGPL 8
constexpr int NSEND_WG = 7 * WGPL;
constexpr int SW = SWV;
constexpr int WPL = WGPL * SW;
#define NSETO 12
#define NSETR 12
constexpr int NSET_O = NSETO, NSET_R = NSETR;
static_assert(NSET_O % 4 == 0 && NSET_R % 4 == 0, "a row block's workgroups sit on one pair of XCD labels");
constexpr int NCOMP_WG = 8 * (NSET_O + NSET_R);
constexpr int NWG = NSEND_WG + NCOMP_WG + 1;
constexpr int NWV = 12;
constexpr int NTHR = NWV * 64;
constexpr int NSET_MIN = NSET_O < NSET_R ? NSET_O : NSET_R;
constexpr int MAXB = (NSLOT + NSET_MIN - 1) / NSET_MIN;
static_assert(NWG <= 256 && MAXB <= 8, "");
static_assert(SW <= NWV, "");
constexpr int LDS_SCR = MAXB * 16384;
constexpr int LDS_RDY = LDS_SCR + 4 * 4096;
constexpr int STAMP_LDS = 0;
constexpr int LDS_BYTES = LDS_RDY + 64 + STAMP_LDS;
static_assert(2 * NWV * 4096 <= LDS_RDY, "");

constexpr size_t OFF_CNT = 0;
constexpr size_t CNT_BYTES = (size_t)NWG * NWV * 32;
constexpr size_t OFF_TMO = 128 << 10;
constexpr size_t OFF_TICKET = OFF_TMO + 1024;
constexpr int NRX = (3 * P + 4 * PO) * 16, NRU = NRX + 256, NLU = P * 16 + 256;
constexpr size_t OFF_RLINE = 132 << 10, RLINE_BYTES = (size_t)NRU * 128;
constexpr size_t OFF_LLINE = OFF_RLINE + 3 * RLINE_BYTES;
constexpr size_t OFF_OFLAG = OFF_LLINE + (size_t)NLU * 128;
constexpr size_t OFF_FLAG_END = OFF_OFLAG + (size_t)512 * FS * 4;
constexpr size_t OFF_RSTAGE = 1 << 20, RSTAGE_BYTES = (size_t)NRU * 4096;
constexpr size_t OFF_LSTAGE = OFF_RSTAGE + 3 * RSTAGE_BYTES;
constexpr size_t WS_NEED = OFF_LSTAGE + (size_t)NLU * 4096;
constexpr size_t OFF_DBG = 16 << 20;
static_assert(CNT_BYTES <= OFF_TMO && OFF_FLAG_END <= OFF_RSTAGE && WS_NEED <= OFF_DBG, "");
constexpr u64 DEADLINE = 50000000ull;

constexpr unsigned META_RAW = 0x80000000u;
constexpr unsigned META_SV = 0x40000000u;

struct Args {
  char* link_ws[8];
  char* obase[16];
  char* oflag[16];
  char* all_ws[8];
  char* myws;
  char* myout;
  const char* x;
  const char* w;
  int rank;
  int pad;
};

template <int OFF>
__device__ __forceinline__ void st16_wt(char* p, u32x4 v) {
  asm volatile("global_store_dwordx4 %0, %1, off offset:%c2 sc0 sc1\n\ts_nop 1" ::"v"(p), "v"(v), "i"(OFF) : "memory");
}
__device__ __forceinline__ void st4_wt(char* p, unsigned v) {
  asm volatile("global_store_dword %0, %1, off sc0 sc1\n\ts_nop 1" ::"v"(p), "v"(v) : "memory");
}
__device__ __forceinline__ void st8_wt(char* p, u32x2 v) {
  asm volatile("global_store_dwordx2 %0, %1, off sc0 sc1\n\ts_nop 1" ::"v"(p), "v"(v) : "memory");
}
__device__ __forceinline__ unsigned ticket_issue(char* p) {
  unsigned r = 0, one = 1;
  u64 save;
  asm volatile("s_mov_b64 %1, exec\n\ts_mov_b64 exec, 1\n\tglobal_atomic_add %0, %2, %3, off sc0\n\ts_mov_b64 exec, %1"
               : "+v"(r), "=&s"(save) : "v"(p), "v"(one) : "memory");
  return r;
}
#define WAIT_VM0() asm volatile("s_waitcnt vmcnt(0)" ::: "memory")
#define PIN(x) asm volatile("" : "+v"(x))
#define LIKELY(x) __builtin_expect(!!(x), 1)
#define UNLIKELY(x) __builtin_expect(!!(x), 0)

__device__ __forceinline__ u64 rt() { return __builtin_amdgcn_s_memrealtime(); }

constexpr unsigned OOB = 0x7ffffff0u;
__device__ __forceinline__ u32x4 poll_load(__amdgpu_buffer_rsrc_t rc, unsigned voff) {
  return __builtin_bit_cast(u32x4, __builtin_amdgcn_raw_buffer_load_b128(rc, voff, 0, 17));
}
__device__ __forceinline__ bool lines_ready(const u32x4& v, bool line_lane, bool first, unsigned call) {
  const unsigned late = (unsigned)__builtin_amdgcn_update_dpp(0, (int)v[0], 0x101, 0xf, 0xf, false);
  return __all(!line_lane || (first ? v[0] >= call && ((v[1] & META_SV) || late >= call) : v[1] >= call));
}
__device__ __forceinline__ bool wait_lines(__amdgpu_buffer_rsrc_t rc, unsigned voff, bool line_lane, bool first, int tmo_lane, unsigned call, unsigned* tmo, u32x4& fl) {
  const u64 t0 = rt();
  for (;;) {
    fl = poll_load(rc, voff);
    const unsigned t = (unsigned)__builtin_amdgcn_readlane((int)fl[0], tmo_lane);
    if (lines_ready(fl, line_lane, first, call)) { __builtin_amdgcn_fence(__ATOMIC_ACQUIRE, "wavefront"); return t != call; }
    if (t == call || rt() - t0 > DEADLINE) {
      __hip_atomic_store((GAS unsigned*)tmo, call, __ATOMIC_RELAXED, __HIP_MEMORY_SCOPE_AGENT);
      fl = u32x4{META_RAW, META_RAW, META_RAW, META_RAW};
      return false;
    }
    __builtin_amdgcn_s_sleep(POLL_SLEEP);
  }
}
__device__ __forceinline__ unsigned umin_(unsigned a, unsigned b) { return a < b ? a : b; }
__device__ __forceinline__ unsigned min12(const u32x4 (&v)[3]) {
  unsigned m = umin_(umin_(v[0][0], v[0][1]), v[0][2]);
  m = umin_(umin_(m, v[0][3]), v[1][0]);
  m = umin_(umin_(m, v[1][1]), v[1][2]);
  m = umin_(umin_(m, v[1][3]), v[2][0]);
  m = umin_(umin_(m, v[2][1]), v[2][2]);
  return umin_(m, v[2][3]);
}

__device__ __forceinline__ unsigned x_lane_off(int lane, int i) {
  const int r = 4 * i + (lane >> 4);
  return (unsigned)(r * (KDIM * 2) + (((lane & 15) ^ r) * 16));
}
__device__ __forceinline__ void x_fetch(const char* x, int blk, int u, const unsigned (&off)[4], char* scratch) {
  const GAS char* base = (const GAS char*)x + (size_t)(blk * 16 * (KDIM * 2) + u * 256);
#pragma unroll
  for (int i = 0; i < 4; ++i) __builtin_amdgcn_global_load_lds((const GAS void*)(base + off[i]), (LAS void*)(scratch + i * 1024), 16, 0, 0);
}
__device__ __forceinline__ void x_read(const char* scratch, int lane, u32x4 (&v)[4]) {
  const int m = lane & 15, q = lane >> 4;
  const char* p = scratch + m * 256;
#pragma unroll
  for (int s = 0; s < 4; ++s) v[s] = *(const LAS u32x4*)(p + (((4 * s + q) ^ m) * 16));
}
__device__ __forceinline__ unsigned w_lane_off(int lane) { return (unsigned)((lane >> 2) * (NLOC * 2) + (lane & 3) * 16); }
__device__ __forceinline__ void w_fetch(const char* w, int cg32, int ks2, unsigned off, char* scratch) {
  const GAS char* base = (const GAS char*)w + (size_t)(ks2 * 64 * (NLOC * 2) + cg32 * 64);
#pragma unroll
  for (int i = 0; i < 4; ++i)
    __builtin_amdgcn_global_load_lds((const GAS void*)(base + i * 16 * (NLOC * 2) + off), (LAS void*)(scratch + i * 1024), 16, 0, 0);
}
__device__ __forceinline__ void w_read(const char* scratch, int lane, u32x4 (&v)[4]) {
  typedef short s16x4 __attribute__((ext_vector_type(4)));
  const int g = lane >> 4, i = lane & 15;
  const char* p = scratch + (8 * g + (i >> 2)) * 64 + (i & 3) * 8;
#pragma unroll
  for (int f = 0; f < 4; ++f)
#pragma unroll
    for (int h = 0; h < 2; ++h) {
      const u32x2 t = __builtin_bit_cast(u32x2, __builtin_amdgcn_ds_read_tr16_b64_v4i16((LAS s16x4*)(p + ((f >> 1) * 32 + 4 * h) * 64 + (f & 1) * 32)));
      v[f][2 * h] = t[0];
      v[f][2 * h + 1] = t[1];
    }
}

__device__ __forceinline__ unsigned lds_off(const char* p) { return (unsigned)(size_t)(const LAS char*)p; }
__device__ __forceinline__ void x_read_asm(const char* scratch, int lane, u32x4 (&v)[4]) {
  const int m = lane & 15, q = lane >> 4;
  const unsigned p = lds_off(scratch) + m * 256;
  asm volatile("ds_read_b128 %0, %4\n\tds_read_b128 %1, %5\n\tds_read_b128 %2, %6\n\tds_read_b128 %3, %7\n\ts_waitcnt lgkmcnt(0)"
               : "=&v"(v[0]), "=&v"(v[1]), "=&v"(v[2]), "=&v"(v[3])
               : "v"(p + ((q ^ m) * 16)), "v"(p + (((4 + q) ^ m) * 16)), "v"(p + (((8 + q) ^ m) * 16)), "v"(p + (((12 + q) ^ m) * 16))
               : "memory");
}
__device__ __forceinline__ void w_read_asm(const char* scratch, int lane, u32x4 (&v)[4]) {
  const int g = lane >> 4, i = lane & 15;
  const unsigned p = lds_off(scratch) + (8 * g + (i >> 2)) * 64 + (i & 3) * 8;
  u32x2 t0, t1, t2, t3, t4, t5, t6, t7;
  asm volatile("ds_read_b64_tr_b16 %0, %8\n\tds_read_b64_tr_b16 %1, %8 offset:256\n\tds_read_b64_tr_b16 %2, %8 offset:32\n\t"
               "ds_read_b64_tr_b16 %3, %8 offset:288\n\tds_read_b64_tr_b16 %4, %8 offset:2048\n\tds_read_b64_tr_b16 %5, %8 offset:2304\n\t"
               "ds_read_b64_tr_b16 %6, %8 offset:2080\n\tds_read_b64_tr_b16 %7, %8 offset:2336\n\ts_waitcnt lgkmcnt(0)"
               : "=&v"(t0), "=&v"(t1), "=&v"(t2), "=&v"(t3), "=&v"(t4), "=&v"(t5), "=&v"(t6), "=&v"(t7) : "v"(p) : "memory");
  v[0] = u32x4{t0[0], t0[1], t1[0], t1[1]};
  v[1] = u32x4{t2[0], t2[1], t3[0], t3[1]};
  v[2] = u32x4{t4[0], t4[1], t5[0], t5[1]};
  v[3] = u32x4{t6[0], t6[1], t7[0], t7[1]};
}

__device__ __forceinline__ unsigned umax_(unsigned a, unsigned b) { return a > b ? a : b; }
__device__ __forceinline__ unsigned wave_max(unsigned v) {
  v = umax_(v, (unsigned)__builtin_amdgcn_update_dpp(0, (int)v, 0x111, 0xf, 0xf, false));
  v = umax_(v, (unsigned)__builtin_amdgcn_update_dpp(0, (int)v, 0x112, 0xf, 0xf, false));
  v = umax_(v, (unsigned)__builtin_amdgcn_update_dpp(0, (int)v, 0x114, 0xf, 0xf, false));
  v = umax_(v, (unsigned)__builtin_amdgcn_update_dpp(0, (int)v, 0x118, 0xf, 0xf, false));
  v = umax_(v, (unsigned)__builtin_amdgcn_update_dpp(0, (int)v, 0x142, 0xa, 0xf, false));
  v = umax_(v, (unsigned)__builtin_amdgcn_update_dpp(0, (int)v, 0x143, 0xc, 0xf, false));
  return (unsigned)__builtin_amdgcn_readlane((int)v, 63);
}
typedef unsigned short u16x2 __attribute__((ext_vector_type(2)));
__device__ __forceinline__ unsigned pk_max_u16(unsigned a, unsigned b) {
  return __builtin_bit_cast(unsigned, __builtin_elementwise_max(__builtin_bit_cast(u16x2, a), __builtin_bit_cast(u16x2, b)));
}
struct Packed { u32x4 s[4]; u32x2 ent; unsigned meta; };
__device__ __forceinline__ unsigned bfi(unsigned mask, unsigned a, unsigned b) {
  unsigned r;
  asm("v_bfi_b32 %0, %1, %2, %3" : "=v"(r) : "s"(mask), "v"(a), "v"(b));
  return r;
}
__device__ __forceinline__ void pack_unit(const u32x4 (&f)[4], int lane, Packed& o) {
  u32x4 L0, L1, C;
  unsigned hq[8], uq[8];
#pragma unroll
  for (int i = 0; i < 4; ++i) {
    hq[2 * i] = __builtin_amdgcn_perm(f[i][1], f[i][0], 0x07050301u);
    hq[2 * i + 1] = __builtin_amdgcn_perm(f[i][3], f[i][2], 0x07050301u);
    uq[2 * i] = hq[2 * i] | 0x80808080u;
    uq[2 * i + 1] = hq[2 * i + 1] | 0x80808080u;
  }
  const unsigned mo = pk_max_u16(pk_max_u16(pk_max_u16(uq[0], uq[1]), pk_max_u16(uq[2], uq[3])), pk_max_u16(pk_max_u16(uq[4], uq[5]), pk_max_u16(uq[6], uq[7])));
  const unsigned me = pk_max_u16(pk_max_u16(pk_max_u16(uq[0] << 8, uq[1] << 8), pk_max_u16(uq[2] << 8, uq[3] << 8)),
                                 pk_max_u16(pk_max_u16(uq[4] << 8, uq[5] << 8), pk_max_u16(uq[6] << 8, uq[7] << 8)));
  const unsigned m2 = pk_max_u16(mo, me);
  const unsigned mx = (wave_max(umax_(m2 & 0xffffu, m2 >> 16)) >> 8) & 0x7fu;
  const unsigned base = (mx > 7u ? mx : 7u) - 7u;
  const unsigned base4 = base * 0x01010101u;
  unsigned E[4];
#pragma unroll
  for (int i = 0; i < 4; ++i) {
    const unsigned l0 = __builtin_amdgcn_perm(f[i][1], f[i][0], 0x06040200u), l1 = __builtin_amdgcn_perm(f[i][3], f[i][2], 0x06040200u);
    const unsigned t0 = uq[2 * i] - base4, t1 = uq[2 * i + 1] - base4;
    E[i] = t0 & t1;
    C[i] = bfi(0x0f0f0f0fu, bfi(0x07070707u, t0, hq[2 * i] >> 4), bfi(0x70707070u, t1 << 4, hq[2 * i + 1]));
    if (i < 2) { L0[2 * i] = l0; L0[2 * i + 1] = l1; } else { L1[2 * i - 4] = l0; L1[2 * i - 3] = l1; }
  }
  unsigned fmask = 0;
  bool raw = false;
  u32x2 ent = {0u, 0u};
  if (UNLIKELY(__any((E[0] & E[1] & E[2] & E[3] & 0x80808080u) != 0x80808080u))) {
#pragma unroll
    for (int i = 0; i < 4; ++i) {
      u64 bm = __ballot((E[i] & 0x80808080u) != 0x80808080u);
      unsigned cnt = 0, e0 = 0, e1 = 0;
      while (bm && cnt <= 2u) {
        const int l = __builtin_ctzll(bm);
        bm &= bm - 1;
#pragma unroll
        for (int q = 0; q < 2; ++q) {
          const unsigned h = (unsigned)__builtin_amdgcn_readlane((int)hq[2 * i + q], l);
          unsigned m2 = ~((h | 0x80808080u) - base4) & 0x80808080u;
          while (m2) {
            const unsigned b = (unsigned)__builtin_ctz(m2) >> 3;
            m2 &= m2 - 1;
            const unsigned e = 0x80000000u | ((unsigned)l << 16) | ((4u * q + b) << 8) | ((h >> (8u * b)) & 0xffu);
            e1 = cnt == 1u ? e : e1;
            e0 = cnt == 0u ? e : e0;
            ++cnt;
          }
        }
      }
      raw = raw || cnt > 2u;
      fmask |= cnt ? 1u << i : 0u;
      ent[0] = lane == i ? e0 : ent[0];
      ent[1] = lane == i ? e1 : ent[1];
    }
  }
  o.ent = ent;
  if (UNLIKELY(raw)) {
    o.s[0] = f[0]; o.s[1] = f[1]; o.s[2] = f[2]; o.s[3] = f[3];
    o.meta = META_RAW;
  } else {
    o.s[0] = L0; o.s[1] = L1; o.s[2] = C; o.s[3] = C;
    const u32x4 t[3] = {L0, L1, C};
    o.meta = (fmask << 8) | base | (__all(min12(t) != 0u) ? META_SV : 0u);
  }
}
__device__ __forceinline__ void store_pieces(char* dst, const Packed& p) {
  st16_wt<0>(dst, p.s[0]);
  st16_wt<1024>(dst, p.s[1]);
  st16_wt<2048>(dst, p.s[2]);
  if (UNLIKELY(p.meta & META_RAW)) st16_wt<3072>(dst, p.s[3]);
}
__device__ __forceinline__ void store_line(char* line, const Packed& p, unsigned meta, unsigned call, unsigned late, int lane) {
  const u32x4 v = {lane == 0 ? call : lane == 1 ? late : 0u, lane == 0 ? meta : call, p.ent[0], p.ent[1]};
  if (lane < 4) st16_wt<0>(line + lane * 16, v);
}
__device__ __forceinline__ void patch_frag(u32x4& o, unsigned e, int lane) {
  const unsigned sh = 8u + 16u * ((e >> 8) & 1u), d = (e >> 9) & 3u;
  const unsigned vm = (unsigned)lane == ((e >> 16) & 63u) ? 0xffu << sh : 0u, put = (e & 0xffu) << sh;
#pragma unroll
  for (int t = 0; t < 4; ++t) {
    const unsigned m = d == (unsigned)t ? vm : 0u;
    o[t] = (o[t] & ~m) | (put & m);
  }
}
template <int I, bool EXC>
__device__ __forceinline__ u32x4 unpack_frag(const u32x4 (&v)[3], unsigned meta, unsigned e0, unsigned e1, int fb, int lane) {
  const unsigned base4 = (meta & 0x7fu) * 0x01010101u;
  const unsigned l0 = v[I >> 1][2 * (I & 1)], l1 = v[I >> 1][2 * (I & 1) + 1];
  const unsigned c = v[2][I];
  const unsigned h0 = ((c & 0x08080808u) << 4) | ((c & 0x07070707u) + base4);
  const unsigned h1 = (c & 0x80808080u) | (((c >> 4) & 0x07070707u) + base4);
  u32x4 o;
  o[0] = __builtin_amdgcn_perm(h0, l0, 0x05010400u);
  o[1] = __builtin_amdgcn_perm(h0, l0, 0x07030602u);
  o[2] = __builtin_amdgcn_perm(h1, l1, 0x05010400u);
  o[3] = __builtin_amdgcn_perm(h1, l1, 0x07030602u);
  if (EXC && (meta & (0x100u << I))) {
    patch_frag(o, (unsigned)__builtin_amdgcn_readlane((int)e0, fb + I), lane);
    const unsigned e = (unsigned)__builtin_amdgcn_readlane((int)e1, fb + I);
    if (e) patch_frag(o, e, lane);
  }
  return o;
}

__device__ __forceinline__ float gelu_f(float y) {
  const float c = 0.7978845608028654f;
  const float k2 = -2.0f * 1.4426950408889634f * c;
  const float t = y * (1.0f + 0.044715f * y * y);
  const float e = __builtin_amdgcn_exp2f(k2 * t);
  return y * __builtin_amdgcn_rcpf(1.0f + e);
}
__device__ __forceinline__ unsigned pack_bf16(float lo, float hi) {
  typedef __bf16 bf2 __attribute__((ext_vector_type(2)));
  typedef float f2 __attribute__((ext_vector_type(2)));
  f2 f = {lo, hi};
  return __builtin_bit_cast(unsigned, __builtin_convertvector(f, bf2));
}

#define STAMP(idx) do {} while (0)
#define STAMP_FLUSH() do {} while (0)

__device__ __forceinline__ int seq_to_slot(int q, int idx_self) {
  if (q < P) return idx_self * P + q;
  q -= P;
  constexpr int WST = WSTEPS;
  for (int s = 0; s < 32; ++s) {
    if (s < P) {
      if (q < 3) { int ia = q + (q >= idx_self ? 1 : 0); return ia * P + s; }
      q -= 3;
    }
    const int j = s - WST;
    if (j >= 0 && j < PO) {
      if (q < 4) return 4 * P + q * PO + j;
      q -= 4;
    }
  }
  return 0;
}

__global__ void __launch_bounds__(NTHR, 3) ag_gemm_kernel(Args a) {
  __shared__ __attribute__((aligned(16))) char lds[LDS_BYTES];
  const int tid = threadIdx.x, lane = tid & 63;
  const int wv = __builtin_amdgcn_readfirstlane(tid >> 6);
  const int bid = blockIdx.x;
  const int rank = a.rank, type = rank & 1, idx = rank >> 1;
  char* const myws = a.myws;
  unsigned* const tmo = (unsigned*)(myws + OFF_TMO);
  STAMP(0);
  auto get_call = [&]() -> unsigned {
    asm volatile("" ::: "memory");
    unsigned ticket = 0;
    if (lane == 0)
      ticket = __hip_atomic_fetch_add((GAS unsigned*)(myws + OFF_CNT + (size_t)(bid * NWV + wv) * 32), 1u, __ATOMIC_RELAXED, __HIP_MEMORY_SCOPE_AGENT);
    return (unsigned)__builtin_amdgcn_readfirstlane(ticket) + 1u;
  };

  if (bid == NWG - 1) {
    const unsigned call = get_call();
    if (wv >= 8) { STAMP_FLUSH(); return; }
    bool ok = true;
    {
      const u64 t0 = rt();
      const char* fp = myws + OFF_OFLAG + (size_t)tid * FS * 4;
      for (;;) {
        const unsigned v = __hip_atomic_load((const GAS unsigned*)fp, __ATOMIC_RELAXED, __HIP_MEMORY_SCOPE_SYSTEM);
        const unsigned t = __hip_atomic_load((GAS unsigned*)tmo, __ATOMIC_RELAXED, __HIP_MEMORY_SCOPE_AGENT);
        if (__all(v >= call)) { ok = t != call; break; }
        if (t == call || rt() - t0 > DEADLINE) {
          __hip_atomic_store((GAS unsigned*)tmo, call, __ATOMIC_RELAXED, __HIP_MEMORY_SCOPE_AGENT);
          ok = false;
          break;
        }
        __builtin_amdgcn_s_sleep(1);
      }
    }
    if (!ok && lane == 0) {
      *(GAS unsigned short*)(a.myout + (size_t)(rank * MLOC) * (NLOC * 2)) = 0x7fc0;
    }
    STAMP(23);
    STAMP_FLUSH();
    return;
  }
  if (bid < NSEND_WG) {
    const int link = bid % 7, lq = bid / 7;
    if (wv >= SW) { get_call(); STAMP_FLUSH(); return; }
    const int j = lq * SW + wv;
    const bool same = link < 3;
    const int didx = same ? link + (link >= idx ? 1 : 0) : link - 3;
    char* const dws = a.link_ws[link];
    u32x4 v[4];
    const int b0 = same ? 0 : P, nb = same ? P : PO;
    const int rx0 = (same ? (idx < didx ? idx : idx - 1) * P : 3 * P + idx * PO) * 16, rw0 = NRX + idx * 64;
    char* const scr = lds + wv * 4096;
    const int nw = same ? 0 : 64, n = nw + nb * 16;
    char* const sc1 = lds + NWV * 4096 + wv * 4096;
    const unsigned xoffs[4] = {x_lane_off(lane, 0), x_lane_off(lane, 1), x_lane_off(lane, 2), x_lane_off(lane, 3)};
    const unsigned woff = w_lane_off(lane);
    auto fetch = [&](int u, char* sc) {
      u = u < n ? u : j;
      if (u < nw) w_fetch(a.w, didx * 2 + (u & 1), u >> 1, woff, sc);
      else x_fetch(a.x, b0 + ((u - nw) >> 4), (u - nw) & 15, xoffs, sc);
    };
    auto read = [&](int u, const char* sc, u32x4 (&f)[4]) {
      u = u < n ? u : j;
      if (u < nw) w_read_asm(sc, lane, f); else x_read_asm(sc, lane, f);
    };
    __builtin_amdgcn_s_waitcnt(0x0F70);
    STAMP(17);
    fetch(j, scr);
    fetch(j + WPL, sc1);
    const unsigned call = get_call();
    const unsigned buf = call % 3u;
    char* const stage = dws + OFF_RSTAGE + (size_t)buf * RSTAGE_BYTES + lane * 16;
    char* const lines = dws + OFF_RLINE + (size_t)buf * RLINE_BYTES;
    char* const tkp = myws + OFF_TICKET + link * 128;
    WAIT_VM0();
    STAMP(1);
    Packed pc;
    read(j, scr, v);
    pack_unit(v, lane, pc);
    auto send = [&](int u) {
      const int ui = u < nw ? rw0 + (u & 1) * 32 + (u >> 1) : rx0 + u - nw;
      char* const line = lines + (size_t)ui * 128;
      store_pieces(stage + (size_t)ui * 4096, pc);
      store_line(line, pc, pc.meta, call, 0u, lane);
      if (UNLIKELY(!(pc.meta & META_SV))) {
        WAIT_VM0();
        if (lane == 0) st4_wt(line + 16, call);
      }
    };
    send(j);
    int nxt = j + WPL, nn = j + 2 * WPL, sit = 0;
    while (nxt < n) {
      read(nxt, sc1, v);
      unsigned tk = ticket_issue(tkp);
      fetch(nn, sc1);
      pack_unit(v, lane, pc);
      send(nxt);
      asm volatile("s_waitcnt vmcnt(4)" ::: "memory");
      PIN(tk);
      nxt = nn;
      const unsigned d = (unsigned)__builtin_amdgcn_readfirstlane(tk) - (call - 1u) * (unsigned)(n - WPL);
      nn = d < (unsigned)n ? 3 * WPL + (int)d : n + 3 * WPL;
      ++sit;
    }
    STAMP(2);
    STAMP_FLUSH();
    return;
  }

  const int cb = bid - NSEND_WG;
  const int xl = cb & 7, loc = cb >> 3;
  const bool own = loc < NSET_O;
  const int l2 = own ? loc : loc - NSET_O;
  const int nset = own ? NSET_O : NSET_R;
  const int set = (xl >> 1) + 4 * (l2 >> 2);
  const int cg = (own ? 0 : 8) + (xl & 1) * 4 + (l2 & 3);
  unsigned* const ready = (unsigned*)(lds + LDS_RDY);
  if (tid < 16) ready[tid] = 0;
  asm volatile("s_waitcnt lgkmcnt(0)" ::: "memory");
  __builtin_amdgcn_s_barrier();
  if (wv >= 8) {
    const int h = wv - 8;
    unsigned call;
    const int u = h * NCOMP_WG + cb;
    if (u < 256) {
      u32x4 t[4];
      char* const scr = lds + LDS_SCR + h * 4096;
      w_fetch(a.w, u >> 5, u & 31, w_lane_off(lane), scr);
      call = get_call();
      WAIT_VM0();
      w_read(scr, lane, t);
      Packed pk;
      pack_unit(t, lane, pk);
      store_pieces(myws + OFF_LSTAGE + (size_t)(P * 16 + u) * 4096 + lane * 16, pk);
      WAIT_VM0();
      store_line(myws + OFF_LLINE + (size_t)(P * 16 + u) * 128, pk, pk.meta & ~META_SV, call, call, lane);
    } else if (u < 256 + 16 * P) {
      const int xu = u - 256;
      u32x4 t[4];
      char* const scr = lds + LDS_SCR + h * 4096;
      const unsigned xoffs[4] = {x_lane_off(lane, 0), x_lane_off(lane, 1), x_lane_off(lane, 2), x_lane_off(lane, 3)};
      x_fetch(a.x, xu >> 4, xu & 15, xoffs, scr);
      call = get_call();
      WAIT_VM0();
      x_read(scr, lane, t);
      Packed pk;
      pack_unit(t, lane, pk);
      store_pieces(myws + OFF_LSTAGE + (size_t)xu * 4096 + lane * 16, pk);
      WAIT_VM0();
      store_line(myws + OFF_LLINE + (size_t)xu * 128, pk, pk.meta & ~META_SV, call, call, lane);
    } else {
      call = get_call();
    }
    STAMP(1);
    {
      char* const zb = myws + OFF_RSTAGE + (size_t)((call + 2u) % 3u) * RSTAGE_BYTES + lane * 16;
      const u32x4 z = {0u, 0u, 0u, 0u};
      for (int zu = h * NCOMP_WG + cb; zu < NRU; zu += 4 * NCOMP_WG) {
        st16_wt<0>(zb + (size_t)zu * 4096, z);
        st16_wt<1024>(zb + (size_t)zu * 4096, z);
        st16_wt<2048>(zb + (size_t)zu * 4096, z);
      }
    }
    char* const obase = a.obase[cg];
    char* const oflag = a.oflag[cg];
    for (int it = h; set + nset * it < NSLOT; it += 4) {
      const int slot = seq_to_slot(set + nset * it, idx);
      int src, blk;
      if (slot < 4 * P) { const int ia = slot / P; src = 2 * ia + type; blk = slot - ia * P; }
      else { const int s2 = slot - 4 * P; const int ia = s2 / PO; src = 2 * ia + (1 - type); blk = P + s2 - ia * PO; }
      const int row = lane >> 2, m = lane & 3;
      char* const op = obase + (size_t)(src * MLOC + blk * 16 + row) * (NLOC * 2) + m * 16;
      while (__hip_atomic_load(&ready[it], __ATOMIC_RELAXED, __HIP_MEMORY_SCOPE_WORKGROUP) < 8u) __builtin_amdgcn_s_sleep(1);
      __builtin_amdgcn_fence(__ATOMIC_ACQUIRE, "workgroup");
      const char* buf = lds + it * 16384 + row * 128;
      const int sw = ((row & 1) << 2) | (row & 2) | ((row >> 2) & 1);
      f32x4 pl[8], ph[8];
#pragma unroll
      for (int p = 0; p < 8; ++p) {
        pl[p] = *(const f32x4*)(buf + p * 2048 + (((2 * m) ^ sw) * 16));
        ph[p] = *(const f32x4*)(buf + p * 2048 + (((2 * m + 1) ^ sw) * 16));
      }
      __builtin_amdgcn_sched_barrier(0);
      const f32x4 lo = ((pl[0] + pl[1]) + (pl[2] + pl[3])) + ((pl[4] + pl[5]) + (pl[6] + pl[7]));
      const f32x4 hi = ((ph[0] + ph[1]) + (ph[2] + ph[3])) + ((ph[4] + ph[5]) + (ph[6] + ph[7]));
      u32x4 o;
      o[0] = pack_bf16(gelu_f(lo[0]), gelu_f(lo[1]));
      o[1] = pack_bf16(gelu_f(lo[2]), gelu_f(lo[3]));
      o[2] = pack_bf16(gelu_f(hi[0]), gelu_f(hi[1]));
      o[3] = pack_bf16(gelu_f(hi[2]), gelu_f(hi[3]));
      if (cg < 8) {
        *(GAS u32x4*)op = o;
      } else {
        st16_wt<0>(op, o);
        WAIT_VM0();
        if (lane == 0) st8_wt(oflag + (size_t)(slot * 2) * FS * 4, u32x2{call, 0u});
      }
      if (it < 6) STAMP(5 + it * 3);
    }
    STAMP(22);
    STAMP_FLUSH();
    return;
  }
  u64 slotpk = 0;
#pragma unroll
  for (int i = 0; i < MAXB; ++i) {
    const int sl = set + nset * i < NSLOT ? seq_to_slot(set + nset * i, idx) : idx * P;
    const int code = sl >= idx * P && sl < (idx + 1) * P ? 64 + sl - idx * P : sl < idx * P ? sl : sl - P;
    slotpk |= (u64)code << (8 * i);
  }
  const int nblk = (NSLOT - set + nset - 1) / nset;
  const unsigned call = get_call();
  const __amdgpu_buffer_rsrc_t rs = __builtin_amdgcn_make_buffer_rsrc((void*)myws, 0, (int)WS_NEED, 0x00020000);
  const unsigned buf = call % 3u;
  const unsigned rdata = (unsigned)OFF_RSTAGE + buf * (unsigned)RSTAGE_BYTES, rline = (unsigned)OFF_RLINE + buf * (unsigned)RLINE_BYTES;
  auto code_of = [&](int it) -> unsigned { return (unsigned)(slotpk >> (8 * it)) & 127u; };
  auto xdata = [&](unsigned code) -> int {
    return (int)((code < 64u ? rdata + code * 65536u : (unsigned)OFF_LSTAGE + (code - 64u) * 65536u) + (unsigned)wv * 8192u);
  };
  const bool xline = lane < 8, first = (lane & 3) == 0;
  const unsigned xlane = (unsigned)((wv * 2 + (lane >> 2)) * 128 + (lane & 3) * 16);
  auto xoff = [&](unsigned code) -> unsigned {
    const unsigned b = code < 64u ? rline + code * 2048u : (unsigned)OFF_LLINE + (code - 64u) * 2048u;
    return lane < 8 ? b + xlane : lane == 8 ? (unsigned)OFF_TMO : OOB;
  };
  auto ldp = [&](int soff, int piece) -> u32x4 {
    return __builtin_bit_cast(u32x4, __builtin_amdgcn_raw_buffer_load_b128(rs, lane * 16 + piece * 1024, soff, DATA_AUX));
  };
  auto load_x = [&](u32x4 (&af)[2][3], unsigned code) {
    const int so = xdata(code);
#pragma unroll
    for (int h = 0; h < 2; ++h)
#pragma unroll
      for (int p = 0; p < 3; ++p) af[h][p] = ldp(so + h * 4096, p);
  };
  u32x4 pv;
  auto issue = [&](u32x4 (&af)[2][3], int it) {
    load_x(af, code_of(it));
    pv = poll_load(rs, xoff(code_of(it + 1)));
  };
  struct Fl { unsigned meta0, meta1, e0, e1; bool ok; };
  auto take = [&](const u32x4& v, bool ok) -> Fl {
    return Fl{(unsigned)__builtin_amdgcn_readlane((int)v[1], 0), (unsigned)__builtin_amdgcn_readlane((int)v[1], 4), v[2], v[3], ok};
  };
  u32x4 bufA[2][3], bufB[2][3];
  Fl flA, flB = {META_RAW, META_RAW, 0u, 0u, true};
  { const bool ok = wait_lines(rs, xoff(code_of(0)), xline, first, 8, call, tmo, pv); flA = take(pv, ok); }
  STAMP(3);
  load_x(bufA, code_of(0));
  u32x4 wfl;
  const unsigned wunit = (unsigned)((cg & 7) * 32 + wv * 4);
  const unsigned wline = cg < 8 ? (unsigned)OFF_LLINE + ((unsigned)P * 16u + wunit) * 128u : rline + ((unsigned)NRX + wunit) * 128u;
  bool okw = wait_lines(rs, lane < 16 ? wline + (unsigned)((lane >> 2) * 128 + (lane & 3) * 16) : lane == 16 ? (unsigned)OFF_TMO : OOB,
                        lane < 16, first, 16, call, tmo, wfl);
  STAMP(2);
  u32x4 wf[4][4];
  {
    const int soff = (int)(cg < 8 ? (unsigned)OFF_LSTAGE + ((unsigned)P * 16u + wunit) * 4096u : rdata + ((unsigned)NRX + wunit) * 4096u);
    u32x4 wp[4][3];
    const u64 t0 = rt();
    bool pass0 = true;
    for (;;) {
#pragma unroll
      for (int s = 0; s < 4; ++s)
#pragma unroll
        for (int p = 0; p < 3; ++p) wp[s][p] = ldp(soff + s * 4096, p);
      if (pass0) pv = poll_load(rs, xoff(code_of(1)));
      pass0 = false;
#pragma unroll
      for (int s = 0; s < 4; ++s)
#pragma unroll
        for (int p = 0; p < 3; ++p) PIN(wp[s][p]);
      bool bad = false;
#pragma unroll
      for (int s = 0; s < 4; ++s) bad = bad || (((unsigned)__builtin_amdgcn_readlane((int)wfl[1], 4 * s) & META_SV) && min12(wp[s]) == 0u);
      if (LIKELY(!__any(bad))) break;
      if (rt() - t0 > DEADLINE) { __hip_atomic_store((GAS unsigned*)tmo, call, __ATOMIC_RELAXED, __HIP_MEMORY_SCOPE_AGENT); okw = false; break; }
    }
    STAMP(1);
#pragma unroll
    for (int s = 0; s < 4; ++s) {
      const unsigned meta = (unsigned)__builtin_amdgcn_readlane((int)wfl[1], 4 * s);
      if (LIKELY((meta & 0x80000f00u) == 0u)) {
        wf[s][0] = unpack_frag<0, false>(wp[s], meta, 0u, 0u, 0, lane);
        wf[s][1] = unpack_frag<1, false>(wp[s], meta, 0u, 0u, 0, lane);
        wf[s][2] = unpack_frag<2, false>(wp[s], meta, 0u, 0u, 0, lane);
        wf[s][3] = unpack_frag<3, false>(wp[s], meta, 0u, 0u, 0, lane);
      } else if (meta & META_RAW) {
        wf[s][0] = wp[s][0]; wf[s][1] = wp[s][1]; wf[s][2] = wp[s][2]; wf[s][3] = ldp(soff + s * 4096, 3);
      } else {
        wf[s][0] = unpack_frag<0, true>(wp[s], meta, wfl[2], wfl[3], 4 * s, lane);
        wf[s][1] = unpack_frag<1, true>(wp[s], meta, wfl[2], wfl[3], 4 * s, lane);
        wf[s][2] = unpack_frag<2, true>(wp[s], meta, wfl[2], wfl[3], 4 * s, lane);
        wf[s][3] = unpack_frag<3, true>(wp[s], meta, wfl[2], wfl[3], 4 * s, lane);
      }
    }
  }
  STAMP(21);
  auto step = [&](int it, u32x4 (&cur)[2][3], const Fl& cfl, u32x4 (&nxt)[2][3], Fl& nfl) {
    const bool more = it + 1 < nblk;
#pragma unroll
    for (int h = 0; h < 2; ++h)
#pragma unroll
      for (int p = 0; p < 3; ++p) PIN(cur[h][p]);
    PIN(pv);
    bool cok = cfl.ok;
    if (UNLIKELY(__any(((cfl.meta0 & META_SV) && min12(cur[0]) == 0u) || ((cfl.meta1 & META_SV) && min12(cur[1]) == 0u)))) {
      const u64 t0 = rt();
      for (;;) {
        load_x(cur, code_of(it));
        if (!__any(((cfl.meta0 & META_SV) && min12(cur[0]) == 0u) || ((cfl.meta1 & META_SV) && min12(cur[1]) == 0u))) break;
        if (rt() - t0 > DEADLINE) { __hip_atomic_store((GAS unsigned*)tmo, call, __ATOMIC_RELAXED, __HIP_MEMORY_SCOPE_AGENT); cok = false; break; }
      }
    }
    if (it == 1) STAMP(20);
    const bool rdy = more && lines_ready(pv, xline, first, call);
    cok = cok && (unsigned)__builtin_amdgcn_readlane((int)pv[0], 8) != call;
    __builtin_amdgcn_fence(__ATOMIC_ACQUIRE, "wavefront");
    if (rdy) { nfl = take(pv, true); issue(nxt, it + 1); }
    __builtin_amdgcn_sched_barrier(0);
    f32x4 acc[2];
#pragma unroll
    for (int jj = 0; jj < 2; ++jj) acc[jj] = f32x4{0.f, 0.f, 0.f, 0.f};
    auto mm = [&](int h, int s, u32x4 xf) {
#pragma unroll
      for (int jj = 0; jj < 2; ++jj)
        acc[jj] = __builtin_amdgcn_mfma_f32_16x16x32_bf16(__builtin_bit_cast(bf16x8, wf[(4 * h + s) >> 1][(s & 1) * 2 + jj]),
                                                          __builtin_bit_cast(bf16x8, xf), acc[jj], 0, 0, 0);
    };
    const unsigned meta0 = cfl.meta0, meta1 = cfl.meta1;
    if (LIKELY(((meta0 | meta1) & 0x80000f00u) == 0u)) {
#pragma unroll
      for (int h = 0; h < 2; ++h) {
        const unsigned meta = h ? meta1 : meta0;
        mm(h, 0, unpack_frag<0, false>(cur[h], meta, 0u, 0u, 0, lane));
        mm(h, 1, unpack_frag<1, false>(cur[h], meta, 0u, 0u, 0, lane));
        mm(h, 2, unpack_frag<2, false>(cur[h], meta, 0u, 0u, 0, lane));
        mm(h, 3, unpack_frag<3, false>(cur[h], meta, 0u, 0u, 0, lane));
      }
    } else {
#pragma unroll
      for (int h = 0; h < 2; ++h) {
        const unsigned meta = h ? meta1 : meta0;
        if (UNLIKELY(meta & META_RAW)) {
          mm(h, 0, cur[h][0]);
          cur[h][0] = ldp(xdata(code_of(it)) + h * 4096, 3);
          mm(h, 1, cur[h][1]); mm(h, 2, cur[h][2]); mm(h, 3, cur[h][0]);
        } else {
          mm(h, 0, unpack_frag<0, true>(cur[h], meta, cfl.e0, cfl.e1, 4 * h, lane));
          mm(h, 1, unpack_frag<1, true>(cur[h], meta, cfl.e0, cfl.e1, 4 * h, lane));
          mm(h, 2, unpack_frag<2, true>(cur[h], meta, cfl.e0, cfl.e1, 4 * h, lane));
          mm(h, 3, unpack_frag<3, true>(cur[h], meta, cfl.e0, cfl.e1, 4 * h, lane));
        }
      }
    }
    if (UNLIKELY(!(cok && okw))) {
      const float qn = __builtin_nanf("");
#pragma unroll
      for (int jj = 0; jj < 2; ++jj) acc[jj] = f32x4{qn, qn, qn, qn};
    }
    {
      const int m = lane & 15, qd = lane >> 4;
      const int sw = ((m & 1) << 2) | (m & 2) | ((m >> 2) & 1);
      char* buf = lds + it * 16384 + wv * 2048 + m * 128;
#pragma unroll
      for (int jj = 0; jj < 2; ++jj) *(f32x4*)(buf + (((4 * jj + qd) ^ sw) * 16)) = acc[jj];
    }
    if (lane == 0) __hip_atomic_fetch_add(&ready[it], 1u, __ATOMIC_RELEASE, __HIP_MEMORY_SCOPE_WORKGROUP);
    if (it < 6) STAMP(4 + it * 3);
    if (more && !rdy) {
      const bool ok = wait_lines(rs, xoff(code_of(it + 1)), xline, first, 8, call, tmo, pv);
      nfl = take(pv, ok);
      issue(nxt, it + 1);
    }
    if (more && it < 5) STAMP(6 + it * 3);
  };
  for (int it = 0; it < nblk; it += 2) {
    step(it, bufA, flA, bufB, flB);
    if (it + 1 < nblk) step(it + 1, bufB, flB, bufA, flA);
  }
  STAMP(22);
  STAMP_FLUSH();
}

}

extern "C" void kernel_launch(int rank, int nranks, ncclComm_t comm, hipStream_t stream, void* const* d_in, const int* in_sizes,
                              int n_in, void* d_out, int out_size, const int* in_dtypes, int out_dtype, void* const* d_out_all,
                              void* const* d_ws_all, size_t ws_bytes) {
  using namespace agk;
  (void)comm; (void)in_dtypes; (void)out_dtype;
  if (nranks != NRANK || n_in < 2 || rank < 0 || rank >= NRANK || ws_bytes < WS_NEED || in_sizes[0] != MLOC * KDIM ||
      in_sizes[1] != KDIM * NLOC || out_size != MDIM * NLOC) {
    fprintf(stderr, "kernel_launch: unexpected problem shape\n");
    return;
  }
  const int type = rank & 1, idx = rank >> 1;
  Args a;
  memset(&a, 0, sizeof a);
  for (int link = 0; link < 7; ++link) {
    const bool same = link < 3;
    const int didx = same ? link + (link >= idx ? 1 : 0) : link - 3;
    a.link_ws[link] = (char*)d_ws_all[2 * didx + (same ? type : 1 - type)];
  }
  a.link_ws[7] = (char*)d_ws_all[rank];
  for (int cg = 0; cg < 16; ++cg) {
    const int dest = cg < 8 ? rank : 2 * ((cg - 8) >> 1) + (1 - type);
    a.obase[cg] = (char*)d_out_all[dest] + (cg < 8 ? cg * 32 : idx * 64 + (cg & 1) * 32) * 2;
    a.oflag[cg] = (char*)d_ws_all[dest] + OFF_OFLAG + (size_t)(idx * 128 + (cg & 1)) * FS * 4;
  }
  for (int p = 0; p < NRANK; ++p) a.all_ws[p] = (char*)d_ws_all[p];
  a.myws = (char*)d_ws_all[rank];
  a.myout = (char*)d_out;
  a.x = (const char*)d_in[0];
  a.w = (const char*)d_in[1];
  a.rank = rank;
  hipLaunchKernelGGL(ag_gemm_kernel, dim3(NWG), dim3(NTHR), 0, stream, a);
}
```

```cpp
#include <hip/hip_runtime.h>
#include <rccl/rccl.h>
#include <cstdio>
#include <cstdint>
#include <cstring>

namespace agk {

typedef unsigned u32x4 __attribute__((ext_vector_type(4)));
typedef unsigned u32x2 __attribute__((ext_vector_type(2)));
typedef unsigned u32x16 __attribute__((ext_vector_type(16)));
typedef float f32x4 __attribute__((ext_vector_type(4)));
typedef __bf16 bf16x8 __attribute__((ext_vector_type(8)));
typedef unsigned long long u64;
#define GAS __attribute__((address_space(1)))
#define LAS __attribute__((address_space(3)))

constexpr int NRANK = 8;
constexpr int MDIM = 2048, KDIM = 2048, NLOC = 256, MLOC = 256;
#define PBLK 11
constexpr int P = PBLK;
constexpr int PO = 16 - P;
constexpr int NSLOT = 4 * P + 4 * PO;
static_assert(NSLOT == 64, "");
#define POLL_SLEEP 1
#define SWV 2
#define LINKMAP 1
#define DATA_AUX 17
#define ZERO_DELAY 100
#define COL_T0 500
#define COL_DT 80
#define COL_SLEEP 1
#define HELP_SLEEP 1
#define CLK_MASK 0u
#define COL_MASK 0u
#define STATIC_SEND
#define OUT_WT
#define WSTEPS 4
constexpr int FS = 32;

#define WGPL 8
constexpr int NSEND_WG = 7 * WGPL;
constexpr int SW = SWV;
constexpr int WPL = WGPL * SW;
#define NSETO 12
#define NSETR 12
constexpr int NSET_O = NSETO, NSET_R = NSETR;
static_assert(NSET_O <= 16 && NSET_R <= 16, "");
static_assert(NSET_O % 4 == 0 && NSET_R % 4 == 0, "a row block's workgroups sit on one pair of XCD labels");
constexpr int NCOMP_WG = 8 * (NSET_O + NSET_R);
constexpr int NWG = NSEND_WG + NCOMP_WG;
constexpr int NWV = 12;
constexpr int NTHR = NWV * 64;
constexpr int NSET_MIN = NSET_O < NSET_R ? NSET_O : NSET_R;
constexpr int MAXB = (NSLOT + NSET_MIN - 1) / NSET_MIN;
static_assert(NWG <= 256 && MAXB <= 8, "");
static_assert(SW < NWV, "spare sender waves collect the output tiles");
static_assert(P <= NSET_MIN, "only a workgroup's first block can be one of the rank's own");
constexpr int LDS_RDY = 112 << 10;
constexpr int LDS_WSCR = LDS_RDY - 8 * 12288;
static_assert(MAXB * 16384 <= LDS_RDY, "");
constexpr int STAMP_LDS = 0;
constexpr int LDS_BYTES = LDS_RDY + 64 + STAMP_LDS;
static_assert(2 * NWV * 4096 <= LDS_RDY, "");

constexpr size_t OFF_CNT = 0;
constexpr size_t CNT_BYTES = (size_t)NWG * NWV * 32;
constexpr size_t OFF_TMO = 128 << 10;
constexpr size_t OFF_TICKET = OFF_TMO + 1024;
constexpr int NRX = (3 * P + 4 * PO) * 16, NRU = NRX + 256;
constexpr size_t OFF_RLINE = 132 << 10, RLINE_BYTES = (size_t)NRU * 128;
constexpr size_t OFF_OFLAG = OFF_RLINE + 3 * RLINE_BYTES;
constexpr size_t OFF_FLAG_END = OFF_OFLAG + (size_t)512 * FS * 4;
constexpr size_t OFF_RSTAGE = 1 << 20, RSTAGE_BYTES = (size_t)NRU * 4096;
constexpr size_t OFF_OSTAGE = OFF_RSTAGE + 3 * RSTAGE_BYTES;
constexpr size_t WS_NEED = OFF_OSTAGE + (size_t)512 * 1024;
constexpr size_t OFF_DBG = 16 << 20;
static_assert(CNT_BYTES <= OFF_TMO && OFF_FLAG_END <= OFF_RSTAGE && WS_NEED <= OFF_DBG, "");
constexpr u64 DEADLINE = 50000000ull;

constexpr unsigned META_RAW = 0x80000000u;
constexpr unsigned META_SV = 0x40000000u;

struct Args {
  char* link_ws[8];
  char* obase[16];
  char* oflag[16];
  char* otile[16];
  unsigned char orow[4][64];
  char* all_ws[8];
  u64 codes[2][16];
  u64 rows[2][16];
  char* myws;
  char* myout;
  const char* x;
  const char* w;
  int rank;
  int pad;
};

template <int OFF>
__device__ __forceinline__ void st16_wt(char* p, u32x4 v) {
  asm volatile("global_store_dwordx4 %0, %1, off offset:%c2 sc0 sc1\n\ts_nop 1" ::"v"(p), "v"(v), "i"(OFF) : "memory");
}
__device__ __forceinline__ void st4_wt(char* p, unsigned v) {
  asm volatile("global_store_dword %0, %1, off sc0 sc1\n\ts_nop 1" ::"v"(p), "v"(v) : "memory");
}
__device__ __forceinline__ void st8_wt(char* p, u32x2 v) {
  asm volatile("global_store_dwordx2 %0, %1, off sc0 sc1\n\ts_nop 1" ::"v"(p), "v"(v) : "memory");
}
__device__ __forceinline__ unsigned ticket_issue(char* p) {
  unsigned r = 0, one = 1;
  u64 save;
  asm volatile("s_mov_b64 %1, exec\n\ts_mov_b64 exec, 1\n\tglobal_atomic_add %0, %2, %3, off sc0\n\ts_mov_b64 exec, %1"
               : "+v"(r), "=&s"(save) : "v"(p), "v"(one) : "memory");
  return r;
}
#define WAIT_VM0() asm volatile("s_waitcnt vmcnt(0)" ::: "memory")
#define PIN(x) asm volatile("" : "+v"(x))
#define LIKELY(x) __builtin_expect(!!(x), 1)
#define UNLIKELY(x) __builtin_expect(!!(x), 0)

__device__ __forceinline__ u64 rt() { return __builtin_amdgcn_s_memrealtime(); }

constexpr unsigned OOB = 0x7ffffff0u;
__device__ __forceinline__ u32x4 poll_load(__amdgpu_buffer_rsrc_t rc, unsigned voff) {
  return __builtin_bit_cast(u32x4, __builtin_amdgcn_raw_buffer_load_b128(rc, voff, 0, 17));
}
__device__ __forceinline__ bool lines_ready(const u32x4& v, bool line_lane, bool first, unsigned call) {
  const unsigned late = (unsigned)__builtin_amdgcn_update_dpp(0, (int)v[0], 0x101, 0xf, 0xf, false);
  return __all(!line_lane || (first ? v[0] >= call && ((v[1] & META_SV) || late >= call) : v[1] >= call));
}
__device__ __forceinline__ bool wait_lines(__amdgpu_buffer_rsrc_t rc, unsigned voff, bool line_lane, bool first, int tmo_lane, unsigned call, unsigned* tmo, u32x4& fl) {
  const u64 t0 = rt();
  for (unsigned n = 1;; ++n) {
    fl = poll_load(rc, voff);
    const unsigned t = (unsigned)__builtin_amdgcn_readlane((int)fl[0], tmo_lane);
    if (lines_ready(fl, line_lane, first, call)) { __builtin_amdgcn_fence(__ATOMIC_ACQUIRE, "wavefront"); return t != call; }
    if (t == call || ((n & CLK_MASK) == 0u && rt() - t0 > DEADLINE)) {
      __hip_atomic_store((GAS unsigned*)tmo, call, __ATOMIC_RELAXED, __HIP_MEMORY_SCOPE_AGENT);
      fl = u32x4{META_RAW, META_RAW, META_RAW, META_RAW};
      return false;
    }
    __builtin_amdgcn_s_sleep(POLL_SLEEP);
  }
}
__device__ __forceinline__ unsigned umin_(unsigned a, unsigned b) { return a < b ? a : b; }
__device__ __forceinline__ unsigned min12(const u32x4 (&v)[3]) {
  unsigned m = umin_(umin_(v[0][0], v[0][1]), v[0][2]);
  m = umin_(umin_(m, v[0][3]), v[1][0]);
  m = umin_(umin_(m, v[1][1]), v[1][2]);
  m = umin_(umin_(m, v[1][3]), v[2][0]);
  m = umin_(umin_(m, v[2][1]), v[2][2]);
  return umin_(m, v[2][3]);
}

__device__ __forceinline__ unsigned x_lane_off(int lane, int i) {
  const int r = 4 * i + (lane >> 4);
  return (unsigned)(r * (KDIM * 2) + (((lane & 15) ^ r) * 16));
}
__device__ __forceinline__ void x_fetch(const char* x, int blk, int u, const unsigned (&off)[4], char* scratch) {
  const GAS char* base = (const GAS char*)x + (size_t)(blk * 16 * (KDIM * 2) + u * 256);
#pragma unroll
  for (int i = 0; i < 4; ++i) __builtin_amdgcn_global_load_lds((const GAS void*)(base + off[i]), (LAS void*)(scratch + i * 1024), 16, 0, 0);
}
__device__ __forceinline__ void x_read(const char* scratch, int lane, u32x4 (&v)[4]) {
  const int m = lane & 15, q = lane >> 4;
  const char* p = scratch + m * 256;
#pragma unroll
  for (int s = 0; s < 4; ++s) v[s] = *(const LAS u32x4*)(p + (((4 * s + q) ^ m) * 16));
}
__device__ __forceinline__ unsigned w_lane_off(int lane) { return (unsigned)((lane >> 2) * (NLOC * 2) + (lane & 3) * 16); }
__device__ __forceinline__ void w_fetch(const char* w, int cg32, int ks2, unsigned off, char* scratch) {
  const GAS char* base = (const GAS char*)w + (size_t)(ks2 * 64 * (NLOC * 2) + cg32 * 64);
#pragma unroll
  for (int i = 0; i < 4; ++i)
    __builtin_amdgcn_global_load_lds((const GAS void*)(base + i * 16 * (NLOC * 2) + off), (LAS void*)(scratch + i * 1024), 16, 0, 0);
}
__device__ __forceinline__ void w_read(const char* scratch, int lane, u32x4 (&v)[4]) {
  typedef short s16x4 __attribute__((ext_vector_type(4)));
  const int g = lane >> 4, i = lane & 15;
  const char* p = scratch + (8 * g + (i >> 2)) * 64 + (i & 3) * 8;
#pragma unroll
  for (int f = 0; f < 4; ++f)
#pragma unroll
    for (int h = 0; h < 2; ++h) {
      const u32x2 t = __builtin_bit_cast(u32x2, __builtin_amdgcn_ds_read_tr16_b64_v4i16((LAS s16x4*)(p + ((f >> 1) * 32 + 4 * h) * 64 + (f & 1) * 32)));
      v[f][2 * h] = t[0];
      v[f][2 * h + 1] = t[1];
    }
}

__device__ __forceinline__ unsigned lds_off(const char* p) { return (unsigned)(size_t)(const LAS char*)p; }
__device__ __forceinline__ void x_read_asm(const char* scratch, int lane, u32x4 (&v)[4]) {
  const int m = lane & 15, q = lane >> 4;
  const unsigned p = lds_off(scratch) + m * 256;
  asm volatile("ds_read_b128 %0, %4\n\tds_read_b128 %1, %5\n\tds_read_b128 %2, %6\n\tds_read_b128 %3, %7\n\ts_waitcnt lgkmcnt(0)"
               : "=&v"(v[0]), "=&v"(v[1]), "=&v"(v[2]), "=&v"(v[3])
               : "v"(p + ((q ^ m) * 16)), "v"(p + (((4 + q) ^ m) * 16)), "v"(p + (((8 + q) ^ m) * 16)), "v"(p + (((12 + q) ^ m) * 16))
               : "memory");
}
__device__ __forceinline__ void w_read_asm(const char* scratch, int lane, u32x4 (&v)[4]) {
  const int g = lane >> 4, i = lane & 15;
  const unsigned p = lds_off(scratch) + (8 * g + (i >> 2)) * 64 + (i & 3) * 8;
  u32x2 t0, t1, t2, t3, t4, t5, t6, t7;
  asm volatile("ds_read_b64_tr_b16 %0, %8\n\tds_read_b64_tr_b16 %1, %8 offset:256\n\tds_read_b64_tr_b16 %2, %8 offset:32\n\t"
               "ds_read_b64_tr_b16 %3, %8 offset:288\n\tds_read_b64_tr_b16 %4, %8 offset:2048\n\tds_read_b64_tr_b16 %5, %8 offset:2304\n\t"
               "ds_read_b64_tr_b16 %6, %8 offset:2080\n\tds_read_b64_tr_b16 %7, %8 offset:2336\n\ts_waitcnt lgkmcnt(0)"
               : "=&v"(t0), "=&v"(t1), "=&v"(t2), "=&v"(t3), "=&v"(t4), "=&v"(t5), "=&v"(t6), "=&v"(t7) : "v"(p) : "memory");
  v[0] = u32x4{t0[0], t0[1], t1[0], t1[1]};
  v[1] = u32x4{t2[0], t2[1], t3[0], t3[1]};
  v[2] = u32x4{t4[0], t4[1], t5[0], t5[1]};
  v[3] = u32x4{t6[0], t6[1], t7[0], t7[1]};
}

__device__ __forceinline__ unsigned umax_(unsigned a, unsigned b) { return a > b ? a : b; }
__device__ __forceinline__ unsigned wave_max(unsigned v) {
  v = umax_(v, (unsigned)__builtin_amdgcn_update_dpp(0, (int)v, 0x111, 0xf, 0xf, false));
  v = umax_(v, (unsigned)__builtin_amdgcn_update_dpp(0, (int)v, 0x112, 0xf, 0xf, false));
  v = umax_(v, (unsigned)__builtin_amdgcn_update_dpp(0, (int)v, 0x114, 0xf, 0xf, false));
  v = umax_(v, (unsigned)__builtin_amdgcn_update_dpp(0, (int)v, 0x118, 0xf, 0xf, false));
  v = umax_(v, (unsigned)__builtin_amdgcn_update_dpp(0, (int)v, 0x142, 0xa, 0xf, false));
  v = umax_(v, (unsigned)__builtin_amdgcn_update_dpp(0, (int)v, 0x143, 0xc, 0xf, false));
  return (unsigned)__builtin_amdgcn_readlane((int)v, 63);
}
typedef unsigned short u16x2 __attribute__((ext_vector_type(2)));
__device__ __forceinline__ unsigned pk_max_u16(unsigned a, unsigned b) {
  return __builtin_bit_cast(unsigned, __builtin_elementwise_max(__builtin_bit_cast(u16x2, a), __builtin_bit_cast(u16x2, b)));
}
struct Packed { u32x4 s[4]; u32x2 ent; unsigned meta; };
__device__ __forceinline__ unsigned bfi(unsigned mask, unsigned a, unsigned b) {
  unsigned r;
  asm("v_bfi_b32 %0, %1, %2, %3" : "=v"(r) : "s"(mask), "v"(a), "v"(b));
  return r;
}
__device__ __forceinline__ void pack_unit(const u32x4 (&f)[4], int lane, Packed& o) {
  u32x4 L0, L1, C;
  unsigned hq[8], uq[8];
#pragma unroll
  for (int i = 0; i < 4; ++i) {
    hq[2 * i] = __builtin_amdgcn_perm(f[i][1], f[i][0], 0x07050301u);
    hq[2 * i + 1] = __builtin_amdgcn_perm(f[i][3], f[i][2], 0x07050301u);
    uq[2 * i] = hq[2 * i] | 0x80808080u;
    uq[2 * i + 1] = hq[2 * i + 1] | 0x80808080u;
  }
  const unsigned mo = pk_max_u16(pk_max_u16(pk_max_u16(uq[0], uq[1]), pk_max_u16(uq[2], uq[3])), pk_max_u16(pk_max_u16(uq[4], uq[5]), pk_max_u16(uq[6], uq[7])));
  const unsigned me = pk_max_u16(pk_max_u16(pk_max_u16(uq[0] << 8, uq[1] << 8), pk_max_u16(uq[2] << 8, uq[3] << 8)),
                                 pk_max_u16(pk_max_u16(uq[4] << 8, uq[5] << 8), pk_max_u16(uq[6] << 8, uq[7] << 8)));
  const unsigned m2 = pk_max_u16(mo, me);
  const unsigned mx = (wave_max(umax_(m2 & 0xffffu, m2 >> 16)) >> 8) & 0x7fu;
  const unsigned base = (mx > 7u ? mx : 7u) - 7u;
  const unsigned base4 = base * 0x01010101u;
  unsigned E[4];
#pragma unroll
  for (int i = 0; i < 4; ++i) {
    const unsigned l0 = __builtin_amdgcn_perm(f[i][1], f[i][0], 0x06040200u), l1 = __builtin_amdgcn_perm(f[i][3], f[i][2], 0x06040200u);
    const unsigned t0 = uq[2 * i] - base4, t1 = uq[2 * i + 1] - base4;
    E[i] = t0 & t1;
    C[i] = bfi(0x0f0f0f0fu, bfi(0x07070707u, t0, hq[2 * i] >> 4), bfi(0x70707070u, t1 << 4, hq[2 * i + 1]));
    if (i < 2) { L0[2 * i] = l0; L0[2 * i + 1] = l1; } else { L1[2 * i - 4] = l0; L1[2 * i - 3] = l1; }
  }
  unsigned fmask = 0;
  bool raw = false;
  u32x2 ent = {0u, 0u};
  if (UNLIKELY(__any((E[0] & E[1] & E[2] & E[3] & 0x80808080u) != 0x80808080u))) {
#pragma unroll
    for (int i = 0; i < 4; ++i) {
      u64 bm = __ballot((E[i] & 0x80808080u) != 0x80808080u);
      unsigned cnt = 0, e0 = 0, e1 = 0;
      while (bm && cnt <= 2u) {
        const int l = __builtin_ctzll(bm);
        bm &= bm - 1;
#pragma unroll
        for (int q = 0; q < 2; ++q) {
          const unsigned h = (unsigned)__builtin_amdgcn_readlane((int)hq[2 * i + q], l);
          unsigned m2 = ~((h | 0x80808080u) - base4) & 0x80808080u;
          while (m2) {
            const unsigned b = (unsigned)__builtin_ctz(m2) >> 3;
            m2 &= m2 - 1;
            const unsigned e = 0x80000000u | ((unsigned)l << 16) | ((4u * q + b) << 8) | ((h >> (8u * b)) & 0xffu);
            e1 = cnt == 1u ? e : e1;
            e0 = cnt == 0u ? e : e0;
            ++cnt;
          }
        }
      }
      raw = raw || cnt > 2u;
      fmask |= cnt ? 1u << i : 0u;
      ent[0] = lane == i ? e0 : ent[0];
      ent[1] = lane == i ? e1 : ent[1];
    }
  }
  o.ent = ent;
  if (UNLIKELY(raw)) {
    o.s[0] = f[0]; o.s[1] = f[1]; o.s[2] = f[2]; o.s[3] = f[3];
    o.meta = META_RAW;
  } else {
    o.s[0] = L0; o.s[1] = L1; o.s[2] = C; o.s[3] = C;
    const u32x4 t[3] = {L0, L1, C};
    o.meta = (fmask << 8) | base | (__all(min12(t) != 0u) ? META_SV : 0u);
  }
}
__device__ __forceinline__ void store_pieces(char* dst, const Packed& p) {
  st16_wt<0>(dst, p.s[0]);
  st16_wt<1024>(dst, p.s[1]);
  st16_wt<2048>(dst, p.s[2]);
  if (UNLIKELY(p.meta & META_RAW)) st16_wt<3072>(dst, p.s[3]);
}
__device__ __forceinline__ void store_line(char* line, const Packed& p, unsigned meta, unsigned call, unsigned late, int lane) {
  const u32x4 v = {lane == 0 ? call : lane == 1 ? late : 0u, lane == 0 ? meta : call, p.ent[0], p.ent[1]};
  if (lane < 4) st16_wt<0>(line + lane * 16, v);
}
__device__ __forceinline__ void patch_frag(u32x4& o, unsigned e, int lane) {
  const unsigned sh = 8u + 16u * ((e >> 8) & 1u), d = (e >> 9) & 3u;
  const unsigned vm = (unsigned)lane == ((e >> 16) & 63u) ? 0xffu << sh : 0u, put = (e & 0xffu) << sh;
#pragma unroll
  for (int t = 0; t < 4; ++t) {
    const unsigned m = d == (unsigned)t ? vm : 0u;
    o[t] = (o[t] & ~m) | (put & m);
  }
}
template <int I, bool EXC>
__device__ __forceinline__ u32x4 unpack_frag(const u32x4 (&v)[3], unsigned meta, unsigned e0, unsigned e1, int fb, int lane) {
  const unsigned base4 = (meta & 0x7fu) * 0x01010101u;
  const unsigned l0 = v[I >> 1][2 * (I & 1)], l1 = v[I >> 1][2 * (I & 1) + 1];
  const unsigned c = v[2][I];
  const unsigned h0 = ((c & 0x08080808u) << 4) | ((c & 0x07070707u) + base4);
  const unsigned h1 = (c & 0x80808080u) | (((c >> 4) & 0x07070707u) + base4);
  u32x4 o;
  o[0] = __builtin_amdgcn_perm(h0, l0, 0x05010400u);
  o[1] = __builtin_amdgcn_perm(h0, l0, 0x07030602u);
  o[2] = __builtin_amdgcn_perm(h1, l1, 0x05010400u);
  o[3] = __builtin_amdgcn_perm(h1, l1, 0x07030602u);
  if (EXC && (meta & (0x100u << I))) {
    patch_frag(o, (unsigned)__builtin_amdgcn_readlane((int)e0, fb + I), lane);
    const unsigned e = (unsigned)__builtin_amdgcn_readlane((int)e1, fb + I);
    if (e) patch_frag(o, e, lane);
  }
  return o;
}

__device__ __forceinline__ float gelu_f(float y) {
  const float c = 0.7978845608028654f;
  const float k2 = -2.0f * 1.4426950408889634f * c;
  const float t = y * (1.0f + 0.044715f * y * y);
  const float e = __builtin_amdgcn_exp2f(k2 * t);
  return y * __builtin_amdgcn_rcpf(1.0f + e);
}
__device__ __forceinline__ unsigned pack_bf16(float lo, float hi) {
  typedef __bf16 bf2 __attribute__((ext_vector_type(2)));
  typedef float f2 __attribute__((ext_vector_type(2)));
  f2 f = {lo, hi};
  return __builtin_bit_cast(unsigned, __builtin_convertvector(f, bf2));
}

#define STAMP(idx) do {} while (0)
#define STAMP_FLUSH() do {} while (0)
#define STAMP2(idx) do {} while (0)

static inline int seq_to_slot(int q, int idx_self) {
  if (q < P) return idx_self * P + q;
  q -= P;
  constexpr int WST = WSTEPS;
  for (int s = 0; s < 32; ++s) {
    if (s < P) {
      if (q < 3) { int ia = q + (q >= idx_self ? 1 : 0); return ia * P + s; }
      q -= 3;
    }
    const int j = s - WST;
    if (j >= 0 && j < PO) {
      if (q < 4) return 4 * P + q * PO + j;
      q -= 4;
    }
  }
  return 0;
}

__global__ void __launch_bounds__(NTHR, 3) ag_gemm_kernel(Args a) {
  __shared__ __attribute__((aligned(16))) char lds[LDS_BYTES];
  const int tid = threadIdx.x, lane = tid & 63;
  const int wv = __builtin_amdgcn_readfirstlane(tid >> 6);
  const int bid = blockIdx.x;
  const int rank = a.rank, type = rank & 1, idx = rank >> 1;
  char* const myws = a.myws;
  unsigned* const tmo = (unsigned*)(myws + OFF_TMO);
  STAMP(0);
  auto get_call = [&]() -> unsigned {
    asm volatile("" ::: "memory");
    unsigned ticket = 0;
    if (lane == 0)
      ticket = __hip_atomic_fetch_add((GAS unsigned*)(myws + OFF_CNT + (size_t)(bid * NWV + wv) * 32), 1u, __ATOMIC_RELAXED, __HIP_MEMORY_SCOPE_AGENT);
    return (unsigned)__builtin_amdgcn_readfirstlane(ticket) + 1u;
  };

  if (bid < NSEND_WG) {
    const int link = bid % 7, lq = bid / 7;
    if (wv >= SW) {
      const u64 tstart = rt();
      const unsigned call = get_call();
      for (int o = bid * (NWV - SW) + wv - SW; o < 512; o += NSEND_WG * (NWV - SW)) {
        const int q = o >> 3, src = (o >> 1) & 3, half = o & 1;
        const int t = src * 128 + q * 2 + half;
        const __amdgpu_buffer_rsrc_t ro = __builtin_amdgcn_make_buffer_rsrc((void*)myws, 0, (int)WS_NEED, 0x00020000);
        while (rt() - tstart < (u64)(COL_T0 + q * COL_DT / 8)) __builtin_amdgcn_s_sleep(8);
        const u64 t0 = rt();
        u32x4 v;
        bool good = true;
        for (unsigned n = 1;; ++n) {
          v = __builtin_bit_cast(u32x4, __builtin_amdgcn_raw_buffer_load_b128(ro, lane * 16, (int)(OFF_OSTAGE + (size_t)t * 1024), 17));
          const unsigned tm = __hip_atomic_load((GAS unsigned*)tmo, __ATOMIC_RELAXED, __HIP_MEMORY_SCOPE_AGENT);
          if (__all(umin_(umin_(v[0], v[1]), umin_(v[2], v[3])) != 0u)) { good = tm != call; break; }
          if ((n & COL_MASK) == 0u) {
            const unsigned f = __hip_atomic_load((const GAS unsigned*)(myws + OFF_OFLAG + (size_t)t * FS * 4), __ATOMIC_RELAXED, __HIP_MEMORY_SCOPE_SYSTEM);
            if (f >= call) {
              __builtin_amdgcn_fence(__ATOMIC_ACQUIRE, "wavefront");
              v = __builtin_bit_cast(u32x4, __builtin_amdgcn_raw_buffer_load_b128(ro, lane * 16, (int)(OFF_OSTAGE + (size_t)t * 1024), 17));
              good = tm != call;
              break;
            }
            if (tm == call || rt() - t0 > DEADLINE) {
              __hip_atomic_store((GAS unsigned*)tmo, call, __ATOMIC_RELAXED, __HIP_MEMORY_SCOPE_AGENT);
              good = false;
              break;
            }
          }
          __builtin_amdgcn_s_sleep(COL_SLEEP);
        }
        if (UNLIKELY(!good)) v = u32x4{0x7fc07fc0u, 0x7fc07fc0u, 0x7fc07fc0u, 0x7fc07fc0u};
        st16_wt<0>(a.myout + (size_t)(a.orow[src][q] * 16 + (lane >> 2)) * (NLOC * 2) + (src * 64 + half * 32) * 2 + (lane & 3) * 16, v);
        PIN(v);
        st16_wt<0>(myws + OFF_OSTAGE + (size_t)t * 1024 + lane * 16, u32x4{0u, 0u, 0u, 0u});
      }
      STAMP(23);
      STAMP_FLUSH();
      return;
    }
    const int j = lq * SW + wv;
    const bool same = link < 3;
    const int didx = same ? link + (link >= idx ? 1 : 0) : link - 3;
    char* const dws = a.link_ws[link];
    u32x4 v[4];
    const int b0 = same ? 0 : P, nb = same ? P : PO;
    const int rx0 = (same ? (idx < didx ? idx : idx - 1) * P : 3 * P + idx * PO) * 16, rw0 = NRX + idx * 64;
    char* const scr = lds + wv * 4096;
    const int nw = same ? 0 : 64, n = nw + nb * 16;
    char* const sc1 = lds + NWV * 4096 + wv * 4096;
    const unsigned xoffs[4] = {x_lane_off(lane, 0), x_lane_off(lane, 1), x_lane_off(lane, 2), x_lane_off(lane, 3)};
    const unsigned woff = w_lane_off(lane);
    auto fetch = [&](int u, char* sc) {
      u = u < n ? u : j;
      if (u < nw) w_fetch(a.w, didx * 2 + (u & 1), u >> 1, woff, sc);
      else x_fetch(a.x, b0 + ((u - nw) >> 4), (u - nw) & 15, xoffs, sc);
    };
    auto read = [&](int u, const char* sc, u32x4 (&f)[4]) {
      u = u < n ? u : j;
      if (u < nw) w_read_asm(sc, lane, f); else x_read_asm(sc, lane, f);
    };
    __builtin_amdgcn_s_waitcnt(0x0F70);
    fetch(j, scr);
    fetch(j + WPL, sc1);
    const unsigned call = get_call();
    const unsigned buf = call % 3u;
    char* const stage = dws + OFF_RSTAGE + (size_t)buf * RSTAGE_BYTES + lane * 16;
    char* const lines = dws + OFF_RLINE + (size_t)buf * RLINE_BYTES;
    char* const tkp = myws + OFF_TICKET + link * 128;
    WAIT_VM0();
    STAMP(1);
    Packed pc;
    read(j, scr, v);
    pack_unit(v, lane, pc);
    auto send = [&](int u) {
      const int ui = u < nw ? rw0 + (u & 1) * 32 + (u >> 1) : rx0 + u - nw;
      char* const line = lines + (size_t)ui * 128;
      store_pieces(stage + (size_t)ui * 4096, pc);
      store_line(line, pc, pc.meta, call, 0u, lane);
      if (UNLIKELY(!(pc.meta & META_SV))) {
        WAIT_VM0();
        if (lane == 0) st4_wt(line + 16, call);
      }
    };
    fetch(j + 2 * WPL, scr);
    send(j);
    int sit = 0;
    char* sa = sc1;
    char* sb = scr;
    for (int u = j + WPL; u < n; u += WPL) {
      read(u, sa, v);
      fetch(u + 2 * WPL, sa);
      pack_unit(v, lane, pc);
      send(u);
      asm volatile("s_waitcnt vmcnt(8)" ::: "memory");
      char* t = sa; sa = sb; sb = t;
      ++sit;
    }
    (void)tkp;
    STAMP(2);
    STAMP_FLUSH();
    return;
  }

  const int cb = bid - NSEND_WG;
  const int xl = cb & 7, loc = cb >> 3;
  const bool own = loc < NSET_O;
  const int l2 = own ? loc : loc - NSET_O;
  const int nset = own ? NSET_O : NSET_R;
  const int set = (xl >> 1) + 4 * (l2 >> 2);
  const int cg = (own ? 0 : 8) + (xl & 1) * 4 + (l2 & 3);
  unsigned* const ready = (unsigned*)(lds + LDS_RDY);
  if (tid < 16) ready[tid] = 0;
  asm volatile("s_waitcnt lgkmcnt(0)" ::: "memory");
  __builtin_amdgcn_s_barrier();
  if (wv >= 8) {
    const int h = wv - 8;
    const unsigned call = get_call();
    STAMP(1);
    {
      __builtin_amdgcn_s_sleep(ZERO_DELAY);
      char* const zb = myws + OFF_RSTAGE + (size_t)((call + 2u) % 3u) * RSTAGE_BYTES + lane * 16;
      const u32x4 z = {0u, 0u, 0u, 0u};
      for (int zu = h * NCOMP_WG + cb; zu < NRU; zu += 4 * NCOMP_WG) {
        st16_wt<0>(zb + (size_t)zu * 4096, z);
        st16_wt<1024>(zb + (size_t)zu * 4096, z);
        st16_wt<2048>(zb + (size_t)zu * 4096, z);
      }
    }
    char* const obase = a.obase[cg];
    char* const oflag = a.oflag[cg];
    char* const otile = a.otile[cg];
    const u64 rows = a.rows[own ? 0 : 1][set];
    for (int it = h; set + nset * it < NSLOT; it += 4) {
      const int gblk = (int)(rows >> (8 * it)) & 127;
      const int row = lane >> 2, m = lane & 3;
      char* const op = obase + (size_t)(gblk * 16 + row) * (NLOC * 2) + m * 16;
      while (__hip_atomic_load(&ready[it], __ATOMIC_RELAXED, __HIP_MEMORY_SCOPE_WORKGROUP) < 8u) __builtin_amdgcn_s_sleep(HELP_SLEEP);
      __builtin_amdgcn_fence(__ATOMIC_ACQUIRE, "workgroup");
      const char* buf = lds + it * 16384 + row * 128;
      const int sw = ((row & 1) << 2) | (row & 2) | ((row >> 2) & 1);
      f32x4 pl[8], ph[8];
#pragma unroll
      for (int p = 0; p < 8; ++p) {
        pl[p] = *(const f32x4*)(buf + p * 2048 + (((2 * m) ^ sw) * 16));
        ph[p] = *(const f32x4*)(buf + p * 2048 + (((2 * m + 1) ^ sw) * 16));
      }
      __builtin_amdgcn_sched_barrier(0);
      const f32x4 lo = ((pl[0] + pl[1]) + (pl[2] + pl[3])) + ((pl[4] + pl[5]) + (pl[6] + pl[7]));
      const f32x4 hi = ((ph[0] + ph[1]) + (ph[2] + ph[3])) + ((ph[4] + ph[5]) + (ph[6] + ph[7]));
      u32x4 o;
      o[0] = pack_bf16(gelu_f(lo[0]), gelu_f(lo[1]));
      o[1] = pack_bf16(gelu_f(lo[2]), gelu_f(lo[3]));
      o[2] = pack_bf16(gelu_f(hi[0]), gelu_f(hi[1]));
      o[3] = pack_bf16(gelu_f(hi[2]), gelu_f(hi[3]));
      if (cg < 8) {
        st16_wt<0>(op, o);
      } else {
        st16_wt<0>(otile + (size_t)((set + nset * it) * 2) * 1024 + lane * 16, o);
        if (UNLIKELY(!__all(umin_(umin_(o[0], o[1]), umin_(o[2], o[3])) != 0u))) {
          WAIT_VM0();
          if (lane == 0) st8_wt(oflag + (size_t)((set + nset * it) * 2) * FS * 4, u32x2{call, 0u});
        }
      }
      if (it < 6) STAMP(5 + it * 3);
    }
    STAMP(22);
    STAMP_FLUSH();
    return;
  }
  const bool own0 = set < P;
  char* const wscr = lds + LDS_WSCR + wv * 12288;
  const unsigned xoffs[4] = {x_lane_off(lane, 0), x_lane_off(lane, 1), x_lane_off(lane, 2), x_lane_off(lane, 3)};
  auto own_fetch = [&](int r) {
    if (own) {
      w_fetch(a.w, cg, wv * 4 + 2 * r, w_lane_off(lane), wscr);
      w_fetch(a.w, cg, wv * 4 + 2 * r + 1, w_lane_off(lane), wscr + 4096);
    }
    if (own0) x_fetch(a.x, set, wv * 2 + r, xoffs, wscr + 8192);
  };
  own_fetch(0);
  const u64 slotpk = a.codes[own ? 0 : 1][set];
  const int nblk = own ? (NSLOT - set + NSET_O - 1) / NSET_O : (NSLOT - set + NSET_R - 1) / NSET_R;
  const unsigned call = get_call();
  const __amdgpu_buffer_rsrc_t rs = __builtin_amdgcn_make_buffer_rsrc((void*)myws, 0, (int)WS_NEED, 0x00020000);
  const unsigned buf = call % 3u;
  const unsigned rdata = (unsigned)OFF_RSTAGE + buf * (unsigned)RSTAGE_BYTES, rline = (unsigned)OFF_RLINE + buf * (unsigned)RLINE_BYTES;
  auto code_of = [&](int it) -> unsigned { return (unsigned)(slotpk >> (8 * it)) & 63u; };
  auto xdata = [&](unsigned code) -> int { return (int)(rdata + code * 65536u + (unsigned)wv * 8192u); };
  const bool xline = lane < 8, first = (lane & 3) == 0;
  const unsigned xlane = (unsigned)((wv * 2 + (lane >> 2)) * 128 + (lane & 3) * 16);
  auto xoff = [&](unsigned code) -> unsigned { return lane < 8 ? rline + code * 2048u + xlane : lane == 8 ? (unsigned)OFF_TMO : OOB; };
  auto ldp = [&](int soff, int piece) -> u32x4 {
    return __builtin_bit_cast(u32x4, __builtin_amdgcn_raw_buffer_load_b128(rs, lane * 16 + piece * 1024, soff, DATA_AUX));
  };
  auto load_x = [&](u32x4 (&af)[2][3], unsigned code) {
    const int so = xdata(code);
#pragma unroll
    for (int h = 0; h < 2; ++h)
#pragma unroll
      for (int p = 0; p < 3; ++p) af[h][p] = ldp(so + h * 4096, p);
  };
  auto issue = [&](u32x4 (&af)[2][3], u32x4& cv, int it) {
    cv = poll_load(rs, xoff(code_of(it)));
    load_x(af, code_of(it));
  };
  u32x4 wf[4][4];
  u32x4 x0[2][4];
  bool okw = true;
#pragma unroll
  for (int r = 0; r < 2; ++r) {
    WAIT_VM0();
    if (own) {
      w_read(wscr, lane, wf[2 * r]);
      w_read(wscr + 4096, lane, wf[2 * r + 1]);
#pragma unroll
      for (int f = 0; f < 4; ++f) { PIN(wf[2 * r][f]); PIN(wf[2 * r + 1][f]); }
    }
    if (own0) {
      x_read(wscr + 8192, lane, x0[r]);
#pragma unroll
      for (int f = 0; f < 4; ++f) PIN(x0[r][f]);
    }
    if (r == 0) own_fetch(1);
  }
  constexpr int WSCR_IT = LDS_WSCR / 16384;
  if (lane == 0) __hip_atomic_fetch_add(&ready[8], 1u, __ATOMIC_RELEASE, __HIP_MEMORY_SCOPE_WORKGROUP);
  u32x4 wfl = {0u, 0u, 0u, 0u};
  const unsigned wunit = (unsigned)(NRX + (cg - 8) * 32 + wv * 4);
  if (!own)
    okw = wait_lines(rs, lane < 16 ? rline + wunit * 128u + (unsigned)((lane >> 2) * 128 + (lane & 3) * 16) : lane == 16 ? (unsigned)OFF_TMO : OOB,
                     lane < 16, first, 16, call, tmo, wfl);
  const int it0 = own0 ? 1 : 0;
  u32x4 bufA[2][3], bufB[2][3], cvA, cvB;
  issue(bufA, cvA, it0);
  STAMP(2);
  if (!own) {
    const int soff = (int)(rdata + wunit * 4096u);
    u32x4 wp[4][3];
    const u64 t0 = rt();
    for (;;) {
#pragma unroll
      for (int s = 0; s < 4; ++s)
#pragma unroll
        for (int p = 0; p < 3; ++p) wp[s][p] = ldp(soff + s * 4096, p);
#pragma unroll
      for (int s = 0; s < 4; ++s)
#pragma unroll
        for (int p = 0; p < 3; ++p) PIN(wp[s][p]);
      bool bad = false;
#pragma unroll
      for (int s = 0; s < 4; ++s) bad = bad || (((unsigned)__builtin_amdgcn_readlane((int)wfl[1], 4 * s) & META_SV) && min12(wp[s]) == 0u);
      if (LIKELY(!__any(bad))) break;
      if (rt() - t0 > DEADLINE) { __hip_atomic_store((GAS unsigned*)tmo, call, __ATOMIC_RELAXED, __HIP_MEMORY_SCOPE_AGENT); okw = false; break; }
    }
    STAMP2(1);
#pragma unroll
    for (int s = 0; s < 4; ++s) {
      const unsigned meta = (unsigned)__builtin_amdgcn_readlane((int)wfl[1], 4 * s);
      if (LIKELY((meta & 0x80000f00u) == 0u)) {
        wf[s][0] = unpack_frag<0, false>(wp[s], meta, 0u, 0u, 0, lane);
        wf[s][1] = unpack_frag<1, false>(wp[s], meta, 0u, 0u, 0, lane);
        wf[s][2] = unpack_frag<2, false>(wp[s], meta, 0u, 0u, 0, lane);
        wf[s][3] = unpack_frag<3, false>(wp[s], meta, 0u, 0u, 0, lane);
      } else if (meta & META_RAW) {
        wf[s][0] = wp[s][0]; wf[s][1] = wp[s][1]; wf[s][2] = wp[s][2]; wf[s][3] = ldp(soff + s * 4096, 3);
      } else {
        wf[s][0] = unpack_frag<0, true>(wp[s], meta, wfl[2], wfl[3], 4 * s, lane);
        wf[s][1] = unpack_frag<1, true>(wp[s], meta, wfl[2], wfl[3], 4 * s, lane);
        wf[s][2] = unpack_frag<2, true>(wp[s], meta, wfl[2], wfl[3], 4 * s, lane);
        wf[s][3] = unpack_frag<3, true>(wp[s], meta, wfl[2], wfl[3], 4 * s, lane);
      }
    }
  }
  STAMP2(21);
  f32x4 acc[2];
  auto mm = [&](int ks, u32x4 xf) {
#pragma unroll
    for (int jj = 0; jj < 2; ++jj)
      acc[jj] = __builtin_amdgcn_mfma_f32_16x16x32_bf16(__builtin_bit_cast(bf16x8, wf[ks >> 1][(ks & 1) * 2 + jj]), __builtin_bit_cast(bf16x8, xf), acc[jj], 0, 0, 0);
  };
  int pw[2];
  {
    const int m = lane & 15, qd = lane >> 4;
    const int sw = ((m & 1) << 2) | (m & 2) | ((m >> 2) & 1);
#pragma unroll
    for (int jj = 0; jj < 2; ++jj) pw[jj] = wv * 2048 + m * 128 + (((4 * jj + qd) ^ sw) * 16);
  }
  bool scr_free = false;
  auto finish = [&](int it, bool good) {
    if (UNLIKELY(!good)) {
      asm volatile("; poison" ::: "memory");
      const float qn = __builtin_nanf("");
#pragma unroll
      for (int jj = 0; jj < 2; ++jj) acc[jj] = f32x4{qn, qn, qn, qn};
    }
    if (UNLIKELY(it >= WSCR_IT && !scr_free)) {
      while (__hip_atomic_load(&ready[8], __ATOMIC_RELAXED, __HIP_MEMORY_SCOPE_WORKGROUP) < 8u) __builtin_amdgcn_s_sleep(1);
      scr_free = true;
    }
    char* pb = lds + it * 16384;
#pragma unroll
    for (int jj = 0; jj < 2; ++jj) *(f32x4*)(pb + pw[jj]) = acc[jj];
    if (lane == 0) __hip_atomic_fetch_add(&ready[it], 1u, __ATOMIC_RELEASE, __HIP_MEMORY_SCOPE_WORKGROUP);
  };
  if (own0) {
#pragma unroll
    for (int jj = 0; jj < 2; ++jj) acc[jj] = f32x4{0.f, 0.f, 0.f, 0.f};
#pragma unroll
    for (int s = 0; s < 8; ++s) mm(s, x0[s >> 2][s & 3]);
    finish(0, okw);
    STAMP(4);
  }
  auto step = [&](int it, u32x4 (&cur)[2][3], u32x4& cv, u32x4 (&nxt)[2][3], u32x4& ncv) {
#pragma unroll
    for (int h = 0; h < 2; ++h)
#pragma unroll
      for (int p = 0; p < 3; ++p) PIN(cur[h][p]);
    PIN(cv);
    bool cok = true;
    {
      auto complete = [&]() -> bool {
        const unsigned m0 = (unsigned)__builtin_amdgcn_readlane((int)cv[1], 0), m1 = (unsigned)__builtin_amdgcn_readlane((int)cv[1], 4);
        return lines_ready(cv, xline, first, call) && (m0 & m1 & META_SV) && __all(umin_(min12(cur[0]), min12(cur[1])) != 0u);
      };
      bool have = complete();
      if (!have) {
        cok = wait_lines(rs, xoff(code_of(it)), xline, first, 8, call, tmo, cv);
        const unsigned n0 = (unsigned)__builtin_amdgcn_readlane((int)cv[1], 0), n1 = (unsigned)__builtin_amdgcn_readlane((int)cv[1], 4);
        const u64 t0 = rt();
        for (;;) {
          load_x(cur, code_of(it));
          if (!__any(((n0 & META_SV) && min12(cur[0]) == 0u) || ((n1 & META_SV) && min12(cur[1]) == 0u))) break;
          if (rt() - t0 > DEADLINE) { __hip_atomic_store((GAS unsigned*)tmo, call, __ATOMIC_RELAXED, __HIP_MEMORY_SCOPE_AGENT); cok = false; break; }
        }
      }
    }
    if (it + 1 == nblk) STAMP2(20);
    cok = cok && (unsigned)__builtin_amdgcn_readlane((int)cv[0], 8) != call;
    const unsigned meta0 = (unsigned)__builtin_amdgcn_readlane((int)cv[1], 0), meta1 = (unsigned)__builtin_amdgcn_readlane((int)cv[1], 4);
    __builtin_amdgcn_fence(__ATOMIC_ACQUIRE, "wavefront");
    issue(nxt, ncv, it + 1);
    __builtin_amdgcn_sched_barrier(0);
#pragma unroll
    for (int jj = 0; jj < 2; ++jj) acc[jj] = f32x4{0.f, 0.f, 0.f, 0.f};
    if (LIKELY(((meta0 | meta1) & 0x80000f00u) == 0u)) {
#pragma unroll
      for (int h = 0; h < 2; ++h) {
        const unsigned meta = h ? meta1 : meta0;
        mm(4 * h + 0, unpack_frag<0, false>(cur[h], meta, 0u, 0u, 0, lane));
        mm(4 * h + 1, unpack_frag<1, false>(cur[h], meta, 0u, 0u, 0, lane));
        mm(4 * h + 2, unpack_frag<2, false>(cur[h], meta, 0u, 0u, 0, lane));
        mm(4 * h + 3, unpack_frag<3, false>(cur[h], meta, 0u, 0u, 0, lane));
      }
    } else {
#pragma unroll
      for (int h = 0; h < 2; ++h) {
        const unsigned meta = h ? meta1 : meta0;
        if (UNLIKELY(meta & META_RAW)) {
          mm(4 * h, cur[h][0]);
          cur[h][0] = ldp(xdata(code_of(it)) + h * 4096, 3);
          mm(4 * h + 1, cur[h][1]); mm(4 * h + 2, cur[h][2]);
          PIN(cur[h][0]);
          mm(4 * h + 3, cur[h][0]);
        } else {
          mm(4 * h + 0, unpack_frag<0, true>(cur[h], meta, cv[2], cv[3], 4 * h, lane));
          mm(4 * h + 1, unpack_frag<1, true>(cur[h], meta, cv[2], cv[3], 4 * h, lane));
          mm(4 * h + 2, unpack_frag<2, true>(cur[h], meta, cv[2], cv[3], 4 * h, lane));
          mm(4 * h + 3, unpack_frag<3, true>(cur[h], meta, cv[2], cv[3], 4 * h, lane));
        }
      }
    }
    finish(it, cok && okw);
    if (it < 6) STAMP(4 + it * 3);
  };
  for (int it = it0; it < nblk; it += 2) {
    step(it, bufA, cvA, bufB, cvB);
    if (it + 1 < nblk) step(it + 1, bufB, cvB, bufA, cvA);
  }
  STAMP(22);
  STAMP_FLUSH();
}

}

extern "C" void kernel_launch(int rank, int nranks, ncclComm_t comm, hipStream_t stream, void* const* d_in, const int* in_sizes,
                              int n_in, void* d_out, int out_size, const int* in_dtypes, int out_dtype, void* const* d_out_all,
                              void* const* d_ws_all, size_t ws_bytes) {
  using namespace agk;
  (void)comm; (void)in_dtypes; (void)out_dtype;
  if (nranks != NRANK || n_in < 2 || rank < 0 || rank >= NRANK || ws_bytes < WS_NEED || in_sizes[0] != MLOC * KDIM ||
      in_sizes[1] != KDIM * NLOC || out_size != MDIM * NLOC) {
    fprintf(stderr, "kernel_launch: unexpected problem shape\n");
    return;
  }
  const int type = rank & 1, idx = rank >> 1;
  Args a;
  memset(&a, 0, sizeof a);
  for (int link = 0; link < 7; ++link) {
    const bool same = link < 3;
    const int didx = same ? link + (link >= idx ? 1 : 0) : link - 3;
    a.link_ws[link] = (char*)d_ws_all[2 * didx + (same ? type : 1 - type)];
  }
  a.link_ws[7] = (char*)d_ws_all[rank];
  for (int cg = 0; cg < 16; ++cg) {
    const int dest = cg < 8 ? rank : 2 * ((cg - 8) >> 1) + (1 - type);
    a.obase[cg] = (char*)d_out_all[dest] + (cg < 8 ? cg * 32 : idx * 64 + (cg & 1) * 32) * 2;
    a.oflag[cg] = (char*)d_ws_all[dest] + OFF_OFLAG + (size_t)(idx * 128 + (cg & 1)) * FS * 4;
    a.otile[cg] = (char*)d_ws_all[dest] + OFF_OSTAGE + (size_t)(idx * 128 + (cg & 1)) * 1024;
  }
  for (int o = 0; o < 2; ++o) {
    const int nset = o == 0 ? NSET_O : NSET_R;
    for (int set = 0; set < nset; ++set)
      for (int i = 0; i < MAXB; ++i) {
        const int q = set + nset * i;
        const int slot = q < NSLOT ? seq_to_slot(q, idx) : (idx ? 0 : P);
        const int code = slot < idx * P ? slot : slot - P;
        int src, blk;
        if (slot < 4 * P) { const int ia = slot / P; src = 2 * ia + type; blk = slot - ia * P; }
        else { const int s2 = slot - 4 * P, ia = s2 / PO; src = 2 * ia + (1 - type); blk = P + s2 - ia * PO; }
        a.codes[o][set] |= (u64)(code < 0 ? 0 : code) << (8 * i);
        a.rows[o][set] |= (u64)(src * 16 + blk) << (8 * i);
      }
  }
  for (int src = 0; src < 4; ++src)
    for (int q = 0; q < NSLOT; ++q) {
      const int slot = seq_to_slot(q, src), ty = 1 - type;
      int r, blk;
      if (slot < 4 * P) { const int ia = slot / P; r = 2 * ia + ty; blk = slot - ia * P; }
      else { const int s2 = slot - 4 * P, ia = s2 / PO; r = 2 * ia + (1 - ty); blk = P + s2 - ia * PO; }
      a.orow[src][q] = (unsigned char)(r * 16 + blk);
    }
  for (int p = 0; p < NRANK; ++p) a.all_ws[p] = (char*)d_ws_all[p];
  a.myws = (char*)d_ws_all[rank];
  a.myout = (char*)d_out;
  a.x = (const char*)d_in[0];
  a.w = (const char*)d_in[1];
  a.rank = rank;
  hipLaunchKernelGGL(ag_gemm_kernel, dim3(NWG), dim3(NTHR), 0, stream, a);
}
```

```cpp
#include <hip/hip_runtime.h>
#include <rccl/rccl.h>
#include <cstdio>
#include <cstdint>
#include <cstring>

namespace agk {

typedef unsigned u32x4 __attribute__((ext_vector_type(4)));
typedef unsigned u32x2 __attribute__((ext_vector_type(2)));
typedef unsigned u32x16 __attribute__((ext_vector_type(16)));
typedef float f32x4 __attribute__((ext_vector_type(4)));
typedef __bf16 bf16x8 __attribute__((ext_vector_type(8)));
typedef unsigned long long u64;
#define GAS __attribute__((address_space(1)))
#define LAS __attribute__((address_space(3)))

constexpr int NRANK = 8;
constexpr int MDIM = 2048, KDIM = 2048, NLOC = 256, MLOC = 256;
#define PBLK 10
#define NBC 1
constexpr int P = PBLK, NB = NBC, PO = 16 - P - NB;
constexpr int SB = P + NB, OB = PO + NB;
constexpr int NL_REM = 4 * P + 4 * PO;
constexpr int NL_OWN = NL_REM + 8 * NB;
#define POLL_SLEEP 1
#define SWV 2
#define LINKMAP 1
#define DATA_AUX 17
#define ZERO_DELAY 100
#define COL_T0 500
#define COL_DT 80
#define COL_SLEEP 1
#define HELP_SLEEP 1
#define CLK_MASK 0u
#define COL_MASK 0u
#define ZMODE 1
#define COL_GAP 6
#define Z_T0 100
#define Z_DT 10
#define OUT_WT
#define WSTEPS 4
constexpr int FS = 32;

#define WGPL 8
constexpr int NSEND_WG = 7 * WGPL;
constexpr int SW = SWV;
constexpr int WPL = WGPL * SW;
#define NSETO 12
#define NSETR 12
constexpr int NSET_O = NSETO, NSET_R = NSETR;
static_assert(NSET_O <= 16 && NSET_R <= 16, "");
static_assert(NSET_O % 4 == 0 && NSET_R % 4 == 0, "a row block's workgroups sit on one pair of XCD labels");
constexpr int NCOMP_WG = 8 * (NSET_O + NSET_R);
constexpr int NWG = NSEND_WG + NCOMP_WG;
constexpr int NWV = 12;
constexpr int NTHR = NWV * 64;
constexpr int NSET_MIN = NSET_O < NSET_R ? NSET_O : NSET_R;
constexpr int MAXB_O = (NL_OWN + NSET_O - 1) / NSET_O, MAXB_R = (NL_REM + NSET_R - 1) / NSET_R;
constexpr int MAXB = MAXB_O > MAXB_R ? MAXB_O : MAXB_R;
static_assert(NWG <= 256 && MAXB <= 6 && NL_REM <= 64 && 3 * SB + 4 * OB <= 64, "");
static_assert(SW < NWV, "spare sender waves collect the output tiles");
static_assert(SB <= NSET_O && P <= NSET_R, "only a workgroup's first block can be one of the rank's own");
constexpr int LDS_RDY = 112 << 10;
constexpr int LDS_WSCR = LDS_RDY - 8 * 12288;
static_assert(MAXB * 16384 <= LDS_RDY, "");
constexpr int STAMP_LDS = 0;
constexpr int LDS_BYTES = LDS_RDY + 64 + STAMP_LDS;
static_assert(SW * 12288 <= LDS_RDY, "");

constexpr size_t OFF_CNT = 0;
constexpr size_t CNT_BYTES = (size_t)(NWG + 1) * NWV * 32;
constexpr size_t OFF_TMO = 128 << 10;
constexpr size_t OFF_TICKET = OFF_TMO + 1024;
constexpr int NRX = (3 * SB + 4 * OB) * 16, NRU = NRX + 256;
constexpr size_t OFF_RLINE = 132 << 10, RLINE_BYTES = (size_t)NRU * 128;
constexpr size_t OFF_OFLAG = OFF_RLINE + 3 * RLINE_BYTES;
constexpr size_t OFF_FLAG_END = OFF_OFLAG + (size_t)512 * FS * 4;
constexpr size_t OFF_RSTAGE = 1 << 20, RSTAGE_BYTES = (size_t)NRU * 4096;
constexpr size_t OFF_OSTAGE = OFF_RSTAGE + 3 * RSTAGE_BYTES;
constexpr size_t WS_NEED = OFF_OSTAGE + (size_t)512 * 1024;
constexpr size_t OFF_DBG = 16 << 20;
static_assert(CNT_BYTES <= OFF_TMO && OFF_FLAG_END <= OFF_RSTAGE && WS_NEED <= OFF_DBG, "");
constexpr u64 DEADLINE = 50000000ull;

constexpr unsigned META_RAW = 0x80000000u;
constexpr unsigned META_SV = 0x40000000u;

struct Args {
  char* link_ws[8];
  char* obase[16];
  char* oflag[16];
  char* otile[16];
  unsigned char orow[4][64];
  char* all_ws[8];
  u64 codes[2][16];
  u64 rows[2][16];
  char* myws;
  char* myout;
  const char* x;
  const char* w;
  int rank;
  int pad;
};

template <int OFF>
__device__ __forceinline__ void st16_wt(char* p, u32x4 v) {
  asm volatile("global_store_dwordx4 %0, %1, off offset:%c2 sc0 sc1\n\ts_nop 1" ::"v"(p), "v"(v), "i"(OFF) : "memory");
}
template <int OFF>
__device__ __forceinline__ void st16_wt_s(const char* base, unsigned voff, u32x4 v) {
  asm volatile("global_store_dwordx4 %0, %1, %2 offset:%c3 sc0 sc1\n\ts_nop 1" ::"v"(voff), "v"(v), "s"(base), "i"(OFF) : "memory");
}
__device__ __forceinline__ void st4_wt(char* p, unsigned v) {
  asm volatile("global_store_dword %0, %1, off sc0 sc1\n\ts_nop 1" ::"v"(p), "v"(v) : "memory");
}
__device__ __forceinline__ void st8_wt(char* p, u32x2 v) {
  asm volatile("global_store_dwordx2 %0, %1, off sc0 sc1\n\ts_nop 1" ::"v"(p), "v"(v) : "memory");
}
__device__ __forceinline__ unsigned ticket_issue(char* p) {
  unsigned r = 0, one = 1;
  u64 save;
  asm volatile("s_mov_b64 %1, exec\n\ts_mov_b64 exec, 1\n\tglobal_atomic_add %0, %2, %3, off sc0\n\ts_mov_b64 exec, %1"
               : "+v"(r), "=&s"(save) : "v"(p), "v"(one) : "memory");
  return r;
}
#define WAIT_VM0() asm volatile("s_waitcnt vmcnt(0)" ::: "memory")
#define PIN(x) asm volatile("" : "+v"(x))
#define LIKELY(x) __builtin_expect(!!(x), 1)
#define UNLIKELY(x) __builtin_expect(!!(x), 0)

__device__ __forceinline__ u64 rt() { return __builtin_amdgcn_s_memrealtime(); }

constexpr unsigned OOB = 0x7ffffff0u;
__device__ __forceinline__ u32x4 poll_load(__amdgpu_buffer_rsrc_t rc, unsigned voff) {
  return __builtin_bit_cast(u32x4, __builtin_amdgcn_raw_buffer_load_b128(rc, voff, 0, 17));
}
__device__ __forceinline__ bool lines_ready(const u32x4& v, bool line_lane, bool first, unsigned call) {
  const unsigned late = (unsigned)__builtin_amdgcn_update_dpp(0, (int)v[0], 0x101, 0xf, 0xf, false);
  return __all(!line_lane || (first ? v[0] >= call && ((v[1] & META_SV) || late >= call) : v[1] >= call));
}
__device__ __forceinline__ bool wait_lines(__amdgpu_buffer_rsrc_t rc, unsigned voff, bool line_lane, bool first, int tmo_lane, unsigned call, unsigned* tmo, u32x4& fl) {
  const u64 t0 = rt();
  for (unsigned n = 1;; ++n) {
    fl = poll_load(rc, voff);
    const unsigned t = (unsigned)__builtin_amdgcn_readlane((int)fl[0], tmo_lane);
    if (lines_ready(fl, line_lane, first, call)) { __builtin_amdgcn_fence(__ATOMIC_ACQUIRE, "wavefront"); return t != call; }
    if (t == call || ((n & CLK_MASK) == 0u && rt() - t0 > DEADLINE)) {
      __hip_atomic_store((GAS unsigned*)tmo, call, __ATOMIC_RELAXED, __HIP_MEMORY_SCOPE_AGENT);
      fl = u32x4{META_RAW, META_RAW, META_RAW, META_RAW};
      return false;
    }
    __builtin_amdgcn_s_sleep(POLL_SLEEP);
  }
}
__device__ __forceinline__ unsigned umin_(unsigned a, unsigned b) { return a < b ? a : b; }
__device__ __forceinline__ unsigned min12(const u32x4 (&v)[3]) {
  unsigned m = umin_(umin_(v[0][0], v[0][1]), v[0][2]);
  m = umin_(umin_(m, v[0][3]), v[1][0]);
  m = umin_(umin_(m, v[1][1]), v[1][2]);
  m = umin_(umin_(m, v[1][3]), v[2][0]);
  m = umin_(umin_(m, v[2][1]), v[2][2]);
  return umin_(m, v[2][3]);
}

__device__ __forceinline__ unsigned x_lane_off(int lane, int i) {
  const int r = 4 * i + (lane >> 4);
  return (unsigned)(r * (KDIM * 2) + (((lane & 15) ^ r) * 16));
}
__device__ __forceinline__ void x_fetch(const char* x, int blk, int u, const unsigned (&off)[4], char* scratch) {
  const GAS char* base = (const GAS char*)x + (size_t)(blk * 16 * (KDIM * 2) + u * 256);
#pragma unroll
  for (int i = 0; i < 4; ++i) __builtin_amdgcn_global_load_lds((const GAS void*)(base + off[i]), (LAS void*)(scratch + i * 1024), 16, 0, 0);
}
__device__ __forceinline__ void x_read(const char* scratch, int lane, u32x4 (&v)[4]) {
  const int m = lane & 15, q = lane >> 4;
  const char* p = scratch + m * 256;
#pragma unroll
  for (int s = 0; s < 4; ++s) v[s] = *(const LAS u32x4*)(p + (((4 * s + q) ^ m) * 16));
}
__device__ __forceinline__ unsigned w_lane_off(int lane) { return (unsigned)((lane >> 2) * (NLOC * 2) + (lane & 3) * 16); }
__device__ __forceinline__ void w_fetch(const char* w, int cg32, int ks2, unsigned off, char* scratch) {
  const GAS char* base = (const GAS char*)w + (size_t)(ks2 * 64 * (NLOC * 2) + cg32 * 64);
#pragma unroll
  for (int i = 0; i < 4; ++i)
    __builtin_amdgcn_global_load_lds((const GAS void*)(base + i * 16 * (NLOC * 2) + off), (LAS void*)(scratch + i * 1024), 16, 0, 0);
}
__device__ __forceinline__ void w_read(const char* scratch, int lane, u32x4 (&v)[4]) {
  typedef short s16x4 __attribute__((ext_vector_type(4)));
  const int g = lane >> 4, i = lane & 15;
  const char* p = scratch + (8 * g + (i >> 2)) * 64 + (i & 3) * 8;
#pragma unroll
  for (int f = 0; f < 4; ++f)
#pragma unroll
    for (int h = 0; h < 2; ++h) {
      const u32x2 t = __builtin_bit_cast(u32x2, __builtin_amdgcn_ds_read_tr16_b64_v4i16((LAS s16x4*)(p + ((f >> 1) * 32 + 4 * h) * 64 + (f & 1) * 32)));
      v[f][2 * h] = t[0];
      v[f][2 * h + 1] = t[1];
    }
}

__device__ __forceinline__ unsigned lds_off(const char* p) { return (unsigned)(size_t)(const LAS char*)p; }
__device__ __forceinline__ void x_read_asm(const char* scratch, int lane, u32x4 (&v)[4]) {
  const int m = lane & 15, q = lane >> 4;
  const unsigned p = lds_off(scratch) + m * 256;
  asm volatile("ds_read_b128 %0, %4\n\tds_read_b128 %1, %5\n\tds_read_b128 %2, %6\n\tds_read_b128 %3, %7\n\ts_waitcnt lgkmcnt(0)"
               : "=&v"(v[0]), "=&v"(v[1]), "=&v"(v[2]), "=&v"(v[3])
               : "v"(p + ((q ^ m) * 16)), "v"(p + (((4 + q) ^ m) * 16)), "v"(p + (((8 + q) ^ m) * 16)), "v"(p + (((12 + q) ^ m) * 16))
               : "memory");
}
__device__ __forceinline__ void w_read_asm(const char* scratch, int lane, u32x4 (&v)[4]) {
  const int g = lane >> 4, i = lane & 15;
  const unsigned p = lds_off(scratch) + (8 * g + (i >> 2)) * 64 + (i & 3) * 8;
  u32x2 t0, t1, t2, t3, t4, t5, t6, t7;
  asm volatile("ds_read_b64_tr_b16 %0, %8\n\tds_read_b64_tr_b16 %1, %8 offset:256\n\tds_read_b64_tr_b16 %2, %8 offset:32\n\t"
               "ds_read_b64_tr_b16 %3, %8 offset:288\n\tds_read_b64_tr_b16 %4, %8 offset:2048\n\tds_read_b64_tr_b16 %5, %8 offset:2304\n\t"
               "ds_read_b64_tr_b16 %6, %8 offset:2080\n\tds_read_b64_tr_b16 %7, %8 offset:2336\n\ts_waitcnt lgkmcnt(0)"
               : "=&v"(t0), "=&v"(t1), "=&v"(t2), "=&v"(t3), "=&v"(t4), "=&v"(t5), "=&v"(t6), "=&v"(t7) : "v"(p) : "memory");
  v[0] = u32x4{t0[0], t0[1], t1[0], t1[1]};
  v[1] = u32x4{t2[0], t2[1], t3[0], t3[1]};
  v[2] = u32x4{t4[0], t4[1], t5[0], t5[1]};
  v[3] = u32x4{t6[0], t6[1], t7[0], t7[1]};
}

__device__ __forceinline__ void x_read_issue(unsigned sc, const unsigned (&ro)[4], u32x4 (&v)[4]) {
  asm volatile("ds_read_b128 %0, %4\n\tds_read_b128 %1, %5\n\tds_read_b128 %2, %6\n\tds_read_b128 %3, %7"
               : "=&v"(v[0]), "=&v"(v[1]), "=&v"(v[2]), "=&v"(v[3]) : "v"(sc + ro[0]), "v"(sc + ro[1]), "v"(sc + ro[2]), "v"(sc + ro[3]) : "memory");
}
__device__ __forceinline__ void lds_wait4(u32x4 (&v)[4]) {
  asm volatile("s_waitcnt lgkmcnt(0)" : "+v"(v[0]), "+v"(v[1]), "+v"(v[2]), "+v"(v[3]) :: "memory");
}

__device__ __forceinline__ unsigned umax_(unsigned a, unsigned b) { return a > b ? a : b; }
__device__ __forceinline__ unsigned wave_max(unsigned v) {
  v = umax_(v, (unsigned)__builtin_amdgcn_update_dpp(0, (int)v, 0x111, 0xf, 0xf, false));
  v = umax_(v, (unsigned)__builtin_amdgcn_update_dpp(0, (int)v, 0x112, 0xf, 0xf, false));
  v = umax_(v, (unsigned)__builtin_amdgcn_update_dpp(0, (int)v, 0x114, 0xf, 0xf, false));
  v = umax_(v, (unsigned)__builtin_amdgcn_update_dpp(0, (int)v, 0x118, 0xf, 0xf, false));
  v = umax_(v, (unsigned)__builtin_amdgcn_update_dpp(0, (int)v, 0x142, 0xa, 0xf, false));
  v = umax_(v, (unsigned)__builtin_amdgcn_update_dpp(0, (int)v, 0x143, 0xc, 0xf, false));
  return (unsigned)__builtin_amdgcn_readlane((int)v, 63);
}
typedef unsigned short u16x2 __attribute__((ext_vector_type(2)));
__device__ __forceinline__ unsigned pk_max_u16(unsigned a, unsigned b) {
  return __builtin_bit_cast(unsigned, __builtin_elementwise_max(__builtin_bit_cast(u16x2, a), __builtin_bit_cast(u16x2, b)));
}
struct Packed { u32x4 s[4]; u32x2 ent; unsigned meta; };
__device__ __forceinline__ unsigned bfi(unsigned mask, unsigned a, unsigned b) {
  unsigned r;
  asm("v_bfi_b32 %0, %1, %2, %3" : "=v"(r) : "s"(mask), "v"(a), "v"(b));
  return r;
}
__device__ __forceinline__ void pack_unit(const u32x4 (&f)[4], int lane, Packed& o) {
  u32x4 L0, L1, C;
  unsigned hq[8], uq[8];
#pragma unroll
  for (int i = 0; i < 4; ++i) {
    hq[2 * i] = __builtin_amdgcn_perm(f[i][1], f[i][0], 0x07050301u);
    hq[2 * i + 1] = __builtin_amdgcn_perm(f[i][3], f[i][2], 0x07050301u);
    uq[2 * i] = hq[2 * i] | 0x80808080u;
    uq[2 * i + 1] = hq[2 * i + 1] | 0x80808080u;
  }
  const unsigned mo = pk_max_u16(pk_max_u16(pk_max_u16(uq[0], uq[1]), pk_max_u16(uq[2], uq[3])), pk_max_u16(pk_max_u16(uq[4], uq[5]), pk_max_u16(uq[6], uq[7])));
  const unsigned me = pk_max_u16(pk_max_u16(pk_max_u16(uq[0] << 8, uq[1] << 8), pk_max_u16(uq[2] << 8, uq[3] << 8)),
                                 pk_max_u16(pk_max_u16(uq[4] << 8, uq[5] << 8), pk_max_u16(uq[6] << 8, uq[7] << 8)));
  const unsigned m2 = pk_max_u16(mo, me);
  const unsigned mx = (wave_max(umax_(m2 & 0xffffu, m2 >> 16)) >> 8) & 0x7fu;
  const unsigned base = (mx > 7u ? mx : 7u) - 7u;
  const unsigned base4 = base * 0x01010101u;
  unsigned E[4];
#pragma unroll
  for (int i = 0; i < 4; ++i) {
    const unsigned l0 = __builtin_amdgcn_perm(f[i][1], f[i][0], 0x06040200u), l1 = __builtin_amdgcn_perm(f[i][3], f[i][2], 0x06040200u);
    const unsigned t0 = uq[2 * i] - base4, t1 = uq[2 * i + 1] - base4;
    E[i] = t0 & t1;
    C[i] = bfi(0x0f0f0f0fu, bfi(0x07070707u, t0, hq[2 * i] >> 4), bfi(0x70707070u, t1 << 4, hq[2 * i + 1]));
    if (i < 2) { L0[2 * i] = l0; L0[2 * i + 1] = l1; } else { L1[2 * i - 4] = l0; L1[2 * i - 3] = l1; }
  }
  unsigned fmask = 0;
  bool raw = false;
  u32x2 ent = {0u, 0u};
  if (UNLIKELY(__any((E[0] & E[1] & E[2] & E[3] & 0x80808080u) != 0x80808080u))) {
#pragma unroll
    for (int i = 0; i < 4; ++i) {
      u64 bm = __ballot((E[i] & 0x80808080u) != 0x80808080u);
      unsigned cnt = 0, e0 = 0, e1 = 0;
      while (bm && cnt <= 2u) {
        const int l = __builtin_ctzll(bm);
        bm &= bm - 1;
#pragma unroll
        for (int q = 0; q < 2; ++q) {
          const unsigned h = (unsigned)__builtin_amdgcn_readlane((int)hq[2 * i + q], l);
          unsigned m2 = ~((h | 0x80808080u) - base4) & 0x80808080u;
          while (m2) {
            const unsigned b = (unsigned)__builtin_ctz(m2) >> 3;
            m2 &= m2 - 1;
            const unsigned e = 0x80000000u | ((unsigned)l << 16) | ((4u * q + b) << 8) | ((h >> (8u * b)) & 0xffu);
            e1 = cnt == 1u ? e : e1;
            e0 = cnt == 0u ? e : e0;
            ++cnt;
          }
        }
      }
      raw = raw || cnt > 2u;
      fmask |= cnt ? 1u << i : 0u;
      ent[0] = lane == i ? e0 : ent[0];
      ent[1] = lane == i ? e1 : ent[1];
    }
  }
  o.ent = ent;
  if (UNLIKELY(raw)) {
    o.s[0] = f[0]; o.s[1] = f[1]; o.s[2] = f[2]; o.s[3] = f[3];
    o.meta = META_RAW;
  } else {
    o.s[0] = L0; o.s[1] = L1; o.s[2] = C; o.s[3] = C;
    const u32x4 t[3] = {L0, L1, C};
    o.meta = (fmask << 8) | base | (__all(min12(t) != 0u) ? META_SV : 0u);
  }
}
__device__ __forceinline__ void store_pieces(char* dst, const Packed& p) {
  st16_wt<0>(dst, p.s[0]);
  st16_wt<1024>(dst, p.s[1]);
  st16_wt<2048>(dst, p.s[2]);
  if (UNLIKELY(p.meta & META_RAW)) st16_wt<3072>(dst, p.s[3]);
}
__device__ __forceinline__ void store_line(char* line, const Packed& p, unsigned meta, unsigned call, unsigned late, int lane) {
  const u32x4 v = {lane == 0 ? call : lane == 1 ? late : 0u, lane == 0 ? meta : call, p.ent[0], p.ent[1]};
  if (lane < 4) st16_wt<0>(line + lane * 16, v);
}
__device__ __forceinline__ void patch_frag(u32x4& o, unsigned e, int lane) {
  const unsigned sh = 8u + 16u * ((e >> 8) & 1u), d = (e >> 9) & 3u;
  const unsigned vm = (unsigned)lane == ((e >> 16) & 63u) ? 0xffu << sh : 0u, put = (e & 0xffu) << sh;
#pragma unroll
  for (int t = 0; t < 4; ++t) {
    const unsigned m = d == (unsigned)t ? vm : 0u;
    o[t] = (o[t] & ~m) | (put & m);
  }
}
template <int I, bool EXC>
__device__ __forceinline__ u32x4 unpack_frag(const u32x4 (&v)[3], unsigned meta, unsigned e0, unsigned e1, int fb, int lane) {
  const unsigned base4 = (meta & 0x7fu) * 0x01010101u;
  const unsigned l0 = v[I >> 1][2 * (I & 1)], l1 = v[I >> 1][2 * (I & 1) + 1];
  const unsigned c = v[2][I];
  const unsigned h0 = ((c & 0x08080808u) << 4) | ((c & 0x07070707u) + base4);
  const unsigned h1 = (c & 0x80808080u) | (((c >> 4) & 0x07070707u) + base4);
  u32x4 o;
  o[0] = __builtin_amdgcn_perm(h0, l0, 0x05010400u);
  o[1] = __builtin_amdgcn_perm(h0, l0, 0x07030602u);
  o[2] = __builtin_amdgcn_perm(h1, l1, 0x05010400u);
  o[3] = __builtin_amdgcn_perm(h1, l1, 0x07030602u);
  if (EXC && (meta & (0x100u << I))) {
    patch_frag(o, (unsigned)__builtin_amdgcn_readlane((int)e0, fb + I), lane);
    const unsigned e = (unsigned)__builtin_amdgcn_readlane((int)e1, fb + I);
    if (e) patch_frag(o, e, lane);
  }
  return o;
}

__device__ __forceinline__ float gelu_f(float y) {
  const float c = 0.7978845608028654f;
  const float k2 = -2.0f * 1.4426950408889634f * c;
  const float t = y * (1.0f + 0.044715f * y * y);
  const float e = __builtin_amdgcn_exp2f(k2 * t);
  return y * __builtin_amdgcn_rcpf(1.0f + e);
}
__device__ __forceinline__ unsigned pack_bf16(float lo, float hi) {
  typedef __bf16 bf2 __attribute__((ext_vector_type(2)));
  typedef float f2 __attribute__((ext_vector_type(2)));
  f2 f = {lo, hi};
  return __builtin_bit_cast(unsigned, __builtin_convertvector(f, bf2));
}

#define STAMP(idx) do {} while (0)
#define STAMP_FLUSH() do {} while (0)
#define STAMP2(idx) do {} while (0)

__global__ void __launch_bounds__(NTHR, 3) ag_gemm_kernel(Args a) {
  __shared__ __attribute__((aligned(16))) char lds[LDS_BYTES];
  const int tid = threadIdx.x, lane = tid & 63;
  const int wv = __builtin_amdgcn_readfirstlane(tid >> 6);
  const int bid = blockIdx.x;
  const int rank = a.rank, type = rank & 1, idx = rank >> 1;
  char* const myws = a.myws;
  unsigned* const tmo = (unsigned*)(myws + OFF_TMO);
  STAMP(0);
  auto get_call = [&]() -> unsigned {
    asm volatile("" ::: "memory");
    unsigned ticket = 0;
    if (lane == 0)
      ticket = __hip_atomic_fetch_add((GAS unsigned*)(myws + OFF_CNT + (size_t)(bid * NWV + wv) * 32), 1u, __ATOMIC_RELAXED, __HIP_MEMORY_SCOPE_AGENT);
    return (unsigned)__builtin_amdgcn_readfirstlane(ticket) + 1u;
  };

  auto zero_units = [&](int first, int step, unsigned call) {
    char* const zb = myws + OFF_RSTAGE + (size_t)((call + 2u) % 3u) * RSTAGE_BYTES + lane * 16;
    const u32x4 z = {0u, 0u, 0u, 0u};
    for (int zu = first; zu < NRU; zu += step) {
      st16_wt<0>(zb + (size_t)zu * 4096, z);
      st16_wt<1024>(zb + (size_t)zu * 4096, z);
      st16_wt<2048>(zb + (size_t)zu * 4096, z);
    }
  };
  if (bid < NSEND_WG) {
    const int link = bid % 7, lq = bid / 7;
    if (wv >= SW) {
      const u64 tstart = rt();
      const unsigned call = get_call();
      for (int o = bid * (NWV - SW) + wv - SW; o < NL_REM * 8; o += NSEND_WG * (NWV - SW)) {
        const int q = o >> 3, src = (o >> 1) & 3, half = o & 1;
        const int t = src * 128 + q * 2 + half;
        const __amdgpu_buffer_rsrc_t ro = __builtin_amdgcn_make_buffer_rsrc((void*)myws, 0, (int)WS_NEED, 0x00020000);
        while (rt() - tstart < (u64)(COL_T0 + q * COL_DT / 8)) __builtin_amdgcn_s_sleep(8);
        const u64 t0 = rt();
        u32x4 v;
        bool good = true;
        auto ldt = [&]() -> u32x4 { return __builtin_bit_cast(u32x4, __builtin_amdgcn_raw_buffer_load_b128(ro, lane * 16, (int)(OFF_OSTAGE + (size_t)t * 1024), 17)); };
        auto full = [&](const u32x4& x) -> bool { return __all(umin_(umin_(x[0], x[1]), umin_(x[2], x[3])) != 0u); };
        for (unsigned n = 1;; ++n) {
          v = ldt();
          const unsigned tm = __hip_atomic_load((GAS unsigned*)tmo, __ATOMIC_RELAXED, __HIP_MEMORY_SCOPE_AGENT);
          if (full(v)) { good = tm != call; break; }
          if ((n & COL_MASK) == 0u) {
            const unsigned f = __hip_atomic_load((const GAS unsigned*)(myws + OFF_OFLAG + (size_t)t * FS * 4), __ATOMIC_RELAXED, __HIP_MEMORY_SCOPE_SYSTEM);
            if (f >= call) {
              __builtin_amdgcn_fence(__ATOMIC_ACQUIRE, "wavefront");
              v = ldt();
              good = tm != call;
              break;
            }
            if (tm == call || rt() - t0 > DEADLINE) {
              __hip_atomic_store((GAS unsigned*)tmo, call, __ATOMIC_RELAXED, __HIP_MEMORY_SCOPE_AGENT);
              good = false;
              break;
            }
          }
          __builtin_amdgcn_s_sleep(COL_SLEEP);
        }
        if (UNLIKELY(!good)) v = u32x4{0x7fc07fc0u, 0x7fc07fc0u, 0x7fc07fc0u, 0x7fc07fc0u};
        st16_wt<0>(a.myout + (size_t)(a.orow[src][q] * 16 + (lane >> 2)) * (NLOC * 2) + (src * 64 + half * 32) * 2 + (lane & 3) * 16, v);
        PIN(v);
        st16_wt<0>(myws + OFF_OSTAGE + (size_t)t * 1024 + lane * 16, u32x4{0u, 0u, 0u, 0u});
      }
      STAMP(23);
      STAMP_FLUSH();
      return;
    }
    const int j = lq * SW + wv;
    const bool same = link < 3;
    const int didx = same ? link + (link >= idx ? 1 : 0) : link - 3;
    char* const dws = a.link_ws[link];
    u32x4 v[4];
    const int b0 = same ? 0 : P, nb = same ? SB : OB;
    const int rx0 = (same ? (idx < didx ? idx : idx - 1) * SB : 3 * SB + idx * OB) * 16, rw0 = NRX + idx * 64;
    char* const scr = lds + wv * 4096;
    static_assert(WPL == 16, "");
    const int nws = same ? 0 : 4, nst = nws + nb;
    const unsigned sc0 = lds_off(lds) + (unsigned)wv * 12288u;
    const unsigned xo[4] = {x_lane_off(lane, 0), x_lane_off(lane, 1), x_lane_off(lane, 2), x_lane_off(lane, 3)};
    const unsigned wo = w_lane_off(lane);
    const char* const xsrc = a.x + (size_t)b0 * 65536 + j * 256;
    const char* const wsrc = a.w + (size_t)(j >> 1) * 64 * (NLOC * 2) + (didx * 2 + (j & 1)) * 64;
    auto fetch_x = [&](int k, unsigned sc) {
      const int kb = same && k >= P ? k + PO : k;
      const GAS char* b = (const GAS char*)xsrc + (size_t)kb * 65536;
#pragma unroll
      for (int i = 0; i < 4; ++i) __builtin_amdgcn_global_load_lds((const GAS void*)(b + xo[i]), (LAS void*)(size_t)(sc + i * 1024), 16, 0, 0);
    };
    auto fetch_w = [&](int t, unsigned sc) {
      const GAS char* b = (const GAS char*)wsrc + (size_t)t * 8 * 64 * (NLOC * 2);
#pragma unroll
      for (int i = 0; i < 4; ++i) __builtin_amdgcn_global_load_lds((const GAS void*)(b + i * 16 * (NLOC * 2) + wo), (LAS void*)(size_t)(sc + i * 1024), 16, 0, 0);
    };
    if (same) { fetch_x(0, sc0); fetch_x(1, sc0 + 4096); } else { fetch_w(0, sc0); fetch_w(1, sc0 + 4096); }
    const unsigned call = get_call();
    const unsigned buf = call % 3u;
    const char* const stage = dws + OFF_RSTAGE + (size_t)buf * RSTAGE_BYTES;
    char* const lines = dws + OFF_RLINE + (size_t)buf * RLINE_BYTES;
    const unsigned l16 = (unsigned)lane * 16u;
    unsigned ro[4];
    {
      const int m = lane & 15, q = lane >> 4;
#pragma unroll
      for (int i = 0; i < 4; ++i) ro[i] = (unsigned)(m * 256 + (((4 * i + q) ^ m) * 16));
    }
    WAIT_VM0();
    STAMP(1);
    Packed pc;
    auto send = [&](int ui) {
      const char* d = stage + (size_t)ui * 4096;
      st16_wt_s<0>(d, l16, pc.s[0]);
      st16_wt_s<1024>(d, l16, pc.s[1]);
      st16_wt_s<2048>(d, l16, pc.s[2]);
      if (UNLIKELY(pc.meta & META_RAW)) st16_wt_s<3072>(d, l16, pc.s[3]);
      char* const line = lines + (size_t)ui * 128;
      store_line(line, pc, pc.meta, call, 0u, lane);
      if (UNLIKELY(!(pc.meta & META_SV))) {
        WAIT_VM0();
        if (lane == 0) st4_wt(line + 16, call);
      }
      asm volatile("s_waitcnt vmcnt(8)" ::: "memory");
    };
    unsigned sa = sc0, sb = sc0 + 4096, sc = sc0 + 8192;
    if (!same) {
#pragma nounroll
      for (int t = 0; t < 4; ++t) {
        w_read_asm(lds + (sa - lds_off(lds)), lane, v);
        if (t < 2) fetch_w(t + 2, sc); else fetch_x(t - 2, sc);
        pack_unit(v, lane, pc);
        send(rw0 + (j & 1) * 32 + (j >> 1) + 8 * t);
        const unsigned r = sa; sa = sb; sb = sc; sc = r;
      }
    }
#pragma nounroll
    for (int k = 0; k < nb; ++k) {
      x_read_issue(sa, ro, v);
      if (k + 2 < nb) fetch_x(k + 2, sc);
      lds_wait4(v);
      pack_unit(v, lane, pc);
      send(rx0 + k * 16 + j);
      const unsigned r = sa; sa = sb; sb = sc; sc = r;
    }
    (void)nst;
    STAMP(2);
    STAMP_FLUSH();
    return;
  }

  const int cb = bid - NSEND_WG;
  const int xl = cb & 7, loc = cb >> 3;
  const bool own = loc < NSET_O;
  const int l2 = own ? loc : loc - NSET_O;
  const int nset = own ? NSET_O : NSET_R;
  const int set = (xl >> 1) + 4 * (l2 >> 2);
  const int cg = (own ? 0 : 8) + (xl & 1) * 4 + (l2 & 3);
  unsigned* const ready = (unsigned*)(lds + LDS_RDY);
  if (tid < 16) ready[tid] = 0;
  asm volatile("s_waitcnt lgkmcnt(0)" ::: "memory");
  __builtin_amdgcn_s_barrier();
  if (wv >= 8) {
    const int h = wv - 8;
    const unsigned call = get_call();
    STAMP(1);
    {
      { const u64 tz = rt(); const u64 dz = Z_T0 + (u64)((h * NCOMP_WG + cb) % 96) * Z_DT; while (rt() - tz < dz) __builtin_amdgcn_s_sleep(8); }
      zero_units(h * NCOMP_WG + cb, 4 * NCOMP_WG, call);
    }
    char* const obase = a.obase[cg];
    char* const oflag = a.oflag[cg];
    char* const otile = a.otile[cg];
    const u64 rows = a.rows[own ? 0 : 1][set];
    const int nblk = (int)(a.codes[own ? 0 : 1][set] >> 56);
    for (int it = h; it < nblk; it += 4) {
      const int gblk = (int)(rows >> (8 * it)) & 127;
      const int row = lane >> 2, m = lane & 3;
      char* const op = obase + (size_t)(gblk * 16 + row) * (NLOC * 2) + m * 16;
      while (__hip_atomic_load(&ready[it], __ATOMIC_RELAXED, __HIP_MEMORY_SCOPE_WORKGROUP) < 8u) __builtin_amdgcn_s_sleep(HELP_SLEEP);
      __builtin_amdgcn_fence(__ATOMIC_ACQUIRE, "workgroup");
      const char* buf = lds + it * 16384 + row * 128;
      const int sw = ((row & 1) << 2) | (row & 2) | ((row >> 2) & 1);
      f32x4 pl[8], ph[8];
#pragma unroll
      for (int p = 0; p < 8; ++p) {
        pl[p] = *(const f32x4*)(buf + p * 2048 + (((2 * m) ^ sw) * 16));
        ph[p] = *(const f32x4*)(buf + p * 2048 + (((2 * m + 1) ^ sw) * 16));
      }
      __builtin_amdgcn_sched_barrier(0);
      const f32x4 lo = ((pl[0] + pl[1]) + (pl[2] + pl[3])) + ((pl[4] + pl[5]) + (pl[6] + pl[7]));
      const f32x4 hi = ((ph[0] + ph[1]) + (ph[2] + ph[3])) + ((ph[4] + ph[5]) + (ph[6] + ph[7]));
      u32x4 o;
      o[0] = pack_bf16(gelu_f(lo[0]), gelu_f(lo[1]));
      o[1] = pack_bf16(gelu_f(lo[2]), gelu_f(lo[3]));
      o[2] = pack_bf16(gelu_f(hi[0]), gelu_f(hi[1]));
      o[3] = pack_bf16(gelu_f(hi[2]), gelu_f(hi[3]));
      if (cg < 8) {
        st16_wt<0>(op, o);
      } else {
        st16_wt<0>(otile + (size_t)((set + nset * it) * 2) * 1024 + lane * 16, o);
        if (UNLIKELY(!__all(umin_(umin_(o[0], o[1]), umin_(o[2], o[3])) != 0u))) {
          WAIT_VM0();
          if (lane == 0) st8_wt(oflag + (size_t)((set + nset * it) * 2) * FS * 4, u32x2{call, 0u});
        }
      }
      if (it < 6) STAMP(5 + it * 3);
    }
    STAMP(22);
    STAMP_FLUSH();
    return;
  }
  const u64 slotpk = a.codes[own ? 0 : 1][set];
  const bool own0 = (slotpk & 0x40u) != 0u;
  const int ownblk = (int)slotpk & 15;
  char* const wscr = lds + LDS_WSCR + wv * 12288;
  const unsigned xoffs[4] = {x_lane_off(lane, 0), x_lane_off(lane, 1), x_lane_off(lane, 2), x_lane_off(lane, 3)};
  auto own_fetch = [&](int r) {
    if (own) {
      w_fetch(a.w, cg, wv * 4 + 2 * r, w_lane_off(lane), wscr);
      w_fetch(a.w, cg, wv * 4 + 2 * r + 1, w_lane_off(lane), wscr + 4096);
    }
    if (own0) x_fetch(a.x, ownblk, wv * 2 + r, xoffs, wscr + 8192);
  };
  own_fetch(0);
  const int nblk = (int)(slotpk >> 56);
  const unsigned call = get_call();
  const __amdgpu_buffer_rsrc_t rs = __builtin_amdgcn_make_buffer_rsrc((void*)myws, 0, (int)WS_NEED, 0x00020000);
  const unsigned buf = call % 3u;
  const unsigned rdata = (unsigned)OFF_RSTAGE + buf * (unsigned)RSTAGE_BYTES, rline = (unsigned)OFF_RLINE + buf * (unsigned)RLINE_BYTES;
  auto code_of = [&](int it) -> unsigned { return (unsigned)(slotpk >> (8 * it)) & 63u; };
  auto xdata = [&](unsigned code) -> int { return (int)(rdata + code * 65536u + (unsigned)wv * 8192u); };
  const bool xline = lane < 8, first = (lane & 3) == 0;
  const unsigned xlane = (unsigned)((wv * 2 + (lane >> 2)) * 128 + (lane & 3) * 16);
  auto xoff = [&](unsigned code) -> unsigned { return lane < 8 ? rline + code * 2048u + xlane : lane == 8 ? (unsigned)OFF_TMO : OOB; };
  auto ldp = [&](int soff, int piece) -> u32x4 {
    return __builtin_bit_cast(u32x4, __builtin_amdgcn_raw_buffer_load_b128(rs, lane * 16 + piece * 1024, soff, DATA_AUX));
  };
  auto load_x = [&](u32x4 (&af)[2][3], unsigned code) {
    const int so = xdata(code);
#pragma unroll
    for (int h = 0; h < 2; ++h)
#pragma unroll
      for (int p = 0; p < 3; ++p) af[h][p] = ldp(so + h * 4096, p);
  };
  auto issue = [&](u32x4 (&af)[2][3], u32x4& cv, int it) {
    cv = poll_load(rs, xoff(code_of(it)));
    load_x(af, code_of(it));
  };
  u32x4 wf[4][4];
  u32x4 x0[2][4];
  bool okw = true;
#pragma unroll
  for (int r = 0; r < 2; ++r) {
    WAIT_VM0();
    if (own) {
      w_read(wscr, lane, wf[2 * r]);
      w_read(wscr + 4096, lane, wf[2 * r + 1]);
#pragma unroll
      for (int f = 0; f < 4; ++f) { PIN(wf[2 * r][f]); PIN(wf[2 * r + 1][f]); }
    }
    if (own0) {
      x_read(wscr + 8192, lane, x0[r]);
#pragma unroll
      for (int f = 0; f < 4; ++f) PIN(x0[r][f]);
    }
    if (r == 0) own_fetch(1);
  }
  constexpr int WSCR_IT = LDS_WSCR / 16384;
  if (lane == 0) __hip_atomic_fetch_add(&ready[8], 1u, __ATOMIC_RELEASE, __HIP_MEMORY_SCOPE_WORKGROUP);
  u32x4 wfl = {0u, 0u, 0u, 0u};
  const unsigned wunit = (unsigned)(NRX + (cg - 8) * 32 + wv * 4);
  if (!own)
    okw = wait_lines(rs, lane < 16 ? rline + wunit * 128u + (unsigned)((lane >> 2) * 128 + (lane & 3) * 16) : lane == 16 ? (unsigned)OFF_TMO : OOB,
                     lane < 16, first, 16, call, tmo, wfl);
  const int it0 = own0 ? 1 : 0;
  u32x4 bufA[2][3], bufB[2][3], cvA, cvB;
  issue(bufA, cvA, it0);
  STAMP(2);
  if (!own) {
    const int soff = (int)(rdata + wunit * 4096u);
    u32x4 wp[4][3];
    const u64 t0 = rt();
    for (;;) {
#pragma unroll
      for (int s = 0; s < 4; ++s)
#pragma unroll
        for (int p = 0; p < 3; ++p) wp[s][p] = ldp(soff + s * 4096, p);
#pragma unroll
      for (int s = 0; s < 4; ++s)
#pragma unroll
        for (int p = 0; p < 3; ++p) PIN(wp[s][p]);
      bool bad = false;
#pragma unroll
      for (int s = 0; s < 4; ++s) bad = bad || (((unsigned)__builtin_amdgcn_readlane((int)wfl[1], 4 * s) & META_SV) && min12(wp[s]) == 0u);
      if (LIKELY(!__any(bad))) break;
      if (rt() - t0 > DEADLINE) { __hip_atomic_store((GAS unsigned*)tmo, call, __ATOMIC_RELAXED, __HIP_MEMORY_SCOPE_AGENT); okw = false; break; }
    }
    STAMP2(1);
#pragma unroll
    for (int s = 0; s < 4; ++s) {
      const unsigned meta = (unsigned)__builtin_amdgcn_readlane((int)wfl[1], 4 * s);
      if (LIKELY((meta & 0x80000f00u) == 0u)) {
        wf[s][0] = unpack_frag<0, false>(wp[s], meta, 0u, 0u, 0, lane);
        wf[s][1] = unpack_frag<1, false>(wp[s], meta, 0u, 0u, 0, lane);
        wf[s][2] = unpack_frag<2, false>(wp[s], meta, 0u, 0u, 0, lane);
        wf[s][3] = unpack_frag<3, false>(wp[s], meta, 0u, 0u, 0, lane);
      } else if (meta & META_RAW) {
        wf[s][0] = wp[s][0]; wf[s][1] = wp[s][1]; wf[s][2] = wp[s][2]; wf[s][3] = ldp(soff + s * 4096, 3);
      } else {
        wf[s][0] = unpack_frag<0, true>(wp[s], meta, wfl[2], wfl[3], 4 * s, lane);
        wf[s][1] = unpack_frag<1, true>(wp[s], meta, wfl[2], wfl[3], 4 * s, lane);
        wf[s][2] = unpack_frag<2, true>(wp[s], meta, wfl[2], wfl[3], 4 * s, lane);
        wf[s][3] = unpack_frag<3, true>(wp[s], meta, wfl[2], wfl[3], 4 * s, lane);
      }
    }
  }
  STAMP2(21);
  f32x4 acc[2];
  auto mm = [&](int ks, u32x4 xf) {
#pragma unroll
    for (int jj = 0; jj < 2; ++jj)
      acc[jj] = __builtin_amdgcn_mfma_f32_16x16x32_bf16(__builtin_bit_cast(bf16x8, wf[ks >> 1][(ks & 1) * 2 + jj]), __builtin_bit_cast(bf16x8, xf), acc[jj], 0, 0, 0);
  };
  int pw[2];
  {
    const int m = lane & 15, qd = lane >> 4;
    const int sw = ((m & 1) << 2) | (m & 2) | ((m >> 2) & 1);
#pragma unroll
    for (int jj = 0; jj < 2; ++jj) pw[jj] = wv * 2048 + m * 128 + (((4 * jj + qd) ^ sw) * 16);
  }
  bool scr_free = false;
  auto finish = [&](int it, bool good) {
    if (UNLIKELY(!good)) {
      asm volatile("; poison" ::: "memory");
      const float qn = __builtin_nanf("");
#pragma unroll
      for (int jj = 0; jj < 2; ++jj) acc[jj] = f32x4{qn, qn, qn, qn};
    }
    if (UNLIKELY(it >= WSCR_IT && !scr_free)) {
      while (__hip_atomic_load(&ready[8], __ATOMIC_RELAXED, __HIP_MEMORY_SCOPE_WORKGROUP) < 8u) __builtin_amdgcn_s_sleep(1);
      scr_free = true;
    }
    char* pb = lds + it * 16384;
#pragma unroll
    for (int jj = 0; jj < 2; ++jj) *(f32x4*)(pb + pw[jj]) = acc[jj];
    if (lane == 0) __hip_atomic_fetch_add(&ready[it], 1u, __ATOMIC_RELEASE, __HIP_MEMORY_SCOPE_WORKGROUP);
  };
  if (own0) {
#pragma unroll
    for (int jj = 0; jj < 2; ++jj) acc[jj] = f32x4{0.f, 0.f, 0.f, 0.f};
#pragma unroll
    for (int s = 0; s < 8; ++s) mm(s, x0[s >> 2][s & 3]);
    finish(0, okw);
    STAMP(4);
  }
  auto step = [&](int it, u32x4 (&cur)[2][3], u32x4& cv, u32x4 (&nxt)[2][3], u32x4& ncv) {
#pragma unroll
    for (int h = 0; h < 2; ++h)
#pragma unroll
      for (int p = 0; p < 3; ++p) PIN(cur[h][p]);
    PIN(cv);
    bool cok = true;
    {
      auto complete = [&]() -> bool {
        const unsigned m0 = (unsigned)__builtin_amdgcn_readlane((int)cv[1], 0), m1 = (unsigned)__builtin_amdgcn_readlane((int)cv[1], 4);
        return lines_ready(cv, xline, first, call) && (m0 & m1 & META_SV) && __all(umin_(min12(cur[0]), min12(cur[1])) != 0u);
      };
      bool have = complete();
      if (!have) {
        cok = wait_lines(rs, xoff(code_of(it)), xline, first, 8, call, tmo, cv);
        const unsigned n0 = (unsigned)__builtin_amdgcn_readlane((int)cv[1], 0), n1 = (unsigned)__builtin_amdgcn_readlane((int)cv[1], 4);
        const u64 t0 = rt();
        for (;;) {
          load_x(cur, code_of(it));
          if (!__any(((n0 & META_SV) && min12(cur[0]) == 0u) || ((n1 & META_SV) && min12(cur[1]) == 0u))) break;
          if (rt() - t0 > DEADLINE) { __hip_atomic_store((GAS unsigned*)tmo, call, __ATOMIC_RELAXED, __HIP_MEMORY_SCOPE_AGENT); cok = false; break; }
        }
      }
    }
    if (it + 1 == nblk) STAMP2(20);
    cok = cok && (unsigned)__builtin_amdgcn_readlane((int)cv[0], 8) != call;
    const unsigned meta0 = (unsigned)__builtin_amdgcn_readlane((int)cv[1], 0), meta1 = (unsigned)__builtin_amdgcn_readlane((int)cv[1], 4);
    __builtin_amdgcn_fence(__ATOMIC_ACQUIRE, "wavefront");
    issue(nxt, ncv, it + 1);
    __builtin_amdgcn_sched_barrier(0);
#pragma unroll
    for (int jj = 0; jj < 2; ++jj) acc[jj] = f32x4{0.f, 0.f, 0.f, 0.f};
    if (LIKELY(((meta0 | meta1) & 0x80000f00u) == 0u)) {
#pragma unroll
      for (int h = 0; h < 2; ++h) {
        const unsigned meta = h ? meta1 : meta0;
        mm(4 * h + 0, unpack_frag<0, false>(cur[h], meta, 0u, 0u, 0, lane));
        mm(4 * h + 1, unpack_frag<1, false>(cur[h], meta, 0u, 0u, 0, lane));
        mm(4 * h + 2, unpack_frag<2, false>(cur[h], meta, 0u, 0u, 0, lane));
        mm(4 * h + 3, unpack_frag<3, false>(cur[h], meta, 0u, 0u, 0, lane));
      }
    } else {
#pragma unroll
      for (int h = 0; h < 2; ++h) {
        const unsigned meta = h ? meta1 : meta0;
        if (UNLIKELY(meta & META_RAW)) {
          mm(4 * h, cur[h][0]);
          cur[h][0] = ldp(xdata(code_of(it)) + h * 4096, 3);
          mm(4 * h + 1, cur[h][1]); mm(4 * h + 2, cur[h][2]);
          PIN(cur[h][0]);
          mm(4 * h + 3, cur[h][0]);
        } else {
          mm(4 * h + 0, unpack_frag<0, true>(cur[h], meta, cv[2], cv[3], 4 * h, lane));
          mm(4 * h + 1, unpack_frag<1, true>(cur[h], meta, cv[2], cv[3], 4 * h, lane));
          mm(4 * h + 2, unpack_frag<2, true>(cur[h], meta, cv[2], cv[3], 4 * h, lane));
          mm(4 * h + 3, unpack_frag<3, true>(cur[h], meta, cv[2], cv[3], 4 * h, lane));
        }
      }
    }
    finish(it, cok && okw);
    if (it < 6) STAMP(4 + it * 3);
  };
  for (int it = it0; it < nblk; it += 2) {
    step(it, bufA, cvA, bufB, cvB);
    if (it + 1 < nblk) step(it + 1, bufB, cvB, bufA, cvA);
  }
  STAMP(22);
  STAMP_FLUSH();
}

}

extern "C" void kernel_launch(int rank, int nranks, ncclComm_t comm, hipStream_t stream, void* const* d_in, const int* in_sizes,
                              int n_in, void* d_out, int out_size, const int* in_dtypes, int out_dtype, void* const* d_out_all,
                              void* const* d_ws_all, size_t ws_bytes) {
  using namespace agk;
  (void)comm; (void)in_dtypes; (void)out_dtype;
  if (nranks != NRANK || n_in < 2 || rank < 0 || rank >= NRANK || ws_bytes < WS_NEED || in_sizes[0] != MLOC * KDIM ||
      in_sizes[1] != KDIM * NLOC || out_size != MDIM * NLOC) {
    fprintf(stderr, "kernel_launch: unexpected problem shape\n");
    return;
  }
  const int type = rank & 1, idx = rank >> 1;
  Args a;
  memset(&a, 0, sizeof a);
  for (int link = 0; link < 7; ++link) {
    const bool same = link < 3;
    const int didx = same ? link + (link >= idx ? 1 : 0) : link - 3;
    a.link_ws[link] = (char*)d_ws_all[2 * didx + (same ? type : 1 - type)];
  }
  a.link_ws[7] = (char*)d_ws_all[rank];
  for (int cg = 0; cg < 16; ++cg) {
    const int dest = cg < 8 ? rank : 2 * ((cg - 8) >> 1) + (1 - type);
    a.obase[cg] = (char*)d_out_all[dest] + (cg < 8 ? cg * 32 : idx * 64 + (cg & 1) * 32) * 2;
    a.oflag[cg] = (char*)d_ws_all[dest] + OFF_OFLAG + (size_t)(idx * 128 + (cg & 1)) * FS * 4;
    a.otile[cg] = (char*)d_ws_all[dest] + OFF_OSTAGE + (size_t)(idx * 128 + (cg & 1)) * 1024;
  }
  auto make_list = [](int ty, int ix, bool with_bcast, int* code, int* row) -> int {
    int n = 0;
    for (int b = 0; b < 16; ++b)
      if (b < P || (with_bcast && b >= 16 - NB)) { code[n] = 0x40 | b; row[n] = (2 * ix + ty) * 16 + b; ++n; }
    for (int st = 0; st < 32; ++st) {
      if (st < SB && (st < P || with_bcast))
        for (int ia = 0; ia < 4; ++ia)
          if (ia != ix) { code[n] = (ia < ix ? ia : ia - 1) * SB + st; row[n] = (2 * ia + ty) * 16 + (st < P ? st : st + PO); ++n; }
      const int k = st - WSTEPS;
      if (k >= 0 && k < OB && (k < PO || with_bcast))
        for (int ia = 0; ia < 4; ++ia) { code[n] = 3 * SB + ia * OB + k; row[n] = (2 * ia + 1 - ty) * 16 + P + k; ++n; }
    }
    return n;
  };
  int code[80], row[80];
  for (int o = 0; o < 2; ++o) {
    const int nset = o == 0 ? NSET_O : NSET_R;
    const int n = make_list(type, idx, o == 0, code, row);
    for (int q = 0; q < n; ++q) {
      const int set = q % nset, i = q / nset;
      a.codes[o][set] |= (u64)code[q] << (8 * i);
      a.rows[o][set] |= (u64)row[q] << (8 * i);
      a.codes[o][set] += (u64)1 << 56;
    }
  }
  for (int src = 0; src < 4; ++src) {
    const int n = make_list(1 - type, src, false, code, row);
    for (int q = 0; q < n; ++q) a.orow[src][q] = (unsigned char)row[q];
  }
  for (int p = 0; p < NRANK; ++p) a.all_ws[p] = (char*)d_ws_all[p];
  a.myws = (char*)d_ws_all[rank];
  a.myout = (char*)d_out;
  a.x = (const char*)d_in[0];
  a.w = (const char*)d_in[1];
  a.rank = rank;
  hipLaunchKernelGGL(ag_gemm_kernel, dim3(NWG), dim3(NTHR), 0, stream, a);
}
```

```cpp
#include <hip/hip_runtime.h>
#include <rccl/rccl.h>
#include <cstdio>
#include <cstdint>
#include <cstring>

namespace agk {

typedef unsigned u32x4 __attribute__((ext_vector_type(4)));
typedef unsigned u32x2 __attribute__((ext_vector_type(2)));
typedef float f32x4 __attribute__((ext_vector_type(4)));
typedef __bf16 bf16x8 __attribute__((ext_vector_type(8)));
typedef unsigned long long u64;
#define GAS __attribute__((address_space(1)))
#define LAS __attribute__((address_space(3)))

constexpr int NRANK = 8;
constexpr int MDIM = 2048, KDIM = 2048, NLOC = 256, MLOC = 256;
constexpr int P = 10, NB = 1, PO = 16 - P - NB;
constexpr int SB = P + NB, OB = PO + NB;
constexpr int NL_REM = 4 * P + 4 * PO;
constexpr int NL_OWN = NL_REM + 8 * NB;
constexpr int WSTEPS = 4;

constexpr int WGPL = 8;
constexpr int NSEND_WG = 7 * WGPL;
constexpr int SW = 2;
constexpr int WPL = WGPL * SW;
constexpr int NSET = 12;
constexpr int NCOMP_WG = 16 * NSET;
constexpr int NWG = NSEND_WG + NCOMP_WG;
constexpr int NWV = 12;
constexpr int NTHR = NWV * 64;
constexpr int MAXB = (NL_OWN + NSET - 1) / NSET;
static_assert(WPL == 16 && NWG <= 256 && MAXB <= 6 && NL_REM <= 64 && 3 * SB + 4 * OB <= 64, "");
static_assert(SB <= NSET, "only a workgroup's first block can be one of the rank's own");
constexpr int LDS_RDY = 112 << 10;
constexpr int LDS_WSCR = LDS_RDY - 8 * 12288;
constexpr int LDS_BYTES = LDS_RDY + 64;
static_assert(MAXB * 16384 <= LDS_RDY && SW * 12288 <= LDS_RDY, "");

constexpr size_t OFF_CNT = 0;
constexpr size_t CNT_BYTES = (size_t)NWG * NWV * 32;
constexpr size_t OFF_TMO = 128 << 10;
constexpr int NRX = (3 * SB + 4 * OB) * 16, NRU = NRX + 256;
constexpr size_t OFF_RLINE = 132 << 10, RLINE_BYTES = (size_t)NRU * 128;
constexpr size_t OFF_OFLAG = OFF_RLINE + 3 * RLINE_BYTES;
constexpr size_t OFF_FLAG_END = OFF_OFLAG + (size_t)512 * 128;
constexpr size_t OFF_RSTAGE = 1 << 20, RSTAGE_BYTES = (size_t)NRU * 4096;
constexpr size_t OFF_OSTAGE = OFF_RSTAGE + 3 * RSTAGE_BYTES;
constexpr size_t WS_NEED = OFF_OSTAGE + (size_t)512 * 1024;
static_assert(CNT_BYTES <= OFF_TMO && OFF_FLAG_END <= OFF_RSTAGE, "");
constexpr u64 DEADLINE = 50000000ull;

constexpr unsigned META_RAW = 0x80000000u;
constexpr unsigned META_SV = 0x40000000u;

struct Args {
  char* link_ws[8];
  char* oflag[8];
  char* otile[8];
  unsigned char orow[4][64];
  u64 codes[2][16];
  u64 rows[2][16];
  char* myws;
  char* myout;
  const char* x;
  const char* w;
  int rank;
  int pad;
};

template <int OFF>
__device__ __forceinline__ void st16_wt(char* p, u32x4 v) {
  asm volatile("global_store_dwordx4 %0, %1, off offset:%c2 sc0 sc1\n\ts_nop 1" ::"v"(p), "v"(v), "i"(OFF) : "memory");
}
template <int OFF>
__device__ __forceinline__ void st16_wt_s(const char* base, unsigned voff, u32x4 v) {
  asm volatile("global_store_dwordx4 %0, %1, %2 offset:%c3 sc0 sc1\n\ts_nop 1" ::"v"(voff), "v"(v), "s"(base), "i"(OFF) : "memory");
}
__device__ __forceinline__ void st4_wt(char* p, unsigned v) {
  asm volatile("global_store_dword %0, %1, off sc0 sc1\n\ts_nop 1" ::"v"(p), "v"(v) : "memory");
}
#define WAIT_VM0() asm volatile("s_waitcnt vmcnt(0)" ::: "memory")
#define PIN(x) asm volatile("" : "+v"(x))
#define LIKELY(x) __builtin_expect(!!(x), 1)
#define UNLIKELY(x) __builtin_expect(!!(x), 0)

__device__ __forceinline__ u64 rt() { return __builtin_amdgcn_s_memrealtime(); }
__device__ __forceinline__ unsigned umin_(unsigned a, unsigned b) { return a < b ? a : b; }
__device__ __forceinline__ unsigned umax_(unsigned a, unsigned b) { return a > b ? a : b; }
__device__ __forceinline__ unsigned min4(const u32x4& v) { return umin_(umin_(v[0], v[1]), umin_(v[2], v[3])); }
__device__ __forceinline__ unsigned min12(const u32x4 (&v)[3]) {
  unsigned m = umin_(umin_(v[0][0], v[0][1]), v[0][2]);
  m = umin_(umin_(m, v[0][3]), v[1][0]);
  m = umin_(umin_(m, v[1][1]), v[1][2]);
  m = umin_(umin_(m, v[1][3]), v[2][0]);
  m = umin_(umin_(m, v[2][1]), v[2][2]);
  return umin_(m, v[2][3]);
}

constexpr unsigned OOB = 0x7ffffff0u;
__device__ __forceinline__ u32x4 poll_load(__amdgpu_buffer_rsrc_t rc, unsigned voff) {
  return __builtin_bit_cast(u32x4, __builtin_amdgcn_raw_buffer_load_b128(rc, voff, 0, 17));
}
__device__ __forceinline__ bool lines_ready(const u32x4& v, bool line_lane, bool first, unsigned call) {
  const unsigned late = (unsigned)__builtin_amdgcn_update_dpp(0, (int)v[0], 0x101, 0xf, 0xf, false);
  return __all(!line_lane || (first ? v[0] >= call && ((v[1] & META_SV) || late >= call) : v[1] >= call));
}
__device__ __forceinline__ bool wait_lines(__amdgpu_buffer_rsrc_t rc, unsigned voff, bool line_lane, bool first, int tmo_lane, unsigned call,
                                           unsigned* tmo, u32x4& fl) {
  const u64 t0 = rt();
  for (;;) {
    fl = poll_load(rc, voff);
    const unsigned t = (unsigned)__builtin_amdgcn_readlane((int)fl[0], tmo_lane);
    if (lines_ready(fl, line_lane, first, call)) { __builtin_amdgcn_fence(__ATOMIC_ACQUIRE, "wavefront"); return t != call; }
    if (t == call || rt() - t0 > DEADLINE) {
      __hip_atomic_store((GAS unsigned*)tmo, call, __ATOMIC_RELAXED, __HIP_MEMORY_SCOPE_AGENT);
      fl = u32x4{META_RAW, META_RAW, META_RAW, META_RAW};
      return false;
    }
    __builtin_amdgcn_s_sleep(1);
  }
}

__device__ __forceinline__ unsigned x_lane_off(int lane, int i) {
  const int r = 4 * i + (lane >> 4);
  return (unsigned)(r * (KDIM * 2) + (((lane & 15) ^ r) * 16));
}
__device__ __forceinline__ void x_fetch(const char* x, int blk, int u, const unsigned (&off)[4], char* scratch) {
  const GAS char* base = (const GAS char*)x + (size_t)(blk * 16 * (KDIM * 2) + u * 256);
#pragma unroll
  for (int i = 0; i < 4; ++i) __builtin_amdgcn_global_load_lds((const GAS void*)(base + off[i]), (LAS void*)(scratch + i * 1024), 16, 0, 0);
}
__device__ __forceinline__ void x_read(const char* scratch, int lane, u32x4 (&v)[4]) {
  const int m = lane & 15, q = lane >> 4;
  const char* p = scratch + m * 256;
#pragma unroll
  for (int s = 0; s < 4; ++s) v[s] = *(const LAS u32x4*)(p + (((4 * s + q) ^ m) * 16));
}
__device__ __forceinline__ unsigned w_lane_off(int lane) { return (unsigned)((lane >> 2) * (NLOC * 2) + (lane & 3) * 16); }
__device__ __forceinline__ void w_fetch(const char* w, int cg32, int ks2, unsigned off, char* scratch) {
  const GAS char* base = (const GAS char*)w + (size_t)(ks2 * 64 * (NLOC * 2) + cg32 * 64);
#pragma unroll
  for (int i = 0; i < 4; ++i)
    __builtin_amdgcn_global_load_lds((const GAS void*)(base + i * 16 * (NLOC * 2) + off), (LAS void*)(scratch + i * 1024), 16, 0, 0);
}
__device__ __forceinline__ void w_read(const char* scratch, int lane, u32x4 (&v)[4]) {
  typedef short s16x4 __attribute__((ext_vector_type(4)));
  const int g = lane >> 4, i = lane & 15;
  const char* p = scratch + (8 * g + (i >> 2)) * 64 + (i & 3) * 8;
#pragma unroll
  for (int f = 0; f < 4; ++f)
#pragma unroll
    for (int h = 0; h < 2; ++h) {
      const u32x2 t = __builtin_bit_cast(u32x2, __builtin_amdgcn_ds_read_tr16_b64_v4i16((LAS s16x4*)(p + ((f >> 1) * 32 + 4 * h) * 64 + (f & 1) * 32)));
      v[f][2 * h] = t[0];
      v[f][2 * h + 1] = t[1];
    }
}
__device__ __forceinline__ unsigned lds_off(const char* p) { return (unsigned)(size_t)(const LAS char*)p; }
__device__ __forceinline__ void w_read_asm(unsigned sc, int lane, u32x4 (&v)[4]) {
  const int g = lane >> 4, i = lane & 15;
  const unsigned p = sc + (8 * g + (i >> 2)) * 64 + (i & 3) * 8;
  u32x2 t0, t1, t2, t3, t4, t5, t6, t7;
  asm volatile("ds_read_b64_tr_b16 %0, %8\n\tds_read_b64_tr_b16 %1, %8 offset:256\n\tds_read_b64_tr_b16 %2, %8 offset:32\n\t"
               "ds_read_b64_tr_b16 %3, %8 offset:288\n\tds_read_b64_tr_b16 %4, %8 offset:2048\n\tds_read_b64_tr_b16 %5, %8 offset:2304\n\t"
               "ds_read_b64_tr_b16 %6, %8 offset:2080\n\tds_read_b64_tr_b16 %7, %8 offset:2336\n\ts_waitcnt lgkmcnt(0)"
               : "=&v"(t0), "=&v"(t1), "=&v"(t2), "=&v"(t3), "=&v"(t4), "=&v"(t5), "=&v"(t6), "=&v"(t7) : "v"(p) : "memory");
  v[0] = u32x4{t0[0], t0[1], t1[0], t1[1]};
  v[1] = u32x4{t2[0], t2[1], t3[0], t3[1]};
  v[2] = u32x4{t4[0], t4[1], t5[0], t5[1]};
  v[3] = u32x4{t6[0], t6[1], t7[0], t7[1]};
}
__device__ __forceinline__ void x_read_issue(unsigned sc, const unsigned (&ro)[4], u32x4 (&v)[4]) {
  asm volatile("ds_read_b128 %0, %4\n\tds_read_b128 %1, %5\n\tds_read_b128 %2, %6\n\tds_read_b128 %3, %7"
               : "=&v"(v[0]), "=&v"(v[1]), "=&v"(v[2]), "=&v"(v[3]) : "v"(sc + ro[0]), "v"(sc + ro[1]), "v"(sc + ro[2]), "v"(sc + ro[3]) : "memory");
}
__device__ __forceinline__ void lds_wait4(u32x4 (&v)[4]) {
  asm volatile("s_waitcnt lgkmcnt(0)" : "+v"(v[0]), "+v"(v[1]), "+v"(v[2]), "+v"(v[3]) :: "memory");
}

__device__ __forceinline__ unsigned wave_max(unsigned v) {
  v = umax_(v, (unsigned)__builtin_amdgcn_update_dpp(0, (int)v, 0x111, 0xf, 0xf, false));
  v = umax_(v, (unsigned)__builtin_amdgcn_update_dpp(0, (int)v, 0x112, 0xf, 0xf, false));
  v = umax_(v, (unsigned)__builtin_amdgcn_update_dpp(0, (int)v, 0x114, 0xf, 0xf, false));
  v = umax_(v, (unsigned)__builtin_amdgcn_update_dpp(0, (int)v, 0x118, 0xf, 0xf, false));
  v = umax_(v, (unsigned)__builtin_amdgcn_update_dpp(0, (int)v, 0x142, 0xa, 0xf, false));
  v = umax_(v, (unsigned)__builtin_amdgcn_update_dpp(0, (int)v, 0x143, 0xc, 0xf, false));
  return (unsigned)__builtin_amdgcn_readlane((int)v, 63);
}
typedef unsigned short u16x2 __attribute__((ext_vector_type(2)));
__device__ __forceinline__ unsigned pk_max_u16(unsigned a, unsigned b) {
  return __builtin_bit_cast(unsigned, __builtin_elementwise_max(__builtin_bit_cast(u16x2, a), __builtin_bit_cast(u16x2, b)));
}
__device__ __forceinline__ unsigned bfi(unsigned mask, unsigned a, unsigned b) {
  unsigned r;
  asm("v_bfi_b32 %0, %1, %2, %3" : "=v"(r) : "s"(mask), "v"(a), "v"(b));
  return r;
}
struct Packed { u32x4 s[4]; u32x2 ent; unsigned meta; };
__device__ __forceinline__ void pack_unit(const u32x4 (&f)[4], int lane, Packed& o) {
  u32x4 L0, L1, C;
  unsigned hq[8], uq[8];
#pragma unroll
  for (int i = 0; i < 4; ++i) {
    hq[2 * i] = __builtin_amdgcn_perm(f[i][1], f[i][0], 0x07050301u);
    hq[2 * i + 1] = __builtin_amdgcn_perm(f[i][3], f[i][2], 0x07050301u);
    uq[2 * i] = hq[2 * i] | 0x80808080u;
    uq[2 * i + 1] = hq[2 * i + 1] | 0x80808080u;
  }
  const unsigned mo = pk_max_u16(pk_max_u16(pk_max_u16(uq[0], uq[1]), pk_max_u16(uq[2], uq[3])), pk_max_u16(pk_max_u16(uq[4], uq[5]), pk_max_u16(uq[6], uq[7])));
  const unsigned me = pk_max_u16(pk_max_u16(pk_max_u16(uq[0] << 8, uq[1] << 8), pk_max_u16(uq[2] << 8, uq[3] << 8)),
                                 pk_max_u16(pk_max_u16(uq[4] << 8, uq[5] << 8), pk_max_u16(uq[6] << 8, uq[7] << 8)));
  const unsigned m2 = pk_max_u16(mo, me);
  const unsigned mx = (wave_max(umax_(m2 & 0xffffu, m2 >> 16)) >> 8) & 0x7fu;
  const unsigned base = (mx > 7u ? mx : 7u) - 7u;
  const unsigned base4 = base * 0x01010101u;
  unsigned E[4];
#pragma unroll
  for (int i = 0; i < 4; ++i) {
    const unsigned l0 = __builtin_amdgcn_perm(f[i][1], f[i][0], 0x06040200u), l1 = __builtin_amdgcn_perm(f[i][3], f[i][2], 0x06040200u);
    const unsigned t0 = uq[2 * i] - base4, t1 = uq[2 * i + 1] - base4;
    E[i] = t0 & t1;
    C[i] = bfi(0x0f0f0f0fu, bfi(0x07070707u, t0, hq[2 * i] >> 4), bfi(0x70707070u, t1 << 4, hq[2 * i + 1]));
    if (i < 2) { L0[2 * i] = l0; L0[2 * i + 1] = l1; } else { L1[2 * i - 4] = l0; L1[2 * i - 3] = l1; }
  }
  unsigned fmask = 0;
  bool raw = false;
  u32x2 ent = {0u, 0u};
  if (UNLIKELY(__any((E[0] & E[1] & E[2] & E[3] & 0x80808080u) != 0x80808080u))) {
#pragma unroll
    for (int i = 0; i < 4; ++i) {
      u64 bm = __ballot((E[i] & 0x80808080u) != 0x80808080u);
      unsigned cnt = 0, e0 = 0, e1 = 0;
      while (bm && cnt <= 2u) {
        const int l = __builtin_ctzll(bm);
        bm &= bm - 1;
#pragma unroll
        for (int q = 0; q < 2; ++q) {
          const unsigned h = (unsigned)__builtin_amdgcn_readlane((int)hq[2 * i + q], l);
          unsigned m3 = ~((h | 0x80808080u) - base4) & 0x80808080u;
          while (m3) {
            const unsigned b = (unsigned)__builtin_ctz(m3) >> 3;
            m3 &= m3 - 1;
            const unsigned e = 0x80000000u | ((unsigned)l << 16) | ((4u * q + b) << 8) | ((h >> (8u * b)) & 0xffu);
            e1 = cnt == 1u ? e : e1;
            e0 = cnt == 0u ? e : e0;
            ++cnt;
          }
        }
      }
      raw = raw || cnt > 2u;
      fmask |= cnt ? 1u << i : 0u;
      ent[0] = lane == i ? e0 : ent[0];
      ent[1] = lane == i ? e1 : ent[1];
    }
  }
  o.ent = ent;
  if (UNLIKELY(raw)) {
    o.s[0] = f[0]; o.s[1] = f[1]; o.s[2] = f[2]; o.s[3] = f[3];
    o.meta = META_RAW;
  } else {
    o.s[0] = L0; o.s[1] = L1; o.s[2] = C; o.s[3] = C;
    const u32x4 t[3] = {L0, L1, C};
    o.meta = (fmask << 8) | base | (__all(min12(t) != 0u) ? META_SV : 0u);
  }
}
__device__ __forceinline__ void patch_frag(u32x4& o, unsigned e, int lane) {
  const unsigned sh = 8u + 16u * ((e >> 8) & 1u), d = (e >> 9) & 3u;
  const unsigned vm = (unsigned)lane == ((e >> 16) & 63u) ? 0xffu << sh : 0u, put = (e & 0xffu) << sh;
#pragma unroll
  for (int t = 0; t < 4; ++t) {
    const unsigned m = d == (unsigned)t ? vm : 0u;
    o[t] = (o[t] & ~m) | (put & m);
  }
}
template <int I, bool EXC>
__device__ __forceinline__ u32x4 unpack_frag(const u32x4 (&v)[3], unsigned meta, unsigned e0, unsigned e1, int fb, int lane) {
  const unsigned base4 = (meta & 0x7fu) * 0x01010101u;
  const unsigned l0 = v[I >> 1][2 * (I & 1)], l1 = v[I >> 1][2 * (I & 1) + 1];
  const unsigned c = v[2][I];
  const unsigned h0 = ((c & 0x08080808u) << 4) | ((c & 0x07070707u) + base4);
  const unsigned h1 = (c & 0x80808080u) | (((c >> 4) & 0x07070707u) + base4);
  u32x4 o;
  o[0] = __builtin_amdgcn_perm(h0, l0, 0x05010400u);
  o[1] = __builtin_amdgcn_perm(h0, l0, 0x07030602u);
  o[2] = __builtin_amdgcn_perm(h1, l1, 0x05010400u);
  o[3] = __builtin_amdgcn_perm(h1, l1, 0x07030602u);
  if (EXC && (meta & (0x100u << I))) {
    patch_frag(o, (unsigned)__builtin_amdgcn_readlane((int)e0, fb + I), lane);
    const unsigned e = (unsigned)__builtin_amdgcn_readlane((int)e1, fb + I);
    if (e) patch_frag(o, e, lane);
  }
  return o;
}

__device__ __forceinline__ float gelu_f(float y) {
  const float c = 0.7978845608028654f;
  const float k2 = -2.0f * 1.4426950408889634f * c;
  const float t = y * (1.0f + 0.044715f * y * y);
  const float e = __builtin_amdgcn_exp2f(k2 * t);
  return y * __builtin_amdgcn_rcpf(1.0f + e);
}
__device__ __forceinline__ unsigned pack_bf16(float lo, float hi) {
  typedef __bf16 bf2 __attribute__((ext_vector_type(2)));
  typedef float f2 __attribute__((ext_vector_type(2)));
  f2 f = {lo, hi};
  return __builtin_bit_cast(unsigned, __builtin_convertvector(f, bf2));
}

__global__ void __launch_bounds__(NTHR, 3) ag_gemm_kernel(Args a) {
  __shared__ __attribute__((aligned(16))) char lds[LDS_BYTES];
  const int tid = threadIdx.x, lane = tid & 63;
  const int wv = __builtin_amdgcn_readfirstlane(tid >> 6);
  const int bid = blockIdx.x;
  const int idx = a.rank >> 1;
  char* const myws = a.myws;
  unsigned* const tmo = (unsigned*)(myws + OFF_TMO);

  auto get_call = [&]() -> unsigned {
    asm volatile("" ::: "memory");
    unsigned ticket = 0;
    if (lane == 0)
      ticket = __hip_atomic_fetch_add((GAS unsigned*)(myws + OFF_CNT + (size_t)(bid * NWV + wv) * 32), 1u, __ATOMIC_RELAXED, __HIP_MEMORY_SCOPE_AGENT);
    return (unsigned)__builtin_amdgcn_readfirstlane(ticket) + 1u;
  };

  if (bid < NSEND_WG) {
    const int link = bid % 7, lq = bid / 7;
    if (wv >= SW) {
      const u64 tstart = rt();
      const unsigned call = get_call();
      const __amdgpu_buffer_rsrc_t ro = __builtin_amdgcn_make_buffer_rsrc((void*)myws, 0, (int)WS_NEED, 0x00020000);
      while (rt() - tstart < 1300u) __builtin_amdgcn_s_sleep(8);
      for (int o = bid * (NWV - SW) + wv - SW; o < NL_REM * 8; o += NSEND_WG * (NWV - SW)) {
        const int q = o >> 3, src = (o >> 1) & 3, half = o & 1;
        const int t = src * 128 + q * 2 + half;
        const u64 t0 = rt();
        auto ldt = [&]() -> u32x4 {
          return __builtin_bit_cast(u32x4, __builtin_amdgcn_raw_buffer_load_b128(ro, lane * 16, (int)(OFF_OSTAGE + (size_t)t * 1024), 17));
        };
        u32x4 v;
        bool good;
        for (;;) {
          v = ldt();
          const unsigned tm = __hip_atomic_load((GAS unsigned*)tmo, __ATOMIC_RELAXED, __HIP_MEMORY_SCOPE_AGENT);
          if (__all(min4(v) != 0u)) { good = tm != call; break; }
          const unsigned f = __hip_atomic_load((const GAS unsigned*)(myws + OFF_OFLAG + (size_t)t * 128), __ATOMIC_RELAXED, __HIP_MEMORY_SCOPE_SYSTEM);
          if (f >= call) {
            __builtin_amdgcn_fence(__ATOMIC_ACQUIRE, "wavefront");
            v = ldt();
            good = tm != call;
            break;
          }
          if (tm == call || rt() - t0 > DEADLINE) {
            __hip_atomic_store((GAS unsigned*)tmo, call, __ATOMIC_RELAXED, __HIP_MEMORY_SCOPE_AGENT);
            good = false;
            break;
          }
          __builtin_amdgcn_s_sleep(1);
        }
        if (UNLIKELY(!good)) v = u32x4{0x7fc07fc0u, 0x7fc07fc0u, 0x7fc07fc0u, 0x7fc07fc0u};
        st16_wt<0>(a.myout + (size_t)(a.orow[src][q] * 16 + (lane >> 2)) * (NLOC * 2) + (src * 64 + half * 32) * 2 + (lane & 3) * 16, v);
        PIN(v);
        st16_wt<0>(myws + OFF_OSTAGE + (size_t)t * 1024 + lane * 16, u32x4{0u, 0u, 0u, 0u});
      }

      return;
    }
    const int j = lq * SW + wv;
    const bool same = link < 3;
    const int didx = same ? link + (link >= idx ? 1 : 0) : link - 3;
    char* const dws = a.link_ws[link];
    const int b0 = same ? 0 : P, nb = same ? SB : OB;
    const int rx0 = (same ? (idx < didx ? idx : idx - 1) * SB : 3 * SB + idx * OB) * 16, rw0 = NRX + idx * 64;
    const unsigned sc0 = lds_off(lds) + (unsigned)wv * 12288u;
    const unsigned xo[4] = {x_lane_off(lane, 0), x_lane_off(lane, 1), x_lane_off(lane, 2), x_lane_off(lane, 3)};
    const unsigned wo = w_lane_off(lane);
    const char* const xsrc = a.x + (size_t)b0 * 65536 + j * 256;
    const char* const wsrc = a.w + (size_t)(j >> 1) * 64 * (NLOC * 2) + (didx * 2 + (j & 1)) * 64;
    auto fetch_x = [&](int k, unsigned sc) {
      const int kb = same && k >= P ? k + PO : k;
      const GAS char* b = (const GAS char*)xsrc + (size_t)kb * 65536;
#pragma unroll
      for (int i = 0; i < 4; ++i) __builtin_amdgcn_global_load_lds((const GAS void*)(b + xo[i]), (LAS void*)(size_t)(sc + i * 1024), 16, 0, 0);
    };
    auto fetch_w = [&](int t, unsigned sc) {
      const GAS char* b = (const GAS char*)wsrc + (size_t)t * 8 * 64 * (NLOC * 2);
#pragma unroll
      for (int i = 0; i < 4; ++i)
        __builtin_amdgcn_global_load_lds((const GAS void*)(b + i * 16 * (NLOC * 2) + wo), (LAS void*)(size_t)(sc + i * 1024), 16, 0, 0);
    };
    if (same) { fetch_x(0, sc0); fetch_x(1, sc0 + 4096); } else { fetch_w(0, sc0); fetch_w(1, sc0 + 4096); }
    const unsigned call = get_call();
    const unsigned buf = call % 3u;
    const char* const stage = dws + OFF_RSTAGE + (size_t)buf * RSTAGE_BYTES;
    char* const lines = dws + OFF_RLINE + (size_t)buf * RLINE_BYTES;
    const unsigned l16 = (unsigned)lane * 16u;
    unsigned ro[4];
    {
      const int m = lane & 15, q = lane >> 4;
#pragma unroll
      for (int i = 0; i < 4; ++i) ro[i] = (unsigned)(m * 256 + (((4 * i + q) ^ m) * 16));
    }
    WAIT_VM0();
    u32x4 v[4];
    Packed pc;
    auto send = [&](int ui) {
      const char* d = stage + (size_t)ui * 4096;
      st16_wt_s<0>(d, l16, pc.s[0]);
      st16_wt_s<1024>(d, l16, pc.s[1]);
      st16_wt_s<2048>(d, l16, pc.s[2]);
      if (UNLIKELY(pc.meta & META_RAW)) st16_wt_s<3072>(d, l16, pc.s[3]);
      char* const line = lines + (size_t)ui * 128;
      const u32x4 lv = {lane == 0 ? call : 0u, lane == 0 ? pc.meta : call, pc.ent[0], pc.ent[1]};
      if (lane < 4) st16_wt<0>(line + lane * 16, lv);
      if (UNLIKELY(!(pc.meta & META_SV))) {
        WAIT_VM0();
        if (lane == 0) st4_wt(line + 16, call);
      }
      asm volatile("s_waitcnt vmcnt(8)" ::: "memory");
    };
    unsigned sa = sc0, sb = sc0 + 4096, sc = sc0 + 8192;
    if (!same) {
#pragma nounroll
      for (int t = 0; t < 4; ++t) {
        w_read_asm(sa, lane, v);
        if (t < 2) fetch_w(t + 2, sc); else fetch_x(t - 2, sc);
        pack_unit(v, lane, pc);
        send(rw0 + (j & 1) * 32 + (j >> 1) + 8 * t);
        const unsigned r = sa; sa = sb; sb = sc; sc = r;
      }
    }
#pragma nounroll
    for (int k = 0; k < nb; ++k) {
      x_read_issue(sa, ro, v);
      if (k + 2 < nb) fetch_x(k + 2, sc);
      lds_wait4(v);
      pack_unit(v, lane, pc);
      send(rx0 + k * 16 + j);
      const unsigned r = sa; sa = sb; sb = sc; sc = r;
    }

    return;
  }

  const int cb = bid - NSEND_WG;
  const int xl = cb & 7, loc = cb >> 3;
  const bool own = loc < NSET;
  const int l2 = own ? loc : loc - NSET;
  const int set = (xl >> 1) + 4 * (l2 >> 2);
  const int cg = (xl & 1) * 4 + (l2 & 3);
  const u64 slotpk = a.codes[own ? 0 : 1][set];
  const int nblk = (int)(slotpk >> 56);
  unsigned* const ready = (unsigned*)(lds + LDS_RDY);
  if (tid < 16) ready[tid] = 0;
  asm volatile("s_waitcnt lgkmcnt(0)" ::: "memory");
  __builtin_amdgcn_s_barrier();
  if (wv >= 8) {
    const int h = wv - 8;
    const unsigned call = get_call();
    {
      const u64 tz = rt(), dz = 100u + (u64)((h * NCOMP_WG + cb) % 96) * 10u;
      while (rt() - tz < dz) __builtin_amdgcn_s_sleep(8);
      char* const zb = myws + OFF_RSTAGE + (size_t)((call + 2u) % 3u) * RSTAGE_BYTES + lane * 16;
      const u32x4 z = {0u, 0u, 0u, 0u};
      for (int zu = h * NCOMP_WG + cb; zu < NRU; zu += 4 * NCOMP_WG) {
        st16_wt<0>(zb + (size_t)zu * 4096, z);
        st16_wt<1024>(zb + (size_t)zu * 4096, z);
        st16_wt<2048>(zb + (size_t)zu * 4096, z);
      }
    }
    const u64 rows = a.rows[own ? 0 : 1][set];
    for (int it = h; it < nblk; it += 4) {
      const int row = lane >> 2, m = lane & 3;
      while (__hip_atomic_load(&ready[it], __ATOMIC_RELAXED, __HIP_MEMORY_SCOPE_WORKGROUP) < 8u) __builtin_amdgcn_s_sleep(1);
      __builtin_amdgcn_fence(__ATOMIC_ACQUIRE, "workgroup");
      const char* pb = lds + it * 16384 + row * 128;
      const int sw = ((row & 1) << 2) | (row & 2) | ((row >> 2) & 1);
      f32x4 pl[8], ph[8];
#pragma unroll
      for (int p = 0; p < 8; ++p) {
        pl[p] = *(const f32x4*)(pb + p * 2048 + (((2 * m) ^ sw) * 16));
        ph[p] = *(const f32x4*)(pb + p * 2048 + (((2 * m + 1) ^ sw) * 16));
      }
      __builtin_amdgcn_sched_barrier(0);
      const f32x4 lo = ((pl[0] + pl[1]) + (pl[2] + pl[3])) + ((pl[4] + pl[5]) + (pl[6] + pl[7]));
      const f32x4 hi = ((ph[0] + ph[1]) + (ph[2] + ph[3])) + ((ph[4] + ph[5]) + (ph[6] + ph[7]));
      u32x4 o;
      o[0] = pack_bf16(gelu_f(lo[0]), gelu_f(lo[1]));
      o[1] = pack_bf16(gelu_f(lo[2]), gelu_f(lo[3]));
      o[2] = pack_bf16(gelu_f(hi[0]), gelu_f(hi[1]));
      o[3] = pack_bf16(gelu_f(hi[2]), gelu_f(hi[3]));
      if (own) {
        const int gblk = (int)(rows >> (8 * it)) & 127;
        st16_wt<0>(a.myout + (size_t)(gblk * 16 + row) * (NLOC * 2) + cg * 64 + m * 16, o);
      } else {
        const int q2 = (set + NSET * it) * 2;
        st16_wt<0>(a.otile[cg] + (size_t)q2 * 1024 + lane * 16, o);
        if (UNLIKELY(!__all(min4(o) != 0u))) {
          WAIT_VM0();
          if (lane == 0) st4_wt(a.oflag[cg] + (size_t)q2 * 128, call);
        }
      }
    }

    return;
  }
  const bool own0 = (slotpk & 0x40u) != 0u;
  const int ownblk = (int)slotpk & 15;
  char* const wscr = lds + LDS_WSCR + wv * 12288;
  const unsigned xoffs[4] = {x_lane_off(lane, 0), x_lane_off(lane, 1), x_lane_off(lane, 2), x_lane_off(lane, 3)};
  auto own_fetch = [&](int r) {
    if (own) {
      w_fetch(a.w, cg, wv * 4 + 2 * r, w_lane_off(lane), wscr);
      w_fetch(a.w, cg, wv * 4 + 2 * r + 1, w_lane_off(lane), wscr + 4096);
    }
    if (own0) x_fetch(a.x, ownblk, wv * 2 + r, xoffs, wscr + 8192);
  };
  own_fetch(0);
  const unsigned call = get_call();
  const __amdgpu_buffer_rsrc_t rs = __builtin_amdgcn_make_buffer_rsrc((void*)myws, 0, (int)WS_NEED, 0x00020000);
  const unsigned buf = call % 3u;
  const unsigned rdata = (unsigned)OFF_RSTAGE + buf * (unsigned)RSTAGE_BYTES, rline = (unsigned)OFF_RLINE + buf * (unsigned)RLINE_BYTES;
  auto code_of = [&](int it) -> unsigned { return (unsigned)(slotpk >> (8 * it)) & 63u; };
  auto xdata = [&](unsigned code) -> int { return (int)(rdata + code * 65536u + (unsigned)wv * 8192u); };
  const bool xline = lane < 8, first = (lane & 3) == 0;
  const unsigned xlane = (unsigned)((wv * 2 + (lane >> 2)) * 128 + (lane & 3) * 16);
  auto xoff = [&](unsigned code) -> unsigned { return lane < 8 ? rline + code * 2048u + xlane : lane == 8 ? (unsigned)OFF_TMO : OOB; };
  auto ldp = [&](int soff, int piece) -> u32x4 {
    return __builtin_bit_cast(u32x4, __builtin_amdgcn_raw_buffer_load_b128(rs, lane * 16 + piece * 1024, soff, 17));
  };
  auto load_x = [&](u32x4 (&af)[2][3], unsigned code) {
    const int so = xdata(code);
#pragma unroll
    for (int h = 0; h < 2; ++h)
#pragma unroll
      for (int p = 0; p < 3; ++p) af[h][p] = ldp(so + h * 4096, p);
  };
  auto issue = [&](u32x4 (&af)[2][3], u32x4& cv, int it) {
    cv = poll_load(rs, xoff(code_of(it)));
    load_x(af, code_of(it));
  };
  u32x4 wf[4][4];
  u32x4 x0[2][4];
  bool okw = true;
#pragma unroll
  for (int r = 0; r < 2; ++r) {
    WAIT_VM0();
    if (own) {
      w_read(wscr, lane, wf[2 * r]);
      w_read(wscr + 4096, lane, wf[2 * r + 1]);
#pragma unroll
      for (int f = 0; f < 4; ++f) { PIN(wf[2 * r][f]); PIN(wf[2 * r + 1][f]); }
    }
    if (own0) {
      x_read(wscr + 8192, lane, x0[r]);
#pragma unroll
      for (int f = 0; f < 4; ++f) PIN(x0[r][f]);
    }
    if (r == 0) own_fetch(1);
  }
  constexpr int WSCR_IT = LDS_WSCR / 16384;
  if (lane == 0) __hip_atomic_fetch_add(&ready[8], 1u, __ATOMIC_RELEASE, __HIP_MEMORY_SCOPE_WORKGROUP);
  u32x4 wfl = {0u, 0u, 0u, 0u};
  const unsigned wunit = (unsigned)(NRX + cg * 32 + wv * 4);
  if (!own)
    okw = wait_lines(rs, lane < 16 ? rline + wunit * 128u + (unsigned)((lane >> 2) * 128 + (lane & 3) * 16) : lane == 16 ? (unsigned)OFF_TMO : OOB,
                     lane < 16, first, 16, call, tmo, wfl);
  const int it0 = own0 ? 1 : 0;
  u32x4 bufA[2][3], bufB[2][3], cvA, cvB;
  issue(bufA, cvA, it0);
  if (!own) {
    const int soff = (int)(rdata + wunit * 4096u);
    u32x4 wp[4][3];
    const u64 t0 = rt();
    for (;;) {
#pragma unroll
      for (int s = 0; s < 4; ++s)
#pragma unroll
        for (int p = 0; p < 3; ++p) wp[s][p] = ldp(soff + s * 4096, p);
#pragma unroll
      for (int s = 0; s < 4; ++s)
#pragma unroll
        for (int p = 0; p < 3; ++p) PIN(wp[s][p]);
      bool bad = false;
#pragma unroll
      for (int s = 0; s < 4; ++s) bad = bad || (((unsigned)__builtin_amdgcn_readlane((int)wfl[1], 4 * s) & META_SV) && min12(wp[s]) == 0u);
      if (LIKELY(!__any(bad))) break;
      if (rt() - t0 > DEADLINE) { __hip_atomic_store((GAS unsigned*)tmo, call, __ATOMIC_RELAXED, __HIP_MEMORY_SCOPE_AGENT); okw = false; break; }
    }
#pragma unroll
    for (int s = 0; s < 4; ++s) {
      const unsigned meta = (unsigned)__builtin_amdgcn_readlane((int)wfl[1], 4 * s);
      if (LIKELY((meta & 0x80000f00u) == 0u)) {
        wf[s][0] = unpack_frag<0, false>(wp[s], meta, 0u, 0u, 0, lane);
        wf[s][1] = unpack_frag<1, false>(wp[s], meta, 0u, 0u, 0, lane);
        wf[s][2] = unpack_frag<2, false>(wp[s], meta, 0u, 0u, 0, lane);
        wf[s][3] = unpack_frag<3, false>(wp[s], meta, 0u, 0u, 0, lane);
      } else if (meta & META_RAW) {
        wf[s][0] = wp[s][0]; wf[s][1] = wp[s][1]; wf[s][2] = wp[s][2]; wf[s][3] = ldp(soff + s * 4096, 3);
      } else {
        wf[s][0] = unpack_frag<0, true>(wp[s], meta, wfl[2], wfl[3], 4 * s, lane);
        wf[s][1] = unpack_frag<1, true>(wp[s], meta, wfl[2], wfl[3], 4 * s, lane);
        wf[s][2] = unpack_frag<2, true>(wp[s], meta, wfl[2], wfl[3], 4 * s, lane);
        wf[s][3] = unpack_frag<3, true>(wp[s], meta, wfl[2], wfl[3], 4 * s, lane);
      }
    }
  }
  f32x4 acc[2];
  auto mm = [&](int ks, u32x4 xf) {
#pragma unroll
    for (int jj = 0; jj < 2; ++jj)
      acc[jj] = __builtin_amdgcn_mfma_f32_16x16x32_bf16(__builtin_bit_cast(bf16x8, wf[ks >> 1][(ks & 1) * 2 + jj]), __builtin_bit_cast(bf16x8, xf), acc[jj], 0, 0, 0);
  };
  int pw[2];
  {
    const int m = lane & 15, qd = lane >> 4;
    const int sw = ((m & 1) << 2) | (m & 2) | ((m >> 2) & 1);
#pragma unroll
    for (int jj = 0; jj < 2; ++jj) pw[jj] = wv * 2048 + m * 128 + (((4 * jj + qd) ^ sw) * 16);
  }
  bool scr_free = false;
  auto finish = [&](int it, bool good) {
    if (UNLIKELY(!good)) {
      asm volatile("; poison" ::: "memory");
      const float qn = __builtin_nanf("");
#pragma unroll
      for (int jj = 0; jj < 2; ++jj) acc[jj] = f32x4{qn, qn, qn, qn};
    }
    if (UNLIKELY(it >= WSCR_IT && !scr_free)) {
      while (__hip_atomic_load(&ready[8], __ATOMIC_RELAXED, __HIP_MEMORY_SCOPE_WORKGROUP) < 8u) __builtin_amdgcn_s_sleep(1);
      scr_free = true;
    }
    char* pb = lds + it * 16384;
#pragma unroll
    for (int jj = 0; jj < 2; ++jj) *(f32x4*)(pb + pw[jj]) = acc[jj];
    if (lane == 0) __hip_atomic_fetch_add(&ready[it], 1u, __ATOMIC_RELEASE, __HIP_MEMORY_SCOPE_WORKGROUP);
  };
  if (own0) {
#pragma unroll
    for (int jj = 0; jj < 2; ++jj) acc[jj] = f32x4{0.f, 0.f, 0.f, 0.f};
#pragma unroll
    for (int s = 0; s < 8; ++s) mm(s, x0[s >> 2][s & 3]);
    finish(0, okw);
  }
  auto step = [&](int it, u32x4 (&cur)[2][3], u32x4& cv, u32x4 (&nxt)[2][3], u32x4& ncv) {
#pragma unroll
    for (int h = 0; h < 2; ++h)
#pragma unroll
      for (int p = 0; p < 3; ++p) PIN(cur[h][p]);
    PIN(cv);
    bool cok = true;
    {
      const bool rdy0 = lines_ready(cv, xline, first, call);
      const unsigned m0 = (unsigned)__builtin_amdgcn_readlane((int)cv[1], 0), m1 = (unsigned)__builtin_amdgcn_readlane((int)cv[1], 4);
      const bool have = rdy0 && (m0 & m1 & META_SV) && __all(umin_(min12(cur[0]), min12(cur[1])) != 0u);
      if (!have) {
        if (!rdy0) cok = wait_lines(rs, xoff(code_of(it)), xline, first, 8, call, tmo, cv);
        const unsigned n0 = (unsigned)__builtin_amdgcn_readlane((int)cv[1], 0), n1 = (unsigned)__builtin_amdgcn_readlane((int)cv[1], 4);
        const u64 t0 = rt();
        for (;;) {
          load_x(cur, code_of(it));
          if (!__any(((n0 & META_SV) && min12(cur[0]) == 0u) || ((n1 & META_SV) && min12(cur[1]) == 0u))) break;
          if (rt() - t0 > DEADLINE) { __hip_atomic_store((GAS unsigned*)tmo, call, __ATOMIC_RELAXED, __HIP_MEMORY_SCOPE_AGENT); cok = false; break; }
        }
      }
    }
    cok = cok && (unsigned)__builtin_amdgcn_readlane((int)cv[0], 8) != call;
    const unsigned meta0 = (unsigned)__builtin_amdgcn_readlane((int)cv[1], 0), meta1 = (unsigned)__builtin_amdgcn_readlane((int)cv[1], 4);
    __builtin_amdgcn_fence(__ATOMIC_ACQUIRE, "wavefront");
    issue(nxt, ncv, it + 1);
    __builtin_amdgcn_sched_barrier(0);
#pragma unroll
    for (int jj = 0; jj < 2; ++jj) acc[jj] = f32x4{0.f, 0.f, 0.f, 0.f};
    if (LIKELY(((meta0 | meta1) & 0x80000f00u) == 0u)) {
#pragma unroll
      for (int h = 0; h < 2; ++h) {
        const unsigned meta = h ? meta1 : meta0;
        mm(4 * h + 0, unpack_frag<0, false>(cur[h], meta, 0u, 0u, 0, lane));
        mm(4 * h + 1, unpack_frag<1, false>(cur[h], meta, 0u, 0u, 0, lane));
        mm(4 * h + 2, unpack_frag<2, false>(cur[h], meta, 0u, 0u, 0, lane));
        mm(4 * h + 3, unpack_frag<3, false>(cur[h], meta, 0u, 0u, 0, lane));
      }
    } else {
#pragma unroll
      for (int h = 0; h < 2; ++h) {
        const unsigned meta = h ? meta1 : meta0;
        if (UNLIKELY(meta & META_RAW)) {
          mm(4 * h, cur[h][0]);
          cur[h][0] = ldp(xdata(code_of(it)) + h * 4096, 3);
          mm(4 * h + 1, cur[h][1]); mm(4 * h + 2, cur[h][2]);
          PIN(cur[h][0]);
          mm(4 * h + 3, cur[h][0]);
        } else {
          mm(4 * h + 0, unpack_frag<0, true>(cur[h], meta, cv[2], cv[3], 4 * h, lane));
          mm(4 * h + 1, unpack_frag<1, true>(cur[h], meta, cv[2], cv[3], 4 * h, lane));
          mm(4 * h + 2, unpack_frag<2, true>(cur[h], meta, cv[2], cv[3], 4 * h, lane));
          mm(4 * h + 3, unpack_frag<3, true>(cur[h], meta, cv[2], cv[3], 4 * h, lane));
        }
      }
    }
    finish(it, cok && okw);
  };
  for (int it = it0; it < nblk; it += 2) {
    step(it, bufA, cvA, bufB, cvB);
    if (it + 1 < nblk) step(it + 1, bufB, cvB, bufA, cvA);
  }

}

}

extern "C" void kernel_launch(int rank, int nranks, ncclComm_t comm, hipStream_t stream, void* const* d_in, const int* in_sizes,
                              int n_in, void* d_out, int out_size, const int* in_dtypes, int out_dtype, void* const* d_out_all,
                              void* const* d_ws_all, size_t ws_bytes) {
  using namespace agk;
  (void)comm; (void)in_dtypes; (void)out_dtype; (void)d_out_all;
  if (nranks != NRANK || n_in < 2 || rank < 0 || rank >= NRANK || ws_bytes < WS_NEED || in_sizes[0] != MLOC * KDIM ||
      in_sizes[1] != KDIM * NLOC || out_size != MDIM * NLOC) {
    fprintf(stderr, "kernel_launch: unexpected problem shape\n");
    return;
  }
  const int type = rank & 1, idx = rank >> 1;
  Args a;
  memset(&a, 0, sizeof a);
  for (int link = 0; link < 7; ++link) {
    const bool same = link < 3;
    const int didx = same ? link + (link >= idx ? 1 : 0) : link - 3;
    a.link_ws[link] = (char*)d_ws_all[2 * didx + (same ? type : 1 - type)];
  }
  a.link_ws[7] = (char*)d_ws_all[rank];
  for (int cg = 0; cg < 8; ++cg) {
    char* const dws = (char*)d_ws_all[2 * (cg >> 1) + (1 - type)];
    a.oflag[cg] = dws + OFF_OFLAG + (size_t)(idx * 128 + (cg & 1)) * 128;
    a.otile[cg] = dws + OFF_OSTAGE + (size_t)(idx * 128 + (cg & 1)) * 1024;
  }
  auto make_list = [](int ty, int ix, bool with_bcast, int* code, int* row) -> int {
    int n = 0;
    for (int b = 0; b < 16; ++b)
      if (b < P || (with_bcast && b >= 16 - NB)) { code[n] = 0x40 | b; row[n] = (2 * ix + ty) * 16 + b; ++n; }
    for (int st = 0; st < 32; ++st) {
      if (st < SB && (st < P || with_bcast))
        for (int ia = 0; ia < 4; ++ia)
          if (ia != ix) { code[n] = (ia < ix ? ia : ia - 1) * SB + st; row[n] = (2 * ia + ty) * 16 + (st < P ? st : st + PO); ++n; }
      const int k = st - WSTEPS;
      if (k >= 0 && k < OB && (k < PO || with_bcast))
        for (int ia = 0; ia < 4; ++ia) { code[n] = 3 * SB + ia * OB + k; row[n] = (2 * ia + 1 - ty) * 16 + P + k; ++n; }
    }
    return n;
  };
  int code[80], row[80];
  for (int o = 0; o < 2; ++o) {
    const int n = make_list(type, idx, o == 0, code, row);
    for (int q = 0; q < n; ++q) {
      const int set = q % NSET, i = q / NSET;
      a.codes[o][set] |= (u64)code[q] << (8 * i);
      a.rows[o][set] |= (u64)row[q] << (8 * i);
      a.codes[o][set] += (u64)1 << 56;
    }
  }
  for (int src = 0; src < 4; ++src) {
    const int n = make_list(1 - type, src, false, code, row);
    for (int q = 0; q < n; ++q) a.orow[src][q] = (unsigned char)row[q];
  }
  a.myws = (char*)d_ws_all[rank];
  a.myout = (char*)d_out;
  a.x = (const char*)d_in[0];
  a.w = (const char*)d_in[1];
  a.rank = rank;
  hipLaunchKernelGGL(ag_gemm_kernel, dim3(NWG), dim3(NTHR), 0, stream, a);
}
```
